```python
import numpy as np

kernel_name = "gpu_ag_gemm_m4096_k4096_n16384_e5m2e5m2_dqrelu"


ABSMAX_RTOL = 0.05

COMPUTE_DTYPE = "float32"
EPILOGUE = "dequant_relu"
IN_QUANT = None
OUT_QUANT = None
IN_FP8 = "e5m2e5m2"
IN_DTYPES = {"x": "e5m2", "w_mat": "e5m2",
             "scale_x": "float32", "scale_w": "float32"}
WORLD_SIZE = 8

def _snap_fp8(x, mant_bits, max_abs, min_normal):
    a = np.abs(x.astype(np.float64))
    _m, _e = np.frexp(a)
    q = np.where(a >= min_normal, np.ldexp(1.0, _e - 1 - mant_bits),
                 min_normal * 2.0 ** -mant_bits)
    snapped = np.minimum(np.round(a / q) * q, max_abs)
    return (np.sign(x) * snapped).astype(np.float32)

def setup_inputs():

    x = _snap_fp8(np.random.randn(4096, 4096).astype(np.float32), 2, 57344.0, 2.0 ** -14)
    w_mat = _snap_fp8(np.random.randn(4096, 16384).astype(np.float32), 2, 57344.0, 2.0 ** -14)
    scale_x = (0.02 * (1.0 + np.random.rand(1))).astype(np.float32)
    scale_w = (0.02 * (1.0 + np.random.rand(1))).astype(np.float32)
    return {"x": x, "w_mat": w_mat, "scale_x": scale_x, "scale_w": scale_w}

def reference(x, w_mat, scale_x, scale_w):
    acc = (x @ w_mat).astype(np.float32)
    y = (acc * (scale_x * scale_w)).astype(np.float32)
    return np.maximum(y, 0.0).astype(np.float32)
```

```cpp
#include <hip/hip_runtime.h>
#include <rccl/rccl.h>
#include <cstdio>
#include <cstdint>

#define HIP_CHECK(expr) do { hipError_t _e = (expr); if (_e != hipSuccess) { \
    fprintf(stderr, "HIP error %s at %s:%d\n", hipGetErrorString(_e), __FILE__, __LINE__); \
    return; } } while (0)

namespace agk {

#define LAS __attribute__((address_space(3)))
typedef int i32x8 __attribute__((ext_vector_type(8)));
typedef int i32x4 __attribute__((ext_vector_type(4)));
typedef int i32x2 __attribute__((ext_vector_type(2)));
typedef unsigned u32x4 __attribute__((ext_vector_type(4)));
typedef float f32x4 __attribute__((ext_vector_type(4)));

constexpr int NR = 8;
constexpr int KT = 4096, NL = 2048, ML = 512;
constexpr int BM = 128, BN = 256, BK = 128;
constexpr int NSTEP = KT / BK;
constexpr int XB = BM * BK;
constexpr int WB = BK * BN;
constexpr int STAGE = XB + WB;
constexpr int NSLOT = 3;
constexpr int LDS_CTRL = NSLOT * STAGE;
constexpr int LDS_BYTES = LDS_CTRL + 64;
constexpr int NWG = 256;
constexpr unsigned SHARD_BYTES = ML * KT;
constexpr unsigned W_BYTES = KT * NL;

constexpr size_t WS_CNT = 0;
constexpr size_t WS_TMO = WS_CNT + NWG * 128;
constexpr size_t WS_PROG = WS_TMO + 128;
constexpr size_t WS_DBG = 512u << 10;
constexpr size_t WS_STAGE = 1u << 20;
constexpr size_t WS_NEEDED = WS_STAGE + 2ull * NR * SHARD_BYTES;
constexpr unsigned long long DEADLINE_TICKS = 20000000ull;

#define SK_START {0, 0, 0, 0}
#define SK_RATE {16, 16, 16, 16}
struct Sched {
    unsigned char order[128];
    unsigned char pos[128];
};
constexpr Sched make_sched() {
    Sched s{};
    const int start[4] = SK_START;
    const int rate[4] = SK_RATE;
    int nxt[4] = {0, 0, 0, 0};
    for (int p = 0; p < 128; ++p) {
        int best = -1, bk = 0;
        for (int g = 0; g < 4; ++g) {
            if (nxt[g] >= 32) continue;
            const int key = start[g] + nxt[g] * rate[g];
            if (best < 0 || key < bk) { best = g; bk = key; }
        }
        s.order[p] = (unsigned char)(best * 32 + nxt[best]);
        s.pos[best * 32 + nxt[best]] = (unsigned char)p;
        ++nxt[best];
    }
    return s;
}
__constant__ const Sched c_sched = make_sched();

struct Params {
    const uint8_t* x;
    const uint8_t* w;
    const float* sx;
    const float* sw;
    float* out;
    uint8_t* ws[NR];
    int rank;
    int pad;
};

#define WAIT_V(n) asm volatile("s_waitcnt vmcnt(" #n ")" ::: "memory")
#define WAIT_L0() do { asm volatile("" ::: "memory"); __builtin_amdgcn_s_waitcnt(0xC07F); asm volatile("" ::: "memory"); } while (0)
#define BAR() do { asm volatile("" ::: "memory"); __builtin_amdgcn_s_barrier(); asm volatile("" ::: "memory"); } while (0)
#define SCHED() __builtin_amdgcn_sched_barrier(0)

__device__ __forceinline__ i32x4 make_rsrc(const void* p, unsigned bytes) {
    const unsigned long long a = (unsigned long long)p;
    i32x4 r;
    r.x = __builtin_amdgcn_readfirstlane((int)(unsigned)a);
    r.y = __builtin_amdgcn_readfirstlane((int)((unsigned)(a >> 32) & 0xffffu));
    r.z = __builtin_amdgcn_readfirstlane((int)bytes);
    r.w = 0x00020000;
    return r;
}
__device__ __forceinline__ void dma16(unsigned lds, unsigned voff, i32x4 rsrc, unsigned soff) {
    asm volatile("s_mov_b32 m0, %0\n\ts_nop 0\n\tbuffer_load_dwordx4 %1, %2, %3 offen lds"
                 :: "s"(lds), "v"(voff), "s"(rsrc), "s"(soff) : "memory");
}
__device__ __forceinline__ void dma4_sys(unsigned lds, unsigned voff, i32x4 rsrc) {
    asm volatile("s_mov_b32 m0, %0\n\ts_nop 0\n\tbuffer_load_dword %1, %2, 0 offen sc0 sc1 lds"
                 :: "s"(lds), "v"(voff), "s"(rsrc) : "memory");
}
__device__ __forceinline__ void ld16_asm(u32x4& d, unsigned voff, i32x4 rsrc, unsigned soff) {
    asm volatile("buffer_load_dwordx4 %0, %1, %2, %3 offen" : "=v"(d) : "v"(voff), "s"(rsrc), "s"(soff) : "memory");
}
__device__ __forceinline__ void st16_sys(const u32x4& d, unsigned voff, i32x4 rsrc, unsigned soff) {
    asm volatile("buffer_store_dwordx4 %0, %1, %2, %3 offen sc0 sc1\n\ts_nop 1"
                 :: "v"(d), "v"(voff), "s"(rsrc), "s"(soff) : "memory");
}

struct Lane {
    unsigned vx0, vx1;
    unsigned vw0, vw1;
    unsigned ax;
    unsigned aw;
};
struct Ctx {
    i32x4 rx, rw;
    unsigned kstride;
    unsigned ldsx, ldsw;
    unsigned lds0;
};
template <int NBW> struct Frags { i32x8 x[4]; i32x8 w[NBW]; };
__device__ __forceinline__ void put4(i32x8& d, int i, i32x4 v) { d[4 * i] = v.x; d[4 * i + 1] = v.y; d[4 * i + 2] = v.z; d[4 * i + 3] = v.w; }
__device__ __forceinline__ void put2(i32x8& d, int i, i32x2 v) { d[2 * i] = v.x; d[2 * i + 1] = v.y; }

template <int NBW>
__device__ __forceinline__ void lane_setup(Lane& L, Ctx& c, unsigned lds0, int wid, int lane, unsigned rstride, unsigned col0) {
    const unsigned r0 = wid * 16 + (lane >> 3), pp = lane & 7;
    L.vx0 = r0 * rstride + 16 * (pp ^ (unsigned)(lane >> 4));
    L.vx1 = (r0 + 8) * rstride + 16 * (pp ^ (unsigned)(4 + (lane >> 4)));
    const int wm = wid & 1, wn = wid >> 1;
    const unsigned m = lane & 15, gq = lane >> 4;
    L.ax = (wm * 64 + m) * BK + 16 * (gq ^ (m >> 1));
    const unsigned q = m >> 1, h = m & 1;
    if (NBW == 4) {
        const unsigned k0 = wid * 16 + (lane >> 4), pw = lane & 15, hi = (wid & 1) << 3;
        L.vw0 = k0 * NL + col0 + 16 * (pw ^ ((unsigned)(lane >> 4) | hi));
        L.vw1 = (k0 + 4) * NL + col0 + 16 * (pw ^ ((unsigned)(4 + (lane >> 4)) | hi));
        L.aw = (16 * gq + q) * 256 + 16 * ((unsigned)(wn * 4) ^ (q | ((gq & 1) << 3))) + 8 * h;
        c.ldsw = lds0 + XB + wid * 4096;
    } else {
        const unsigned k0 = wid * 16 + (lane >> 3), pw = lane & 7, hi = (wid & 1) << 2;
        const unsigned f = (((unsigned)lane >> 4) & 3u) | hi;
        L.vw0 = k0 * NL + col0 + 16 * (pw ^ f);
        L.vw1 = (k0 + 8) * NL + col0 + 16 * (pw ^ f);
        L.aw = (16 * gq + q) * 128 + 16 * ((unsigned)(wn * 2) ^ ((q >> 1) | ((gq & 1) << 2))) + 8 * h;
        c.ldsw = lds0 + XB + wid * 2048;
    }
    c.ldsx = lds0 + wid * 2048;
    c.lds0 = lds0;
}

template <int NBW>
__device__ __forceinline__ void issue_half(const Ctx& c, const Lane& L, int stage, unsigned soff, int half) {
    const unsigned sw_off = (unsigned)stage * (BK * NL);
    if (half == 0) {
        const unsigned sx_off = (unsigned)stage * c.kstride;
        dma16(c.ldsx + soff, L.vx0, c.rx, sx_off);
        dma16(c.ldsx + soff + 1024, L.vx1, c.rx, sx_off);
        if (NBW == 4) dma16(c.ldsw + soff, L.vw0, c.rw, sw_off);
    } else if (NBW == 4) {
        dma16(c.ldsw + soff + 1024, L.vw1, c.rw, sw_off);
        dma16(c.ldsw + soff + 2048, L.vw0, c.rw, sw_off + 8 * NL);
        dma16(c.ldsw + soff + 3072, L.vw1, c.rw, sw_off + 8 * NL);
    } else {
        dma16(c.ldsw + soff, L.vw0, c.rw, sw_off);
        dma16(c.ldsw + soff + 1024, L.vw1, c.rw, sw_off);
    }
}
template <int NBW>
__device__ __forceinline__ void issue_stage(const Ctx& c, const Lane& L, int stage, unsigned soff) {
    issue_half<NBW>(c, L, stage, soff, 0);
    issue_half<NBW>(c, L, stage, soff, 1);
}
__device__ __forceinline__ i32x4 read_x1(unsigned ax0, int mb, int i) {
    return *(LAS i32x4*)(size_t)((i ? (ax0 ^ 64u) : ax0) + mb * 2048);
}
template <int NBW>
__device__ __forceinline__ i32x2 read_w1(unsigned aw, int nb, int i) {
    constexpr int RB = 64 * NBW;
    const unsigned a = aw ^ (unsigned)(nb << 4);
    return __builtin_amdgcn_ds_read_tr8_b64_v2i32((LAS i32x2*)(size_t)(a + ((i >> 1) * 64 + (i & 1) * 8) * RB));
}
template <int NBW>
__device__ __forceinline__ void read_part(const Ctx& c, const Lane& L, unsigned soff, Frags<NBW>& f, int part) {
    const unsigned sb = c.lds0 + soff;
    if (part == 0) {
#pragma unroll
        for (int i = 0; i < 4; ++i) put2(f.w[0], i, read_w1<NBW>(sb + XB + L.aw, 0, i));
#pragma unroll
        for (int mb = 0; mb < 4; ++mb) { put4(f.x[mb], 0, read_x1(sb + L.ax, mb, 0)); put4(f.x[mb], 1, read_x1(sb + L.ax, mb, 1)); }
    } else {
#pragma unroll
        for (int nb = 1; nb < NBW; ++nb)
#pragma unroll
            for (int i = 0; i < 4; ++i) put2(f.w[nb], i, read_w1<NBW>(sb + XB + L.aw, nb, i));
    }
}
template <int NBW>
__device__ __forceinline__ void mma1(f32x4 (&acc)[NBW][4], const Frags<NBW>& f, int nb, int mb) {
    acc[nb][mb] = __builtin_amdgcn_mfma_scale_f32_16x16x128_f8f6f4(f.w[nb], f.x[mb], acc[nb][mb], 1, 1, 0, 0, 0, 0);
}
template <int NBW>
__device__ __forceinline__ void mma_group(f32x4 (&acc)[NBW][4], const Frags<NBW>& f, int nb) {
#pragma unroll
    for (int mb = 0; mb < 4; ++mb) mma1<NBW>(acc, f, nb, mb);
}
template <int NBW>
__device__ __forceinline__ void read_nth(unsigned ax, unsigned aw, Frags<NBW>& f, int r) {
    if (r < 8) put4(f.x[r >> 1], r & 1, read_x1(ax, r >> 1, r & 1));
    else put2(f.w[(r - 8) >> 2], (r - 8) & 3, read_w1<NBW>(aw, (r - 8) >> 2, (r - 8) & 3));
}
template <int NBW>
__device__ __forceinline__ void dma_nth(const Ctx& c, const Lane& L, i32x4 rx, i32x4 rw, unsigned sx_off, unsigned sw_off, unsigned soff, int d) {
    if (d == 0) dma16(c.ldsx + soff, L.vx0, rx, sx_off);
    else if (d == 1) dma16(c.ldsx + soff + 1024, L.vx1, rx, sx_off);
    else if (NBW == 4) {
        if (d == 2) dma16(c.ldsw + soff, L.vw0, rw, sw_off);
        else if (d == 3) dma16(c.ldsw + soff + 1024, L.vw1, rw, sw_off);
        else if (d == 4) dma16(c.ldsw + soff + 2048, L.vw0, rw, sw_off + 8 * NL);
        else dma16(c.ldsw + soff + 3072, L.vw1, rw, sw_off + 8 * NL);
    } else {
        if (d == 2) dma16(c.ldsw + soff, L.vw0, rw, sw_off);
        else dma16(c.ldsw + soff + 1024, L.vw1, rw, sw_off);
    }
}
template <int NBW, bool DMA, bool COMP>
__device__ __forceinline__ void step_body(const Ctx& c, const Lane& L, f32x4 (&acc)[NBW][4], const Frags<NBW>& fc, Frags<NBW>& fn,
                                          unsigned nxt, int stage, unsigned isl, bool live) {
    const unsigned ax = c.lds0 + nxt + L.ax, aw = c.lds0 + nxt + XB + L.aw;
    const unsigned sx_off = (unsigned)stage * c.kstride, sw_off = (unsigned)stage * (BK * NL);
    i32x4 rx = c.rx, rw = c.rw;
    rx.z = live ? rx.z : 0;
    rw.z = live ? rw.z : 0;
    constexpr int NM = NBW * 4, NRD = NBW * 4 + 8, ND = NBW + 2;
#pragma unroll
    for (int i = 0; i < NM; ++i) {
        if (COMP) mma1<NBW>(acc, fc, i >> 2, i & 3);
        if (COMP && 2 * i < NRD) read_nth<NBW>(ax, aw, fn, 2 * i);
        if (COMP && 2 * i + 1 < NRD) read_nth<NBW>(ax, aw, fn, 2 * i + 1);
        if (NBW == 4) { if (DMA && i >= 3 && (i & 1) && (i - 3) / 2 < ND) dma_nth<NBW>(c, L, rx, rw, sx_off, sw_off, isl, (i - 3) / 2); }
        else { if (DMA && i >= 2 && i - 2 < ND) dma_nth<NBW>(c, L, rx, rw, sx_off, sw_off, isl, i - 2); }
        SCHED();
    }
}
__device__ __forceinline__ unsigned next_slot(unsigned soff) { return soff == 2u * STAGE ? 0u : soff + STAGE; }

#define OUT_ST 2
template <int NBW>
__device__ __forceinline__ void store_tile(const f32x4 (&acc)[NBW][4], float* out, int row0, int col0, float scale, bool bad) {
    int tz = threadIdx.x;
    asm volatile("" : "+v"(tz));
    const int ln = tz & 63, wv = tz >> 6;
    const int wm = wv & 1, wn = wv >> 1;
    float* o = out + (size_t)(row0 + wm * 64 + (ln & 15)) * NL + (col0 + wn * (16 * NBW) + 4 * (ln >> 4));
    const float qnan = __builtin_nanf("");
#pragma unroll
    for (int mb = 0; mb < 4; ++mb)
#pragma unroll
        for (int nb = 0; nb < NBW; ++nb) {
            f32x4 v = acc[nb][mb] * scale;
            v.x = v.x < 0.f ? 0.f : v.x;
            v.y = v.y < 0.f ? 0.f : v.y;
            v.z = v.z < 0.f ? 0.f : v.z;
            v.w = v.w < 0.f ? 0.f : v.w;
            if (bad) v = (f32x4){qnan, qnan, qnan, qnan};
            float* dst = o + (size_t)mb * 16 * NL + nb * 16;
            asm volatile("global_store_dwordx4 %0, %1, off sc1\n\ts_nop 1" :: "v"(dst), "v"(v) : "memory");
        }
}

#define AGK_MODE 0
#define PUSH_D 10

__global__ void __launch_bounds__(512, 2) ag_gemm_fused(Params p) {
    extern __shared__ __attribute__((aligned(16))) char smem[];
    constexpr bool dbg_nowait = (AGK_MODE & 1) != 0, dbg_nopush = (AGK_MODE & 2) != 0;
    constexpr bool dbg_nodma = (AGK_MODE & 4) != 0, dbg_nocomp = (AGK_MODE & 8) != 0, dbg_nolocal = (AGK_MODE & 16) != 0;
    const int tid = threadIdx.x, lane = tid & 63;
    const int wid = __builtin_amdgcn_readfirstlane(tid >> 6);
    const int b = blockIdx.x;
    const int ct = b & 7, rt = b >> 3;
    const int sh = rt >> 2, g = rt & 3;
    const int rank = p.rank;
    uint8_t* wsl = p.ws[rank];
    unsigned* cntp = (unsigned*)(wsl + WS_CNT + (size_t)b * 128);

    if (sh == 0) {
        if (ct == NR - 1 || dbg_nopush) return;
        const int peer = (rank + 1 + ct) & (NR - 1);
        uint8_t* wsp = p.ws[peer];
        const i32x4 rs = make_rsrc(p.x, SHARD_BYTES);
        const unsigned vs = (wid * 16 + (lane >> 3)) * KT + (lane & 7) * 16;
        const unsigned vd = wid * 2048 + lane * 16;
        unsigned cnt_old;
        asm volatile("global_load_dword %0, %1, off sc1" : "=v"(cnt_old) : "v"(cntp) : "memory");
        const int ordv = (int)c_sched.order[4 * (lane & 31) + g];
        u32x4 pb[4][2];
        auto push_load = [&](int t, u32x4 (&d)[2]) {
            const unsigned u = (unsigned)__builtin_amdgcn_readlane(ordv, t);
            const unsigned so = (u >> 5) * (BM * KT) + (u & 31u) * BK;
            ld16_asm(d[0], vs, rs, so);
            ld16_asm(d[1], vs, rs, so + 8 * KT);
        };
        push_load(0, pb[0]);
        push_load(1, pb[1]);
        push_load(2, pb[2]);
        push_load(3, pb[3]);
        WAIT_V(8);
        asm volatile("" : "+v"(cnt_old));
        const unsigned call = (unsigned)__builtin_amdgcn_readfirstlane(cnt_old) + 1u;
        const unsigned pbase = call * 64u;
        const i32x4 rd = make_rsrc(wsp + WS_STAGE + ((size_t)(call & 1u) * NR + rank) * SHARD_BYTES, SHARD_BYTES);
        unsigned* flag = (unsigned*)(wsp + WS_PROG + (size_t)rank * 128) + g;
        auto iter = [&](int t, u32x4 (&d)[2]) {
            if (t < 4) {
                if (t == 0) { WAIT_V(6); } else if (t == 1) { WAIT_V(8); } else if (t == 2) { WAIT_V(10); } else { WAIT_V(12); }
            }
            const unsigned u = (unsigned)__builtin_amdgcn_readlane(ordv, t);
            st16_sys(d[0], vd, rd, u * XB);
            st16_sys(d[1], vd, rd, u * XB + 1024);
            if (t + 4 < NSTEP) {
                push_load(t + 4, d);
                if (t >= 2) { WAIT_V(10); }
            } else {
                if (t == NSTEP - 4) { WAIT_V(8); } else if (t == NSTEP - 3) { WAIT_V(6); } else { WAIT_V(4); }
            }
            if (t >= 2) {
                BAR();
                if (tid == 0)
                    __hip_atomic_store(flag, pbase + (unsigned)(t - 1), __ATOMIC_RELAXED, __HIP_MEMORY_SCOPE_SYSTEM);
            }
        };
        for (int t = 0; t < NSTEP; t += 4) {
            iter(t, pb[0]);
            iter(t + 1, pb[1]);
            iter(t + 2, pb[2]);
            iter(t + 3, pb[3]);
        }
        WAIT_V(2);
        BAR();
        if (tid == 0) __hip_atomic_store(flag, pbase + (unsigned)(NSTEP - 1), __ATOMIC_RELAXED, __HIP_MEMORY_SCOPE_SYSTEM);
        WAIT_V(0);
        BAR();
        if (tid == 0) {
            __hip_atomic_store(flag, pbase + (unsigned)NSTEP, __ATOMIC_RELAXED, __HIP_MEMORY_SCOPE_SYSTEM);
            __hip_atomic_store(cntp, call, __ATOMIC_RELAXED, __HIP_MEMORY_SCOPE_AGENT);
        }
        return;
    }

    const unsigned lds0 = (unsigned)(uintptr_t)(LAS char*)smem;
    const float scale = p.sx[0] * p.sw[0];
    const unsigned call = (unsigned)__builtin_amdgcn_readfirstlane(
                              __hip_atomic_load(cntp, __ATOMIC_RELAXED, __HIP_MEMORY_SCOPE_AGENT)) + 1u;
    if (wid == 0) __builtin_amdgcn_fence(__ATOMIC_ACQUIRE, "");

    if (g >= 2 && sh <= 4 && !dbg_nolocal) {
        const int h = (sh - 1) * 2 + (g - 2);
        const int lg = h >> 1, half = h & 1;
        Lane L; Ctx c;
        lane_setup<2>(L, c, lds0, wid, lane, KT, ct * BN + half * 128);
        c.rw = make_rsrc(p.w, W_BYTES);
        c.rx = make_rsrc(p.x + (size_t)lg * BM * KT, BM * KT);
        c.kstride = BK;
        f32x4 acc[2][4];
#pragma unroll
        for (int i = 0; i < 2; ++i)
#pragma unroll
            for (int j = 0; j < 4; ++j) acc[i][j] = (f32x4){0.f, 0.f, 0.f, 0.f};
        Frags<2> fa, fb;
        issue_stage<2>(c, L, 0, 0);
        issue_stage<2>(c, L, 1, STAGE);
        issue_stage<2>(c, L, 2, 2 * STAGE);
        WAIT_V(8);
        BAR();
        read_part<2>(c, L, 0, fa, 0);
        read_part<2>(c, L, 0, fa, 1);
        unsigned cur = 0;
        auto step = [&](int s, Frags<2>& fc, Frags<2>& fn) {
            const unsigned nxt = next_slot(cur);
            WAIT_L0();
            WAIT_V(4);
            BAR();
            SCHED();
            step_body<2, true, true>(c, L, acc, fc, fn, nxt, s + 3, cur, s + 3 < NSTEP);
            cur = nxt;
        };
        for (int s = 0; s < NSTEP; s += 2) {
            step(s, fa, fb);
            step(s + 1, fb, fa);
        }
        WAIT_V(0);
        store_tile<2>(acc, p.out, rank * ML + lg * BM, ct * BN + half * 128, scale, false);
        WAIT_L0();
        BAR();
    }

    {
        const int src = (rank + sh) & (NR - 1);
        Lane L; Ctx c;
        lane_setup<4>(L, c, lds0, wid, lane, BK, ct * BN);
        c.rw = make_rsrc(p.w, W_BYTES);
        c.rx = make_rsrc(wsl + WS_STAGE + ((size_t)(call & 1u) * NR + src) * SHARD_BYTES + (size_t)g * NSTEP * XB, NSTEP * XB);
        c.kstride = XB;
        volatile LAS unsigned* ctrl = (volatile LAS unsigned*)(LAS char*)(smem + LDS_CTRL);
        unsigned* tmo = (unsigned*)(wsl + WS_TMO);
        const unsigned pbase = call * 64u;
        unsigned* prog = (unsigned*)(wsl + WS_PROG + (size_t)src * 128);
        const i32x4 rf = make_rsrc(prog, 16);
        const unsigned posv = c_sched.pos[g * 32 + (lane & 31)];
        const unsigned pj = posv & 3u;
        const unsigned need = pbase + (posv >> 2) + 1u;
        f32x4 acc[4][4];
#pragma unroll
        for (int i = 0; i < 4; ++i)
#pragma unroll
            for (int j = 0; j < 4; ++j) acc[i][j] = (f32x4){0.f, 0.f, 0.f, 0.f};
        Frags<4> fa, fb;
        int R = dbg_nowait ? NSTEP : 0;
        int ni = 0;
        unsigned isl = 0;
        if (tid < 8) ctrl[tid] = pbase;
        auto ready_count = [&](unsigned v) -> int {
            const unsigned long long m = __builtin_amdgcn_ballot_w64((int)(v - need) >= 0);
            const unsigned lo = (unsigned)m;
            return lo == 0xffffffffu ? NSTEP : __builtin_ctz(~lo);
        };
        auto slow_fill = [&](int st) {
            if (wid == 0) {
                if (R <= st) {
                    unsigned long long t0 = __builtin_amdgcn_s_memrealtime();
                    for (unsigned it = 1;; ++it) {
                        const unsigned v = __hip_atomic_load(prog + pj, __ATOMIC_RELAXED, __HIP_MEMORY_SCOPE_SYSTEM);
                        const int r = ready_count(v);
                        if (r > st) { R = r; break; }
                        if ((it & 15u) == 0u) {
                            const unsigned tv = (unsigned)__builtin_amdgcn_readfirstlane(
                                __hip_atomic_load(tmo, __ATOMIC_RELAXED, __HIP_MEMORY_SCOPE_AGENT));
                            if (tv == call || __builtin_amdgcn_s_memrealtime() - t0 > DEADLINE_TICKS) {
                                if (lane == 0) __hip_atomic_store(tmo, call, __ATOMIC_RELAXED, __HIP_MEMORY_SCOPE_AGENT);
                                R = NSTEP;
                                break;
                            }
                        }
                        __builtin_amdgcn_s_sleep(1);
                    }
                }
                if (lane == 0) ctrl[8] = (unsigned)R;
                WAIT_V(0);
            }
            WAIT_L0();
            BAR();
            R = __builtin_amdgcn_readfirstlane((int)ctrl[8]);
            const int lim = R < st + 3 ? R : st + 3;
            while (ni < lim) {
                if (!dbg_nodma) issue_stage<4>(c, L, ni, isl);
                isl = next_slot(isl);
                ++ni;
            }
            const int ahead = ni - st - 1;
            if (ahead == 0) { WAIT_V(0); }
            else if (ahead == 1) { WAIT_V(6); }
            else { WAIT_V(12); }
            BAR();
        };
        slow_fill(0);
        read_part<4>(c, L, 0, fa, 0);
        read_part<4>(c, L, 0, fa, 1);
        unsigned cur = 0;
        int last6 = ni - 1;
        auto step = [&](int s, Frags<4>& fc, Frags<4>& fn) {
            const unsigned nxt = next_slot(cur);
            const bool fast = ni >= s + 2 || s + 1 >= NSTEP;
            unsigned fv = pbase;
            WAIT_L0();
            if (fast) {
                if (last6 == s + 1) { WAIT_V(0); } else { WAIT_V(6); }
                BAR();
                if (R < NSTEP) {
                    if (wid == 0 && lane < 4) dma4_sys(lds0 + LDS_CTRL + (((s + 1) & 1) << 4), (unsigned)lane * 4u, rf);
                    fv = ctrl[((s & 1) << 2) + pj];
                }
            }
            {
                const int r = ready_count(fv);
                if (r > R) R = r;
            }
            const bool iss = fast && ni < R;
            SCHED();
            step_body<4, !dbg_nodma, !dbg_nocomp>(c, L, acc, fc, fn, nxt, ni, isl, iss);
            last6 = iss ? ni : -1;
            if (iss) { isl = next_slot(isl); ++ni; }
            if (!fast) {
                slow_fill(s + 1);
                last6 = ni - 1;
                read_part<4>(c, L, nxt, fn, 0);
                read_part<4>(c, L, nxt, fn, 1);
            }
            cur = nxt;
        };
        for (int s = 0; s < NSTEP; s += 2) {
            step(s, fa, fb);
            step(s + 1, fb, fa);
        }
        const bool bad = (unsigned)__builtin_amdgcn_readfirstlane(
                             __hip_atomic_load(tmo, __ATOMIC_RELAXED, __HIP_MEMORY_SCOPE_AGENT)) == call;
        store_tile<4>(acc, p.out, src * ML + g * BM, ct * BN, scale, bad);
        if (tid == 0) __hip_atomic_store(cntp, call, __ATOMIC_RELAXED, __HIP_MEMORY_SCOPE_AGENT);
    }
}

}

extern "C" void kernel_launch(int rank, int nranks, ncclComm_t comm,
                              hipStream_t stream,
                              void* const* d_in, const int* in_sizes, int n_in,
                              void* d_out, int out_size,
                              const int* in_dtypes, int out_dtype,
                              void* const* d_out_all,
                              void* const* d_ws_all, size_t ws_bytes) {
    using namespace agk;
    (void)comm; (void)in_sizes; (void)n_in; (void)out_size; (void)in_dtypes; (void)out_dtype; (void)d_out_all;
    if (nranks != NR || ws_bytes < WS_NEEDED || rank < 0 || rank >= NR) {
        fprintf(stderr, "unexpected configuration\n");
        return;
    }
    static bool attr_set[NR] = {};
    if (!attr_set[rank]) {
        HIP_CHECK(hipFuncSetAttribute((const void*)ag_gemm_fused, hipFuncAttributeMaxDynamicSharedMemorySize, LDS_BYTES));
        attr_set[rank] = true;
    }
    Params p;
    p.x = (const uint8_t*)d_in[0];
    p.w = (const uint8_t*)d_in[1];
    p.sx = (const float*)d_in[2];
    p.sw = (const float*)d_in[3];
    p.out = (float*)d_out;
    for (int i = 0; i < NR; ++i) p.ws[i] = (uint8_t*)d_ws_all[i];
    p.rank = rank;
    p.pad = 0;
    hipLaunchKernelGGL(ag_gemm_fused, dim3(NWG), dim3(512), LDS_BYTES, stream, p);
    HIP_CHECK(hipGetLastError());
}
```

```cpp
#include <hip/hip_runtime.h>
#include <rccl/rccl.h>
#include <cstdio>
#include <cstdint>

#define HIP_CHECK(expr) do { hipError_t _e = (expr); if (_e != hipSuccess) { \
    fprintf(stderr, "HIP error %s at %s:%d\n", hipGetErrorString(_e), __FILE__, __LINE__); \
    return; } } while (0)

namespace agk {

#define LAS __attribute__((address_space(3)))
typedef int i32x8 __attribute__((ext_vector_type(8)));
typedef int i32x4 __attribute__((ext_vector_type(4)));
typedef int i32x2 __attribute__((ext_vector_type(2)));
typedef unsigned u32x4 __attribute__((ext_vector_type(4)));
typedef float f32x4 __attribute__((ext_vector_type(4)));

constexpr int NR = 8;
constexpr int KT = 4096, NL = 2048, ML = 512;
constexpr int BM = 128, BN = 128, BK = 128;
constexpr int NSTEP = KT / BK;
constexpr int XB = BM * BK;
constexpr int WB = BK * BN;
constexpr int STAGE = XB + WB;
constexpr int NSLOT = 4;
constexpr int LDS_CTRL = NSLOT * STAGE;
constexpr int LDS_BYTES = LDS_CTRL + 64;
constexpr int NWG = 256;
constexpr unsigned SHARD_BYTES = ML * KT;
constexpr unsigned W_BYTES = KT * NL;

constexpr size_t WS_CNT = 0;
constexpr size_t WS_TMO = WS_CNT + NWG * 128;
constexpr size_t WS_PROG = WS_TMO + 128;
constexpr size_t WS_DBG = 512u << 10;
constexpr size_t WS_STAGE = 1u << 20;
constexpr size_t WS_NEEDED = WS_STAGE + 2ull * NR * SHARD_BYTES;
constexpr unsigned long long DEADLINE_TICKS = 20000000ull;

#define SK_START {0, 8, 1024, 1032}
#define SK_RATE {16, 16, 16, 16}
struct Sched {
    unsigned char ord[4][32];
    unsigned char pos[128];
};
constexpr Sched make_sched() {
    Sched s{};
    const int start[4] = SK_START;
    const int rate[4] = SK_RATE;
    int nxt[4] = {0, 0, 0, 0};
    for (int p = 0; p < 128; ++p) {
        int best = -1, bk = 0;
        for (int g = 0; g < 4; ++g) {
            if (nxt[g] >= 32) continue;
            const int key = start[g] + nxt[g] * rate[g];
            if (best < 0 || key < bk) { best = g; bk = key; }
        }
        s.ord[p & 3][p >> 2] = (unsigned char)(best * 32 + nxt[best]);
        s.pos[best * 32 + nxt[best]] = (unsigned char)p;
        ++nxt[best];
    }
    return s;
}

struct Params {
    const uint8_t* x;
    const uint8_t* w;
    const float* sx;
    const float* sw;
    float* out;
    uint8_t* ws[NR];
    int rank;
    int pad;
    Sched sched;
};

#define WAIT_V(n) asm volatile("s_waitcnt vmcnt(" #n ")" ::: "memory")
#define WAIT_L0() do { asm volatile("" ::: "memory"); __builtin_amdgcn_s_waitcnt(0xC07F); asm volatile("" ::: "memory"); } while (0)
#define BAR() do { asm volatile("" ::: "memory"); __builtin_amdgcn_s_barrier(); asm volatile("" ::: "memory"); } while (0)
#define SCHED() __builtin_amdgcn_sched_barrier(0)

__device__ __forceinline__ i32x4 make_rsrc(const void* p, unsigned bytes) {
    const unsigned long long a = (unsigned long long)p;
    i32x4 r;
    r.x = __builtin_amdgcn_readfirstlane((int)(unsigned)a);
    r.y = __builtin_amdgcn_readfirstlane((int)((unsigned)(a >> 32) & 0xffffu));
    r.z = __builtin_amdgcn_readfirstlane((int)bytes);
    r.w = 0x00020000;
    return r;
}
__device__ __forceinline__ void dma16(unsigned lds, unsigned voff, i32x4 rsrc, unsigned soff) {
    asm volatile("s_mov_b32 m0, %0\n\ts_nop 0\n\tbuffer_load_dwordx4 %1, %2, %3 offen lds"
                 :: "s"(lds), "v"(voff), "s"(rsrc), "s"(soff) : "memory");
}
__device__ __forceinline__ void dma16x4(unsigned lds, unsigned v0, unsigned v1, unsigned v2, unsigned v3, i32x4 rsrc, unsigned soff) {
    asm volatile("s_mov_b32 m0, %0\n\ts_nop 0\n\t"
                 "buffer_load_dwordx4 %1, %5, %6 offen lds\n\t"
                 "buffer_load_dwordx4 %2, %5, %6 offen offset:1024 lds\n\t"
                 "buffer_load_dwordx4 %3, %5, %6 offen offset:2048 lds\n\t"
                 "buffer_load_dwordx4 %4, %5, %6 offen offset:3072 lds"
                 :: "s"(lds), "v"(v0), "v"(v1), "v"(v2), "v"(v3), "s"(rsrc), "s"(soff) : "memory");
}
__device__ __forceinline__ void dma4_sys(unsigned lds, unsigned voff, i32x4 rsrc) {
    asm volatile("s_mov_b32 m0, %0\n\ts_nop 0\n\tbuffer_load_dword %1, %2, 0 offen sc0 sc1 lds"
                 :: "s"(lds), "v"(voff), "s"(rsrc) : "memory");
}
__device__ __forceinline__ void ld16_asm(u32x4& d, unsigned voff, i32x4 rsrc, unsigned soff) {
    asm volatile("buffer_load_dwordx4 %0, %1, %2, %3 offen" : "=v"(d) : "v"(voff), "s"(rsrc), "s"(soff) : "memory");
}
__device__ __forceinline__ void st16_sys(const u32x4& d, unsigned voff, i32x4 rsrc, unsigned soff) {
    asm volatile("buffer_store_dwordx4 %0, %1, %2, %3 offen sc0 sc1\n\ts_nop 1"
                 :: "v"(d), "v"(voff), "s"(rsrc), "s"(soff) : "memory");
}
__device__ __forceinline__ void wait_vm(int n) {
    if (n >= 12) {
        if (n >= 20) { WAIT_V(20); } else if (n >= 16) { WAIT_V(16); } else { WAIT_V(12); }
    } else {
        if (n >= 8) { WAIT_V(8); } else if (n >= 4) { WAIT_V(4); } else { WAIT_V(0); }
    }
}

struct Frags { i32x8 x[4]; i32x8 w[4]; };
__device__ __forceinline__ void put4(i32x8& d, int i, i32x4 v) { d[4 * i] = v.x; d[4 * i + 1] = v.y; d[4 * i + 2] = v.z; d[4 * i + 3] = v.w; }
__device__ __forceinline__ void put2(i32x8& d, int i, i32x2 v) { d[2 * i] = v.x; d[2 * i + 1] = v.y; }

__device__ __forceinline__ void read_w1(unsigned aw, Frags& f, int nb, int q) {
    put2(f.w[nb], q, __builtin_amdgcn_ds_read_tr8_b64_v2i32((LAS i32x2*)(size_t)((aw ^ (unsigned)(nb << 4)) + ((q >> 1) * 64 + (q & 1) * 8) * BN)));
}
__device__ __forceinline__ void read_x1(unsigned ax, Frags& f, int mb, int q) {
    put4(f.x[mb], q, *(LAS i32x4*)(size_t)((q ? (ax ^ 64u) : ax) + mb * 2048));
}
__device__ __forceinline__ void read_nth(unsigned ax, unsigned aw, Frags& f, int r) {
    if (r < 4) read_w1(aw, f, 0, r);
    else if (r < 12) read_x1(ax, f, (r - 4) >> 1, (r - 4) & 1);
    else read_w1(aw, f, 1 + ((r - 12) >> 2), (r - 12) & 3);
}
__device__ __forceinline__ void mma1(f32x4 (&acc)[4][4], const Frags& f, int nb, int mb) {
    acc[nb][mb] = __builtin_amdgcn_mfma_scale_f32_16x16x128_f8f6f4(f.w[nb], f.x[mb], acc[nb][mb], 1, 1, 0, 0, 0, 0);
}
__device__ __forceinline__ unsigned next_slot(unsigned soff) { return (soff + STAGE) & (NSLOT * STAGE - 1); }

__device__ __forceinline__ f32x4 epi(f32x4 a, float scale, bool bad) {
    f32x4 v = a * scale;
    v.x = v.x < 0.f ? 0.f : v.x;
    v.y = v.y < 0.f ? 0.f : v.y;
    v.z = v.z < 0.f ? 0.f : v.z;
    v.w = v.w < 0.f ? 0.f : v.w;
    const float qnan = __builtin_nanf("");
    if (bad) v = (f32x4){qnan, qnan, qnan, qnan};
    return v;
}
#define MID_ST 0
__device__ __forceinline__ void st_out(float* dst, f32x4 v) {
    asm volatile("global_store_dwordx4 %0, %1, off\n\ts_nop 1" :: "v"(dst), "v"(v) : "memory");
}
__device__ __forceinline__ float* store_tile(f32x4 (&acc)[4][4], f32x4 (&pend)[8], float* out, int row0, int col0, float scale, bool bad) {
    int tz = threadIdx.x;
    asm volatile("" : "+v"(tz));
    const int ln = tz & 63, wv = tz >> 6;
    const int wm = wv & 1, wn = wv >> 1;
    float* o = out + (size_t)(row0 + wm * 64 + (ln & 15)) * NL + (col0 + wn * 64 + 4 * (ln >> 4));
#pragma unroll
    for (int mb = 0; mb < 4; ++mb)
#pragma unroll
        for (int nb = 0; nb < 4; ++nb) {
            const f32x4 v = epi(acc[nb][mb], scale, bad);
            if (mb < 2) st_out(o + (size_t)mb * 16 * NL + nb * 16, v);
            else pend[(mb - 2) * 4 + nb] = v;
            acc[nb][mb] = (f32x4){0.f, 0.f, 0.f, 0.f};
        }
    return o + (size_t)32 * NL;
}

#define AGK_MODE 0
#define OUT_ST 2

__global__ void __launch_bounds__(512, 2) ag_gemm_fused(Params p) {
    extern __shared__ __attribute__((aligned(16))) char smem[];
    constexpr bool dbg_nowait = (AGK_MODE & 1) != 0, dbg_nopush = (AGK_MODE & 2) != 0;
    constexpr bool dbg_nodma = (AGK_MODE & 4) != 0, dbg_nolocal = (AGK_MODE & 16) != 0;
    if (AGK_MODE & 64) return;
    const int tid = threadIdx.x, lane = tid & 63;
    const int wid = __builtin_amdgcn_readfirstlane(tid >> 6);
    const int b = blockIdx.x;
    const int ct = b & 7, rt = b >> 3;
    const int rank = p.rank;
    uint8_t* wsl = p.ws[rank];
    unsigned* cntp = (unsigned*)(wsl + WS_CNT + (size_t)b * 128);

    if (rt < 4) {
        if (ct == NR - 1 || dbg_nopush) return;
        const int peer = (rank + 1 + ct) & (NR - 1);
        uint8_t* wsp = p.ws[peer];
        unsigned cnt_old;
        asm volatile("global_load_dword %0, %1, off sc1" : "=v"(cnt_old) : "v"(cntp) : "memory");
        const i32x4 rs = make_rsrc(p.x, SHARD_BYTES);
        const unsigned vs = (wid * 16 + (lane >> 3)) * KT + (lane & 7) * 16;
        const unsigned vd0 = (wid * 16 + (lane >> 3)) * BK + (((unsigned)lane & 7u) ^ ((unsigned)lane >> 4)) * 16;
        const unsigned vd1 = (wid * 16 + 8 + (lane >> 3)) * BK + (((unsigned)lane & 7u) ^ (4u + ((unsigned)lane >> 4))) * 16;
        const unsigned* ow = (const unsigned*)p.sched.ord[rt];
        u32x4 pb[4][2];
        auto push_load = [&](unsigned u, u32x4 (&d)[2]) {
            const unsigned so = (u >> 5) * (BM * KT) + (u & 31u) * BK;
            ld16_asm(d[0], vs, rs, so);
            ld16_asm(d[1], vs, rs, so + 8 * KT);
        };
        unsigned wc = ow[0];
        push_load(wc & 255u, pb[0]);
        push_load((wc >> 8) & 255u, pb[1]);
        push_load((wc >> 16) & 255u, pb[2]);
        push_load(wc >> 24, pb[3]);
        WAIT_V(8);
        asm volatile("" : "+v"(cnt_old));
        const unsigned call = (unsigned)__builtin_amdgcn_readfirstlane(cnt_old) + 1u;
        const unsigned pbase = call * 64u;
        const i32x4 rd = make_rsrc(wsp + WS_STAGE + ((size_t)(call & 1u) * NR + rank) * SHARD_BYTES, SHARD_BYTES);
        unsigned* flag = (unsigned*)(wsp + WS_PROG + (size_t)rank * 128) + rt;
        auto publish = [&](int done) {
            BAR();
            if (tid == 0) __hip_atomic_store(flag, pbase + (unsigned)done, __ATOMIC_RELAXED, __HIP_MEMORY_SCOPE_SYSTEM);
        };
        auto push_store = [&](unsigned u, u32x4 (&d)[2]) {
            st16_sys(d[0], vd0, rd, u * XB);
            st16_sys(d[1], vd1, rd, u * XB);
        };
#define PUSH_D 1
        {
            const unsigned wn = ow[1];
            WAIT_V(6); push_store(wc & 255u, pb[0]); push_load(wn & 255u, pb[0]);
            WAIT_V(8); push_store((wc >> 8) & 255u, pb[1]); push_load((wn >> 8) & 255u, pb[1]);
            WAIT_V(6); publish(1);
            WAIT_V(10); push_store((wc >> 16) & 255u, pb[2]); push_load((wn >> 16) & 255u, pb[2]);
            WAIT_V(6); publish(2);
            WAIT_V(12); push_store(wc >> 24, pb[3]); push_load(wn >> 24, pb[3]);
            WAIT_V(6); publish(3);
            wc = wn;
        }
#pragma unroll 1
        for (int t = 4; t < NSTEP - 4; t += 4) {
            const unsigned wn = ow[(t >> 2) + 1];
#pragma unroll
            for (int i = 0; i < 4; ++i) {
                push_store((wc >> (8 * i)) & 255u, pb[i]);
                push_load((wn >> (8 * i)) & 255u, pb[i]);
                WAIT_V(6); publish(t + i);
            }
            wc = wn;
        }
        push_store(wc & 255u, pb[0]);          WAIT_V(8); publish(27);
        push_store((wc >> 8) & 255u, pb[1]);   WAIT_V(6); publish(28);
        push_store((wc >> 16) & 255u, pb[2]);  WAIT_V(4); publish(29);
        push_store(wc >> 24, pb[3]);           WAIT_V(4); publish(30);
        WAIT_V(2); publish(31);
        WAIT_V(0); publish(32);
        if (tid == 0) __hip_atomic_store(cntp, call, __ATOMIC_RELAXED, __HIP_MEMORY_SCOPE_AGENT);
        return;
    }

    const int idx = rt - 4;
    const int sh = 1 + (idx >> 2), gg = (idx >> 1) & 1, half = idx & 1;
    const int src = (rank + sh) & (NR - 1);
    const int nloc = (idx < 16 && !dbg_nolocal) ? NSTEP : 0;
    const int lrow = idx * 32 & ~63;
    const int NS = nloc + 2 * NSTEP;
    const int col0 = ct * 256 + half * BN;
    const unsigned lds0 = (unsigned)(uintptr_t)(LAS char*)smem;
    volatile LAS unsigned* ctrl = (volatile LAS unsigned*)(LAS char*)(smem + LDS_CTRL);

    const i32x4 rw = make_rsrc(p.w, W_BYTES);
    unsigned* prog = (unsigned*)(wsl + WS_PROG + (size_t)src * 128);
    unsigned* tmo = (unsigned*)(wsl + WS_TMO);
    const int row_last = src * ML + (gg + 2) * BM;
    float scale = 0.f;

    if (wid < 4) {
        scale = __builtin_bit_cast(float, __builtin_amdgcn_readfirstlane(__builtin_bit_cast(int, p.sx[0] * p.sw[0])));
        if (idx >= 16) {
            const i32x4 rxp = make_rsrc(p.x, SHARD_BYTES);
            const unsigned vw = (unsigned)(tid >> 4) * NL + ct * 256 + (tid & 15) * 16;
            const unsigned vx = (unsigned)(tid >> 3) * KT + (tid & 7) * 16;
            u32x4 d0, d1, d2, d3;
            for (int ks = idx - 16; ks < NSTEP; ks += 12) {
                const unsigned so = (unsigned)ks * (BK * NL), sx = (unsigned)ks * BK;
#pragma unroll
                for (int r = 0; r < 8; r += 4) {
                    ld16_asm(d0, vw, rw, so + (r + 0) * 16 * NL); ld16_asm(d1, vw, rw, so + (r + 1) * 16 * NL);
                    ld16_asm(d2, vw, rw, so + (r + 2) * 16 * NL); ld16_asm(d3, vw, rw, so + (r + 3) * 16 * NL);
                }
#pragma unroll
                for (int r = 0; r < 16; r += 4) {
                    ld16_asm(d0, vx, rxp, sx + (r + 0) * 32 * KT); ld16_asm(d1, vx, rxp, sx + (r + 1) * 32 * KT);
                    ld16_asm(d2, vx, rxp, sx + (r + 2) * 32 * KT); ld16_asm(d3, vx, rxp, sx + (r + 3) * 32 * KT);
                }
            }
            WAIT_V(0);
            asm volatile("" :: "v"(d0), "v"(d1), "v"(d2), "v"(d3));
        }
        unsigned axl, awl;
        {
            const int wm = wid & 1, wn = wid >> 1;
            const unsigned m = lane & 15, gq = lane >> 4, q = m >> 1, h = m & 1;
            axl = lds0 + (wm * 64 + m) * BK + 16 * (gq ^ (m >> 1));
            awl = lds0 + XB + (16 * gq + q) * BN + 16 * ((unsigned)(wn * 4) ^ ((q >> 1) | ((gq & 1) << 2))) + 8 * h;
        }
        f32x4 acc[4][4];
#pragma unroll
        for (int i = 0; i < 4; ++i)
#pragma unroll
            for (int j = 0; j < 4; ++j) acc[i][j] = (f32x4){0.f, 0.f, 0.f, 0.f};
        Frags fa, fb;
        f32x4 pend[8];
#pragma unroll
        for (int i = 0; i < 8; ++i) pend[i] = (f32x4){0.f, 0.f, 0.f, 0.f};
        float* pend_o = nullptr;
        BAR();
        BAR();
        unsigned cur = 0;
        if (nloc) {
            const unsigned axs = axl - (wid & 1) * (32 * BK);
            auto lread = [&](unsigned ax, unsigned aw, Frags& f, int r) {
                if (r < 4) read_w1(aw, f, 0, r);
                else if (r < 8) read_x1(ax, f, (r - 4) >> 1, (r - 4) & 1);
                else read_w1(aw, f, 1 + ((r - 8) >> 2), (r - 8) & 3);
            };
#pragma unroll
            for (int r = 0; r < 20; ++r) lread(axs, awl, fa, r);
            auto lstep = [&](int s, Frags& fc, Frags& fn) {
                const unsigned nxt = next_slot(cur);
                const unsigned ax = axs + nxt, aw = awl + nxt;
                WAIT_L0();
                BAR();
                SCHED();
#pragma unroll
                for (int i = 0; i < 8; ++i) {
                    mma1(acc, fc, i >> 1, i & 1);
#pragma unroll
                    for (int r = 0; r < 3; ++r) if (3 * i + r < 20) lread(ax, aw, fn, 3 * i + r);
                    SCHED();
                }
                cur = nxt;
            };
#pragma unroll 1
            for (int s = 0; s < NSTEP; s += 2) {
                lstep(s, fa, fb);
                lstep(s + 1, fb, fa);
            }
            int tz = threadIdx.x;
            asm volatile("" : "+v"(tz));
            const int ln = tz & 63, wv = tz >> 6;
            float* o = p.out + (size_t)(rank * ML + lrow + (wv & 1) * 32 + (ln & 15)) * NL + (col0 + (wv >> 1) * 64 + 4 * (ln >> 4));
#pragma unroll
            for (int mb = 0; mb < 2; ++mb)
#pragma unroll
                for (int nb = 0; nb < 4; ++nb) {
                    st_out(o + (size_t)mb * 16 * NL + nb * 16, epi(acc[nb][mb], scale, false));
                    acc[nb][mb] = (f32x4){0.f, 0.f, 0.f, 0.f};
                }
            WAIT_L0();
        }
#pragma unroll
        for (int r = 0; r < 24; ++r) read_nth(axl + cur, awl + cur, fa, r);
        auto step = [&](int s, Frags& fc, Frags& fn) {
            const unsigned nxt = next_slot(cur);
            const unsigned ax = axl + nxt, aw = awl + nxt;
            WAIT_L0();
            BAR();
            SCHED();
#pragma unroll
            for (int i = 0; i < 16; ++i) {
                mma1(acc, fc, i >> 2, i & 3);
                if (2 * i < 24) { read_nth(ax, aw, fn, 2 * i); read_nth(ax, aw, fn, 2 * i + 1); }
                SCHED();
            }
            if (s == nloc + NSTEP - 1) {
                const bool bad = __builtin_amdgcn_readfirstlane((int)ctrl[4]) != 0;
                pend_o = store_tile(acc, pend, p.out, src * ML + gg * BM, col0, scale, bad);
            }
            cur = nxt;
        };
#pragma unroll 1
        for (int s = nloc; s < NS; s += 2) {
            step(s, fa, fb);
            const int k = s - nloc - NSTEP;
            if (k >= 2 && k <= 8) {
                if (k == 2) { st_out(pend_o, pend[0]); st_out(pend_o + 16, pend[1]); }
                else if (k == 4) { st_out(pend_o + 32, pend[2]); st_out(pend_o + 48, pend[3]); }
                else if (k == 6) { st_out(pend_o + 16 * NL, pend[4]); st_out(pend_o + 16 * NL + 16, pend[5]); }
                else { st_out(pend_o + 16 * NL + 32, pend[6]); st_out(pend_o + 16 * NL + 48, pend[7]); }
            }
            step(s + 1, fb, fa);
        }
        const bool bad = __builtin_amdgcn_readfirstlane((int)ctrl[4]) != 0;
        const int wm = wid & 1, wn = wid >> 1;
#pragma unroll
        for (int mb = 0; mb < 4; ++mb) {
            const unsigned row = wm * 64 + mb * 16 + (lane & 15);
#pragma unroll
            for (int nb = 0; nb < 4; ++nb) {
                const unsigned chunk = wn * 16 + nb * 4 + (lane >> 4);
                *(LAS f32x4*)(size_t)(lds0 + row * 512 + ((chunk ^ (row & 7u)) << 4)) = epi(acc[nb][mb], scale, bad);
            }
        }
    } else {
        unsigned vw[8];
        if (wid < 6) {
            const unsigned p8 = lane & 7, f0 = (unsigned)(lane >> 4) & 3u;
#pragma unroll
            for (int i = 0; i < 8; ++i) {
                const unsigned pi = (wid - 4) * 8 + i;
                vw[i] = (pi * 8 + (lane >> 3)) * NL + col0 + 16 * (p8 ^ (f0 | (((pi >> 1) & 1u) << 2))) - (i & 3) * 1024;
            }
        }
        const unsigned ldw = lds0 + XB + (wid - 4) * 8192;
        auto issue_w = [&](int j, unsigned slot) {
            if (dbg_nodma) return;
            const unsigned so = (unsigned)(j & 31) * (BK * NL);
            dma16x4(ldw + slot, vw[0], vw[1], vw[2], vw[3], rw, so);
            dma16x4(ldw + slot + 4096, vw[4], vw[5], vw[6], vw[7], rw, so);
        };
        auto wait_w = [&](int s) {
            if (s + 3 < NS) { WAIT_V(16); } else if (s + 2 < NS) { WAIT_V(8); } else { WAIT_V(0); }
        };
        if (wid == 5) {
            issue_w(0, 0); issue_w(1, STAGE); issue_w(2, 2 * STAGE); issue_w(3, 3 * STAGE);
            BAR();
            WAIT_V(24);
            BAR();
            unsigned cur = 0;
#pragma unroll 1
            for (int s = 0; s < NS; ++s) {
                wait_w(s);
                BAR();
                if (s + 4 < NS) issue_w(s + 4, cur);
                cur = next_slot(cur);
            }
        } else {
            const int xw = wid & 1;
            int nix = 0;
            unsigned isl = 0;
            const i32x4 rxl = make_rsrc(p.x + (size_t)lrow * KT, 64 * KT);
            unsigned vl[8];
            {
                const unsigned p8 = lane & 7, f0 = (unsigned)(lane >> 4);
#pragma unroll
                for (int i = 0; i < 8; ++i)
                    vl[i] = (i * 8 + (lane >> 3)) * KT + 16 * (p8 ^ ((f0 + 4u * (i & 1)) & 7u)) - (i & 3) * 1024;
            }
            auto issue_local = [&]() {
                const unsigned so = (unsigned)nix * BK;
                if (!dbg_nodma && wid == 6) {
                    dma16x4(lds0 + isl, vl[0], vl[1], vl[2], vl[3], rxl, so);
                    dma16x4(lds0 + isl + 4096, vl[4], vl[5], vl[6], vl[7], rxl, so);
                }
                isl = next_slot(isl);
                ++nix;
            };
            unsigned cnt_old;
            asm volatile("global_load_dword %0, %1, off sc1" : "=v"(cnt_old) : "v"(cntp) : "memory");
            const unsigned posv = p.sched.pos[(gg + 2 * (lane >> 5)) * 32 + (lane & 31)];
            if (wid == 4) { issue_w(0, 0); issue_w(1, STAGE); issue_w(2, 2 * STAGE); issue_w(3, 3 * STAGE); }
            else if (nloc) { issue_local(); issue_local(); issue_local(); }
            WAIT_V(0);
            asm volatile("" : "+v"(cnt_old));
            const unsigned call = (unsigned)__builtin_amdgcn_readfirstlane(cnt_old) + 1u;
            const unsigned pbase = call * 64u;
            const unsigned pj = posv & 3u;
            const unsigned need = pbase + (posv >> 2) + 1u;
            auto ready_count = [&](unsigned v) -> int {
                const unsigned long long m = __builtin_amdgcn_ballot_w64((int)(v - need) >= 0);
                return nloc + (m == ~0ull ? 64 : __builtin_ctzll(~m));
            };
            int R = dbg_nowait ? NS : nloc;
            auto refresh = [&]() {
                const int r = ready_count(ctrl[pj]);
                if (r > R) R = r;
            };
            if (wid == 4) {
                __builtin_amdgcn_fence(__ATOMIC_ACQUIRE, "");
                if (lane < 8) ctrl[lane] = lane < 4 ? pbase : 0u;
                const i32x4 rf = make_rsrc(prog, 16);
                auto watch = [&](int st) {
                    if (R < NS) refresh();
                    if (R > st || st >= NS) return;
                    unsigned long long t0 = __builtin_amdgcn_s_memrealtime();
                    for (unsigned it = 1;; ++it) {
                        const unsigned v = __hip_atomic_load(prog + pj, __ATOMIC_RELAXED, __HIP_MEMORY_SCOPE_SYSTEM);
                        const int r = ready_count(v);
                        if (r > R) { R = r; ctrl[pj] = v; }
                        if (R > st) break;
                        if ((it & 15u) == 0u) {
                            const unsigned tv = (unsigned)__builtin_amdgcn_readfirstlane(
                                __hip_atomic_load(tmo, __ATOMIC_RELAXED, __HIP_MEMORY_SCOPE_AGENT));
                            if (tv == call || __builtin_amdgcn_s_memrealtime() - t0 > DEADLINE_TICKS) {
                                if (lane == 0) __hip_atomic_store(tmo, call, __ATOMIC_RELAXED, __HIP_MEMORY_SCOPE_AGENT);
                                if (lane == 0) ctrl[4] = 1u;
                                if (lane < 4) ctrl[lane] = pbase + 63u;
                                R = NS;
                                break;
                            }
                        }
                        __builtin_amdgcn_s_sleep(1);
                    }
                };
                WAIT_V(0);
                WAIT_L0();
                BAR();
                unsigned cur = 0;
#pragma unroll 1
                for (int s = -1; s < NS; ++s) {
                    watch(s + 1);
                    if (s >= 0) wait_w(s);
                    WAIT_L0();
                    BAR();
                    if (R < NS) { if (lane < 4) dma4_sys(lds0 + LDS_CTRL, (unsigned)lane * 4u, rf); }
                    if (s >= 0) {
                        if (s + 4 < NS) issue_w(s + 4, cur);
                        cur = next_slot(cur);
                    }
                }
                if (lane == 0) __hip_atomic_store(cntp, call, __ATOMIC_RELAXED, __HIP_MEMORY_SCOPE_AGENT);
            } else {
                const i32x4 rxr = make_rsrc(wsl + WS_STAGE + ((size_t)(call & 1u) * NR + src) * SHARD_BYTES, SHARD_BYTES);
                const unsigned vr = xw * 8192 + lane * 16;
                const unsigned ldx = lds0 + xw * 8192;
                auto issue_x = [&]() {
                    if (nix < nloc) { issue_local(); return; }
                    if (!dbg_nodma) {
                        const int jr = nix - nloc;
                        const unsigned so = (unsigned)((gg + 2 * (jr >> 5)) * 32 + (jr & 31)) * XB;
                        dma16x4(ldx + isl, vr, vr, vr, vr, rxr, so);
                        dma16x4(ldx + isl + 4096, vr, vr, vr, vr, rxr, so + 4096);
                    }
                    isl = next_slot(isl);
                    ++nix;
                };
                BAR();
#pragma unroll 1
                for (int s = -1; s < NS; ++s) {
                    const int st = s + 1;
                    if (st < NS) {
                        if (R < NS) refresh();
                        while (R <= st) { __builtin_amdgcn_s_sleep(1); refresh(); }
                        const int lim = s + 4 < NS ? s + 4 : NS;
                        const int l = R < lim ? R : lim;
                        while (nix < l) issue_x();
                        const int ahead = nix - 1 - st;
                        if (ahead >= 3) { WAIT_V(24); } else if (ahead == 2) { WAIT_V(16); } else if (ahead == 1) { WAIT_V(8); } else { WAIT_V(0); }
                    }
                    WAIT_L0();
                    BAR();
                    if (nix < R && nix < NS && nix <= s + 4) issue_x();
                }
            }
        }
    }
    WAIT_L0();
    BAR();
    {
        int tz = threadIdx.x;
        asm volatile("" : "+v"(tz));
        const int ln = tz & 63, wv = tz >> 6;
        float* o = p.out + (size_t)(row_last + wv * 16 + (ln >> 5)) * NL + col0 + (ln & 31) * 4;
        f32x4 v[8];
#pragma unroll
        for (int i = 0; i < 8; ++i) {
            const unsigned row = wv * 16 + 2 * i + (ln >> 5);
            v[i] = *(LAS f32x4*)(size_t)(lds0 + row * 512 + (((unsigned)(ln & 31) ^ (row & 7u)) << 4));
        }
#pragma unroll
        for (int i = 0; i < 8; ++i) {
            float* dst = o + (size_t)i * 2 * NL;
            asm volatile("global_store_dwordx4 %0, %1, off sc1\n\ts_nop 1" :: "v"(dst), "v"(v[i]) : "memory");
        }
    }
}

}

extern "C" void kernel_launch(int rank, int nranks, ncclComm_t comm,
                              hipStream_t stream,
                              void* const* d_in, const int* in_sizes, int n_in,
                              void* d_out, int out_size,
                              const int* in_dtypes, int out_dtype,
                              void* const* d_out_all,
                              void* const* d_ws_all, size_t ws_bytes) {
    using namespace agk;
    (void)comm; (void)in_sizes; (void)n_in; (void)out_size; (void)in_dtypes; (void)out_dtype; (void)d_out_all;
    if (nranks != NR || ws_bytes < WS_NEEDED || rank < 0 || rank >= NR) {
        fprintf(stderr, "unexpected configuration\n");
        return;
    }
    static bool attr_set[NR] = {};
    if (!attr_set[rank]) {
        HIP_CHECK(hipFuncSetAttribute((const void*)ag_gemm_fused, hipFuncAttributeMaxDynamicSharedMemorySize, LDS_BYTES));
        attr_set[rank] = true;
    }
    static const Sched sched = make_sched();
    Params p;
    p.x = (const uint8_t*)d_in[0];
    p.w = (const uint8_t*)d_in[1];
    p.sx = (const float*)d_in[2];
    p.sw = (const float*)d_in[3];
    p.out = (float*)d_out;
    for (int i = 0; i < NR; ++i) p.ws[i] = (uint8_t*)d_ws_all[i];
    p.rank = rank;
    p.pad = 0;
    p.sched = sched;
    hipLaunchKernelGGL(ag_gemm_fused, dim3(NWG), dim3(512), LDS_BYTES, stream, p);
    HIP_CHECK(hipGetLastError());
}
```

```cpp
#include <hip/hip_runtime.h>
#include <rccl/rccl.h>
#include <cstdio>
#include <cstdint>

#define HIP_CHECK(expr) do { hipError_t _e = (expr); if (_e != hipSuccess) { \
    fprintf(stderr, "HIP error %s at %s:%d\n", hipGetErrorString(_e), __FILE__, __LINE__); \
    return; } } while (0)

namespace agk {

#define LAS __attribute__((address_space(3)))
typedef int i32x8 __attribute__((ext_vector_type(8)));
typedef int i32x4 __attribute__((ext_vector_type(4)));
typedef int i32x2 __attribute__((ext_vector_type(2)));
typedef unsigned u32x4 __attribute__((ext_vector_type(4)));
typedef float f32x4 __attribute__((ext_vector_type(4)));

constexpr int NR = 8;
constexpr int KT = 4096, NL = 2048, ML = 512;
constexpr int BM = 128, BN = 128, BK = 128;
constexpr int NSTEP = KT / BK;
constexpr int XB = BM * BK;
constexpr int WB = BK * BN;
constexpr int STAGE = XB + WB;
constexpr int NSLOT = 4;
constexpr int LDS_CTRL = NSLOT * STAGE;
constexpr int LDS_PARK = LDS_CTRL + 2048;
constexpr int LDS_BYTES = LDS_PARK + 16384;
constexpr int NWG = 256;
constexpr unsigned SHARD_BYTES = ML * KT;
constexpr unsigned W_BYTES = KT * NL;

constexpr size_t WS_CNT = 0;
constexpr size_t WS_TMO = WS_CNT + NWG * 128;
constexpr size_t WS_PROG = WS_TMO + 128;
constexpr size_t WS_DBG = 512u << 10;
constexpr size_t WS_STAGE = 1u << 20;
constexpr size_t WS_NEEDED = WS_STAGE + 2ull * NR * SHARD_BYTES;
constexpr unsigned long long DEADLINE_TICKS = 20000000ull;

#define SK_START {0, 8, 1024, 1032}
#define SK_RATE {16, 16, 16, 16}
struct Sched {
    unsigned char ord[4][32];
    unsigned char pos[128];
};
constexpr Sched make_sched() {
    Sched s{};
    const int start[4] = SK_START;
    const int rate[4] = SK_RATE;
    int nxt[4] = {0, 0, 0, 0};
    for (int p = 0; p < 128; ++p) {
        int best = -1, bk = 0;
        for (int g = 0; g < 4; ++g) {
            if (nxt[g] >= 32) continue;
            const int key = start[g] + nxt[g] * rate[g];
            if (best < 0 || key < bk) { best = g; bk = key; }
        }
        s.ord[p & 3][p >> 2] = (unsigned char)(best * 32 + nxt[best]);
        s.pos[best * 32 + nxt[best]] = (unsigned char)p;
        ++nxt[best];
    }
    return s;
}

struct Params {
    const uint8_t* x;
    const uint8_t* w;
    const float* sx;
    const float* sw;
    float* out;
    uint8_t* ws[NR];
    int rank;
    int pad;
    Sched sched;
};

#define WAIT_V(n) asm volatile("s_waitcnt vmcnt(" #n ")" ::: "memory")
#define WAIT_L0() do { asm volatile("" ::: "memory"); __builtin_amdgcn_s_waitcnt(0xC07F); asm volatile("" ::: "memory"); } while (0)
#define BAR() do { asm volatile("" ::: "memory"); __builtin_amdgcn_s_barrier(); asm volatile("" ::: "memory"); } while (0)
#define SCHED() __builtin_amdgcn_sched_barrier(0)

__device__ __forceinline__ i32x4 make_rsrc(const void* p, unsigned bytes) {
    const unsigned long long a = (unsigned long long)p;
    i32x4 r;
    r.x = __builtin_amdgcn_readfirstlane((int)(unsigned)a);
    r.y = __builtin_amdgcn_readfirstlane((int)((unsigned)(a >> 32) & 0xffffu));
    r.z = __builtin_amdgcn_readfirstlane((int)bytes);
    r.w = 0x00020000;
    return r;
}
__device__ __forceinline__ void dma16(unsigned lds, unsigned voff, i32x4 rsrc, unsigned soff) {
    asm volatile("s_mov_b32 m0, %0\n\ts_nop 0\n\tbuffer_load_dwordx4 %1, %2, %3 offen lds"
                 :: "s"(lds), "v"(voff), "s"(rsrc), "s"(soff) : "memory");
}
__device__ __forceinline__ void dma16x4(unsigned lds, unsigned v0, unsigned v1, unsigned v2, unsigned v3, i32x4 rsrc, unsigned soff) {
    asm volatile("s_mov_b32 m0, %0\n\ts_nop 0\n\t"
                 "buffer_load_dwordx4 %1, %5, %6 offen lds\n\t"
                 "buffer_load_dwordx4 %2, %5, %6 offen offset:1024 lds\n\t"
                 "buffer_load_dwordx4 %3, %5, %6 offen offset:2048 lds\n\t"
                 "buffer_load_dwordx4 %4, %5, %6 offen offset:3072 lds"
                 :: "s"(lds), "v"(v0), "v"(v1), "v"(v2), "v"(v3), "s"(rsrc), "s"(soff) : "memory");
}
__device__ __forceinline__ void dma4_sys(unsigned lds, unsigned voff, i32x4 rsrc) {
    asm volatile("s_mov_b32 m0, %0\n\ts_nop 0\n\tbuffer_load_dword %1, %2, 0 offen sc0 sc1 lds"
                 :: "s"(lds), "v"(voff), "s"(rsrc) : "memory");
}
__device__ __forceinline__ void ld16_asm(u32x4& d, unsigned voff, i32x4 rsrc, unsigned soff) {
    asm volatile("buffer_load_dwordx4 %0, %1, %2, %3 offen" : "=v"(d) : "v"(voff), "s"(rsrc), "s"(soff) : "memory");
}
__device__ __forceinline__ void st16_sys(const u32x4& d, unsigned voff, i32x4 rsrc, unsigned soff) {
    asm volatile("buffer_store_dwordx4 %0, %1, %2, %3 offen sc0 sc1\n\ts_nop 1"
                 :: "v"(d), "v"(voff), "s"(rsrc), "s"(soff) : "memory");
}
__device__ __forceinline__ void wait_vm(int n) {
    if (n >= 12) {
        if (n >= 20) { WAIT_V(20); } else if (n >= 16) { WAIT_V(16); } else { WAIT_V(12); }
    } else {
        if (n >= 8) { WAIT_V(8); } else if (n >= 4) { WAIT_V(4); } else { WAIT_V(0); }
    }
}

struct Frags { i32x8 x[4]; i32x8 w[4]; };
__device__ __forceinline__ void put4(i32x8& d, int i, i32x4 v) { d[4 * i] = v.x; d[4 * i + 1] = v.y; d[4 * i + 2] = v.z; d[4 * i + 3] = v.w; }
__device__ __forceinline__ void put2(i32x8& d, int i, i32x2 v) { d[2 * i] = v.x; d[2 * i + 1] = v.y; }

__device__ __forceinline__ void read_w1(unsigned aw, Frags& f, int nb, int q) {
    put2(f.w[nb], q, __builtin_amdgcn_ds_read_tr8_b64_v2i32((LAS i32x2*)(size_t)((aw ^ (unsigned)(nb << 4)) + ((q >> 1) * 64 + (q & 1) * 8) * BN)));
}
__device__ __forceinline__ void read_x1(unsigned ax, Frags& f, int mb, int q) {
    put4(f.x[mb], q, *(LAS i32x4*)(size_t)((q ? (ax ^ 64u) : ax) + mb * 2048));
}
__device__ __forceinline__ void read_nth(unsigned ax, unsigned aw, Frags& f, int r) {
    if (r < 4) read_w1(aw, f, 0, r);
    else if (r < 12) read_x1(ax, f, (r - 4) >> 1, (r - 4) & 1);
    else read_w1(aw, f, 1 + ((r - 12) >> 2), (r - 12) & 3);
}
__device__ __forceinline__ void mma1(f32x4 (&acc)[4][4], const Frags& f, int nb, int mb) {
    acc[nb][mb] = __builtin_amdgcn_mfma_scale_f32_16x16x128_f8f6f4(f.w[nb], f.x[mb], acc[nb][mb], 1, 1, 0, 0, 0, 0);
}
__device__ __forceinline__ unsigned next_slot(unsigned soff) { return (soff + STAGE) & (NSLOT * STAGE - 1); }

__device__ __forceinline__ f32x4 epi(f32x4 a, float scale, bool bad) {
    f32x4 v = a * scale;
    v.x = v.x < 0.f ? 0.f : v.x;
    v.y = v.y < 0.f ? 0.f : v.y;
    v.z = v.z < 0.f ? 0.f : v.z;
    v.w = v.w < 0.f ? 0.f : v.w;
    const float qnan = __builtin_nanf("");
    if (bad) v = (f32x4){qnan, qnan, qnan, qnan};
    return v;
}
#define MID_ST 0
__device__ __forceinline__ void st_out(float* dst, f32x4 v) {
    asm volatile("global_store_dwordx4 %0, %1, off\n\ts_nop 1" :: "v"(dst), "v"(v) : "memory");
}
__device__ __forceinline__ float* store_tile(f32x4 (&acc)[4][4], f32x4 (&pend)[8], unsigned park, float* out, int row0, int col0,
                                             float scale, bool bad) {
    int tz = threadIdx.x;
    asm volatile("" : "+v"(tz));
    const int ln = tz & 63, wv = tz >> 6;
    const int wm = wv & 1, wn = wv >> 1;
    float* o = out + (size_t)(row0 + wm * 64 + (ln & 15)) * NL + (col0 + wn * 64 + 4 * (ln >> 4));
#pragma unroll
    for (int mb = 0; mb < 4; ++mb)
#pragma unroll
        for (int nb = 0; nb < 4; ++nb) {
            const f32x4 v = epi(acc[nb][mb], scale, bad);
            if (mb == 0) st_out(o + nb * 16, v);
            else if (mb == 1) *(LAS f32x4*)(size_t)(park + nb * 1024) = v;
            else pend[(mb - 2) * 4 + nb] = v;
            acc[nb][mb] = (f32x4){0.f, 0.f, 0.f, 0.f};
        }
    return o;
}

#define AGK_MODE 0
#define OUT_ST 2

__global__ void __launch_bounds__(512, 2) ag_gemm_fused(Params p) {
    extern __shared__ __attribute__((aligned(16))) char smem[];
    constexpr bool dbg_nowait = (AGK_MODE & 1) != 0, dbg_nopush = (AGK_MODE & 2) != 0;
    constexpr bool dbg_nodma = (AGK_MODE & 4) != 0, dbg_nolocal = (AGK_MODE & 16) != 0;
    if (AGK_MODE & 64) return;
    const int tid = threadIdx.x, lane = tid & 63;
    const int wid = __builtin_amdgcn_readfirstlane(tid >> 6);
    const int b = blockIdx.x;
    const int ct = b & 7, rt = b >> 3;
    const int rank = p.rank;
    uint8_t* wsl = p.ws[rank];
    unsigned* cntp = (unsigned*)(wsl + WS_CNT + (size_t)b * 128);

    if (rt < 4) {
        if (ct == NR - 1 || dbg_nopush) return;
        const int peer = (rank + 1 + ct) & (NR - 1);
        uint8_t* wsp = p.ws[peer];
        unsigned cnt_old;
        asm volatile("global_load_dword %0, %1, off sc1" : "=v"(cnt_old) : "v"(cntp) : "memory");
        const i32x4 rs = make_rsrc(p.x, SHARD_BYTES);
        const unsigned vs = (wid * 16 + (lane >> 3)) * KT + (lane & 7) * 16;
        const unsigned vd0 = (wid * 16 + (lane >> 3)) * BK + (((unsigned)lane & 7u) ^ ((unsigned)lane >> 4)) * 16;
        const unsigned vd1 = (wid * 16 + 8 + (lane >> 3)) * BK + (((unsigned)lane & 7u) ^ (4u + ((unsigned)lane >> 4))) * 16;
        const unsigned* ow = (const unsigned*)p.sched.ord[rt];
        u32x4 pb[4][2];
        auto push_load = [&](unsigned u, u32x4 (&d)[2]) {
            const unsigned so = (u >> 5) * (BM * KT) + (u & 31u) * BK;
            ld16_asm(d[0], vs, rs, so);
            ld16_asm(d[1], vs, rs, so + 8 * KT);
        };
        unsigned wc = ow[0];
        push_load(wc & 255u, pb[0]);
        push_load((wc >> 8) & 255u, pb[1]);
        push_load((wc >> 16) & 255u, pb[2]);
        push_load(wc >> 24, pb[3]);
        WAIT_V(8);
        asm volatile("" : "+v"(cnt_old));
        const unsigned call = (unsigned)__builtin_amdgcn_readfirstlane(cnt_old) + 1u;
        const unsigned pbase = call * 64u;
        const i32x4 rd = make_rsrc(wsp + WS_STAGE + ((size_t)(call & 1u) * NR + rank) * SHARD_BYTES, SHARD_BYTES);
        unsigned* flag = (unsigned*)(wsp + WS_PROG + (size_t)rank * 128) + rt;
        auto publish = [&](int done) {
            BAR();
            if (tid == 0) __hip_atomic_store(flag, pbase + (unsigned)done, __ATOMIC_RELAXED, __HIP_MEMORY_SCOPE_SYSTEM);
        };
        auto push_store = [&](unsigned u, u32x4 (&d)[2]) {
            st16_sys(d[0], vd0, rd, u * XB);
            st16_sys(d[1], vd1, rd, u * XB);
        };
#define PUSH_D 1
        {
            const unsigned wn = ow[1];
            WAIT_V(6); push_store(wc & 255u, pb[0]); push_load(wn & 255u, pb[0]);
            WAIT_V(8); push_store((wc >> 8) & 255u, pb[1]); push_load((wn >> 8) & 255u, pb[1]);
            WAIT_V(6); publish(1);
            WAIT_V(10); push_store((wc >> 16) & 255u, pb[2]); push_load((wn >> 16) & 255u, pb[2]);
            WAIT_V(6); publish(2);
            WAIT_V(12); push_store(wc >> 24, pb[3]); push_load(wn >> 24, pb[3]);
            WAIT_V(6); publish(3);
            wc = wn;
        }
#pragma unroll 1
        for (int t = 4; t < NSTEP - 4; t += 4) {
            const unsigned wn = ow[(t >> 2) + 1];
#pragma unroll
            for (int i = 0; i < 4; ++i) {
                push_store((wc >> (8 * i)) & 255u, pb[i]);
                push_load((wn >> (8 * i)) & 255u, pb[i]);
                WAIT_V(6); publish(t + i);
            }
            wc = wn;
        }
        push_store(wc & 255u, pb[0]);          WAIT_V(8); publish(27);
        push_store((wc >> 8) & 255u, pb[1]);   WAIT_V(6); publish(28);
        push_store((wc >> 16) & 255u, pb[2]);  WAIT_V(4); publish(29);
        push_store(wc >> 24, pb[3]);           WAIT_V(4); publish(30);
        WAIT_V(2); publish(31);
        WAIT_V(0); publish(32);
        if (tid == 0) __hip_atomic_store(cntp, call, __ATOMIC_RELAXED, __HIP_MEMORY_SCOPE_AGENT);
        return;
    }

    const int idx = rt - 4;
    const int sh = 1 + (idx >> 2), gg = (idx >> 1) & 1, half = idx & 1;
    const int src = (rank + sh) & (NR - 1);
    const int nloc = (idx < 16 && !dbg_nolocal) ? NSTEP : 0;
    const int lrow = idx * 32 & ~63;
    const int NS = nloc + 2 * NSTEP;
    const int col0 = ct * 256 + half * BN;
    const unsigned lds0 = (unsigned)(uintptr_t)(LAS char*)smem;
    volatile LAS unsigned* ctrl = (volatile LAS unsigned*)(LAS char*)(smem + LDS_CTRL);

    const i32x4 rw = make_rsrc(p.w, W_BYTES);
    unsigned* prog = (unsigned*)(wsl + WS_PROG + (size_t)src * 128);
    unsigned* tmo = (unsigned*)(wsl + WS_TMO);
    const int row_last = src * ML + (gg + 2) * BM;
    float scale = 0.f;

    if (wid < 4) {
        scale = __builtin_bit_cast(float, __builtin_amdgcn_readfirstlane(__builtin_bit_cast(int, p.sx[0] * p.sw[0])));
        if (idx >= 16) {
            const i32x4 rxp = make_rsrc(p.x, SHARD_BYTES);
            const unsigned vw = (unsigned)(tid >> 4) * NL + ct * 256 + (tid & 15) * 16;
            const unsigned vx = (unsigned)(tid >> 3) * KT + (tid & 7) * 16;
            u32x4 d0, d1, d2, d3;
            for (int ks = idx - 16; ks < NSTEP; ks += 12) {
                const unsigned so = (unsigned)ks * (BK * NL), sx = (unsigned)ks * BK;
#pragma unroll
                for (int r = 0; r < 8; r += 4) {
                    ld16_asm(d0, vw, rw, so + (r + 0) * 16 * NL); ld16_asm(d1, vw, rw, so + (r + 1) * 16 * NL);
                    ld16_asm(d2, vw, rw, so + (r + 2) * 16 * NL); ld16_asm(d3, vw, rw, so + (r + 3) * 16 * NL);
                }
#pragma unroll
                for (int r = 0; r < 16; r += 4) {
                    ld16_asm(d0, vx, rxp, sx + (r + 0) * 32 * KT); ld16_asm(d1, vx, rxp, sx + (r + 1) * 32 * KT);
                    ld16_asm(d2, vx, rxp, sx + (r + 2) * 32 * KT); ld16_asm(d3, vx, rxp, sx + (r + 3) * 32 * KT);
                }
            }
            WAIT_V(0);
            asm volatile("" :: "v"(d0), "v"(d1), "v"(d2), "v"(d3));
        }
        unsigned axl, awl;
        {
            const int wm = wid & 1, wn = wid >> 1;
            const unsigned m = lane & 15, gq = lane >> 4, q = m >> 1, h = m & 1;
            axl = lds0 + (wm * 64 + m) * BK + 16 * (gq ^ (m >> 1));
            awl = lds0 + XB + (16 * gq + q) * BN + 16 * ((unsigned)(wn * 4) ^ ((q >> 1) | ((gq & 1) << 2))) + 8 * h;
        }
        f32x4 acc[4][4];
#pragma unroll
        for (int i = 0; i < 4; ++i)
#pragma unroll
            for (int j = 0; j < 4; ++j) acc[i][j] = (f32x4){0.f, 0.f, 0.f, 0.f};
        Frags fa, fb;
        f32x4 pend[8];
#pragma unroll
        for (int i = 0; i < 8; ++i) pend[i] = (f32x4){0.f, 0.f, 0.f, 0.f};
        float* pend_o = nullptr;
        const unsigned park = lds0 + LDS_PARK + wid * 4096 + lane * 16;
        BAR();
        BAR();
        unsigned cur = 0;
        if (nloc) {
            const unsigned axs = axl - (wid & 1) * (32 * BK);
            auto lread = [&](unsigned ax, unsigned aw, Frags& f, int r) {
                if (r < 4) read_w1(aw, f, 0, r);
                else if (r < 8) read_x1(ax, f, (r - 4) >> 1, (r - 4) & 1);
                else read_w1(aw, f, 1 + ((r - 8) >> 2), (r - 8) & 3);
            };
#pragma unroll
            for (int r = 0; r < 20; ++r) lread(axs, awl, fa, r);
            auto lstep = [&](int s, Frags& fc, Frags& fn) {
                const unsigned nxt = next_slot(cur);
                const unsigned ax = axs + nxt, aw = awl + nxt;
                WAIT_L0();
                BAR();
                SCHED();
#pragma unroll
                for (int i = 0; i < 8; ++i) {
                    mma1(acc, fc, i >> 1, i & 1);
#pragma unroll
                    for (int r = 0; r < 3; ++r) if (3 * i + r < 20) lread(ax, aw, fn, 3 * i + r);
                    SCHED();
                }
                cur = nxt;
            };
#pragma unroll 1
            for (int s = 0; s < NSTEP; s += 2) {
                lstep(s, fa, fb);
                lstep(s + 1, fb, fa);
            }
            int tz = threadIdx.x;
            asm volatile("" : "+v"(tz));
            const int ln = tz & 63, wv = tz >> 6;
            float* o = p.out + (size_t)(rank * ML + lrow + (wv & 1) * 32 + (ln & 15)) * NL + (col0 + (wv >> 1) * 64 + 4 * (ln >> 4));
#pragma unroll
            for (int mb = 0; mb < 2; ++mb)
#pragma unroll
                for (int nb = 0; nb < 4; ++nb) {
                    pend[mb * 4 + nb] = epi(acc[nb][mb], scale, false);
                    acc[nb][mb] = (f32x4){0.f, 0.f, 0.f, 0.f};
                }
            pend_o = o;
            WAIT_L0();
        }
#pragma unroll
        for (int r = 0; r < 24; ++r) read_nth(axl + cur, awl + cur, fa, r);
        auto step = [&](int s, Frags& fc, Frags& fn) {
            const unsigned nxt = next_slot(cur);
            const unsigned ax = axl + nxt, aw = awl + nxt;
            WAIT_L0();
            BAR();
            SCHED();
#pragma unroll
            for (int i = 0; i < 16; ++i) {
                mma1(acc, fc, i >> 2, i & 3);
                if (2 * i < 24) { read_nth(ax, aw, fn, 2 * i); read_nth(ax, aw, fn, 2 * i + 1); }
                SCHED();
            }
            if (s == nloc + NSTEP - 1) {
                const bool bad = __builtin_amdgcn_readfirstlane((int)ctrl[4]) != 0;
                pend_o = store_tile(acc, pend, park, p.out, src * ML + gg * BM, col0, scale, bad);
            }
            cur = nxt;
        };
#pragma unroll 1
        for (int s = nloc; s < NS; s += 2) {
            step(s, fa, fb);
            const int k1 = s - nloc, k2 = k1 - NSTEP;
            const int k = (nloc && k1 < NSTEP) ? k1 : k2;
            if (k >= 2 && k <= 12) {
                float* po = k2 >= 0 ? pend_o + (size_t)32 * NL : pend_o;
                if (k == 2) { st_out(po, pend[0]); st_out(po + 16, pend[1]); }
                else if (k == 4) { st_out(po + 32, pend[2]); st_out(po + 48, pend[3]); }
                else if (k == 6) { st_out(po + 16 * NL, pend[4]); st_out(po + 16 * NL + 16, pend[5]); }
                else if (k == 8) { st_out(po + 16 * NL + 32, pend[6]); st_out(po + 16 * NL + 48, pend[7]); }
                else if (k2 >= 0) {
                    const int h = k == 10 ? 0 : 2;
                    const f32x4 v0 = *(LAS f32x4*)(size_t)(park + h * 1024), v1 = *(LAS f32x4*)(size_t)(park + h * 1024 + 1024);
                    st_out(pend_o + 16 * NL + h * 16, v0);
                    st_out(pend_o + 16 * NL + h * 16 + 16, v1);
                    WAIT_L0();
                }
            }
            step(s + 1, fb, fa);
        }
        const bool bad = __builtin_amdgcn_readfirstlane((int)ctrl[4]) != 0;
        const int wm = wid & 1, wn = wid >> 1;
#pragma unroll
        for (int mb = 0; mb < 4; ++mb) {
            const unsigned row = wm * 64 + mb * 16 + (lane & 15);
#pragma unroll
            for (int nb = 0; nb < 4; ++nb) {
                const unsigned chunk = wn * 16 + nb * 4 + (lane >> 4);
                *(LAS f32x4*)(size_t)(lds0 + row * 512 + ((chunk ^ (row & 7u)) << 4)) = epi(acc[nb][mb], scale, bad);
            }
        }
    } else {
        unsigned vw[8];
        if (wid < 6) {
            const unsigned p8 = lane & 7, f0 = (unsigned)(lane >> 4) & 3u;
#pragma unroll
            for (int i = 0; i < 8; ++i) {
                const unsigned pi = (wid - 4) * 8 + i;
                vw[i] = (pi * 8 + (lane >> 3)) * NL + col0 + 16 * (p8 ^ (f0 | (((pi >> 1) & 1u) << 2))) - (i & 3) * 1024;
            }
        }
        const unsigned ldw = lds0 + XB + (wid - 4) * 8192;
        auto issue_w = [&](int j, unsigned slot) {
            if (dbg_nodma) return;
            const unsigned so = (unsigned)(j & 31) * (BK * NL);
            dma16x4(ldw + slot, vw[0], vw[1], vw[2], vw[3], rw, so);
            dma16x4(ldw + slot + 4096, vw[4], vw[5], vw[6], vw[7], rw, so);
        };
        auto wait_w = [&](int s) {
            if (s + 3 < NS) { WAIT_V(16); } else if (s + 2 < NS) { WAIT_V(8); } else { WAIT_V(0); }
        };
        if (wid == 5) {
            issue_w(0, 0); issue_w(1, STAGE); issue_w(2, 2 * STAGE); issue_w(3, 3 * STAGE);
            BAR();
            WAIT_V(24);
            BAR();
            unsigned cur = 0;
#pragma unroll 1
            for (int s = 0; s < NS; ++s) {
                wait_w(s);
                BAR();
                if (s + 4 < NS) issue_w(s + 4, cur);
                cur = next_slot(cur);
            }
        } else {
            const int xw = wid & 1;
            int nix = 0;
            unsigned isl = 0;
            const i32x4 rxl = make_rsrc(p.x + (size_t)lrow * KT, 64 * KT);
            unsigned vl[8];
            {
                const unsigned p8 = lane & 7, f0 = (unsigned)(lane >> 4);
#pragma unroll
                for (int i = 0; i < 8; ++i)
                    vl[i] = (i * 8 + (lane >> 3)) * KT + 16 * (p8 ^ ((f0 + 4u * (i & 1)) & 7u)) - (i & 3) * 1024;
            }
            auto issue_local = [&]() {
                const unsigned so = (unsigned)nix * BK;
                if (!dbg_nodma && wid == 6) {
                    dma16x4(lds0 + isl, vl[0], vl[1], vl[2], vl[3], rxl, so);
                    dma16x4(lds0 + isl + 4096, vl[4], vl[5], vl[6], vl[7], rxl, so);
                }
                isl = next_slot(isl);
                ++nix;
            };
            unsigned cnt_old;
            asm volatile("global_load_dword %0, %1, off sc1" : "=v"(cnt_old) : "v"(cntp) : "memory");
            const unsigned posv = p.sched.pos[(gg + 2 * (lane >> 5)) * 32 + (lane & 31)];
            if (wid == 4) { issue_w(0, 0); issue_w(1, STAGE); issue_w(2, 2 * STAGE); issue_w(3, 3 * STAGE); }
            else if (nloc) { issue_local(); issue_local(); issue_local(); }
            WAIT_V(0);
            asm volatile("" : "+v"(cnt_old));
            const unsigned call = (unsigned)__builtin_amdgcn_readfirstlane(cnt_old) + 1u;
            const unsigned pbase = call * 64u;
            const unsigned pj = posv & 3u;
            const unsigned need = pbase + (posv >> 2) + 1u;
            auto ready_count = [&](unsigned v) -> int {
                const unsigned long long m = __builtin_amdgcn_ballot_w64((int)(v - need) >= 0);
                return nloc + (m == ~0ull ? 64 : __builtin_ctzll(~m));
            };
            int R = dbg_nowait ? NS : nloc;
            auto refresh = [&]() {
                const int r = ready_count(ctrl[pj]);
                if (r > R) R = r;
            };
            if (wid == 4) {
                __builtin_amdgcn_fence(__ATOMIC_ACQUIRE, "");
                if (lane < 8) ctrl[lane] = lane < 4 ? pbase : 0u;
                const i32x4 rf = make_rsrc(prog, 16);
                auto watch = [&](int st) {
                    if (R < NS) refresh();
                    if (R > st || st >= NS) return;
                    unsigned long long t0 = __builtin_amdgcn_s_memrealtime();
                    for (unsigned it = 1;; ++it) {
                        const unsigned v = __hip_atomic_load(prog + pj, __ATOMIC_RELAXED, __HIP_MEMORY_SCOPE_SYSTEM);
                        const int r = ready_count(v);
                        if (r > R) { R = r; ctrl[pj] = v; }
                        if (R > st) break;
                        if ((it & 15u) == 0u) {
                            const unsigned tv = (unsigned)__builtin_amdgcn_readfirstlane(
                                __hip_atomic_load(tmo, __ATOMIC_RELAXED, __HIP_MEMORY_SCOPE_AGENT));
                            if (tv == call || __builtin_amdgcn_s_memrealtime() - t0 > DEADLINE_TICKS) {
                                if (lane == 0) __hip_atomic_store(tmo, call, __ATOMIC_RELAXED, __HIP_MEMORY_SCOPE_AGENT);
                                if (lane == 0) ctrl[4] = 1u;
                                if (lane < 4) ctrl[lane] = pbase + 63u;
                                R = NS;
                                break;
                            }
                        }
                        __builtin_amdgcn_s_sleep(1);
                    }
                };
                WAIT_V(0);
                WAIT_L0();
                BAR();
                unsigned cur = 0;
#pragma unroll 1
                for (int s = -1; s < NS; ++s) {
                    watch(s + 1);
                    if (s >= 0) wait_w(s);
                    WAIT_L0();
                    BAR();
                    if (R < NS) { if (lane < 4) dma4_sys(lds0 + LDS_CTRL, (unsigned)lane * 4u, rf); }
                    if (s >= 0) {
                        if (s + 4 < NS) issue_w(s + 4, cur);
                        cur = next_slot(cur);
                    }
                }
                if (lane == 0) __hip_atomic_store(cntp, call, __ATOMIC_RELAXED, __HIP_MEMORY_SCOPE_AGENT);
            } else {
                const i32x4 rxr = make_rsrc(wsl + WS_STAGE + ((size_t)(call & 1u) * NR + src) * SHARD_BYTES, SHARD_BYTES);
                const unsigned vr = xw * 8192 + lane * 16;
                const unsigned ldx = lds0 + xw * 8192;
                auto issue_x = [&]() {
                    if (nix < nloc) { issue_local(); return; }
                    if (!dbg_nodma) {
                        const int jr = nix - nloc;
                        const unsigned so = (unsigned)((gg + 2 * (jr >> 5)) * 32 + (jr & 31)) * XB;
                        dma16x4(ldx + isl, vr, vr, vr, vr, rxr, so);
                        dma16x4(ldx + isl + 4096, vr, vr, vr, vr, rxr, so + 4096);
                    }
                    isl = next_slot(isl);
                    ++nix;
                };
                BAR();
#pragma unroll 1
                for (int s = -1; s < NS; ++s) {
                    const int st = s + 1;
                    if (st < NS) {
                        if (R < NS) refresh();
                        while (R <= st) { __builtin_amdgcn_s_sleep(1); refresh(); }
                        const int lim = s + 4 < NS ? s + 4 : NS;
                        const int l = R < lim ? R : lim;
                        while (nix < l) issue_x();
                        const int ahead = nix - 1 - st;
                        if (ahead >= 3) { WAIT_V(24); } else if (ahead == 2) { WAIT_V(16); } else if (ahead == 1) { WAIT_V(8); } else { WAIT_V(0); }
                    }
                    WAIT_L0();
                    BAR();
                    if (nix < R && nix < NS && nix <= s + 4) issue_x();
                }
            }
        }
    }
    WAIT_L0();
    BAR();
    {
        int tz = threadIdx.x;
        asm volatile("" : "+v"(tz));
        const int ln = tz & 63, wv = tz >> 6;
        float* o = p.out + (size_t)(row_last + wv * 16 + (ln >> 5)) * NL + col0 + (ln & 31) * 4;
        f32x4 v[8];
#pragma unroll
        for (int i = 0; i < 8; ++i) {
            const unsigned row = wv * 16 + 2 * i + (ln >> 5);
            v[i] = *(LAS f32x4*)(size_t)(lds0 + row * 512 + (((unsigned)(ln & 31) ^ (row & 7u)) << 4));
        }
#pragma unroll
        for (int i = 0; i < 8; ++i) {
            float* dst = o + (size_t)i * 2 * NL;
            asm volatile("global_store_dwordx4 %0, %1, off sc1\n\ts_nop 1" :: "v"(dst), "v"(v[i]) : "memory");
        }
    }
}

}

extern "C" void kernel_launch(int rank, int nranks, ncclComm_t comm,
                              hipStream_t stream,
                              void* const* d_in, const int* in_sizes, int n_in,
                              void* d_out, int out_size,
                              const int* in_dtypes, int out_dtype,
                              void* const* d_out_all,
                              void* const* d_ws_all, size_t ws_bytes) {
    using namespace agk;
    (void)comm; (void)in_sizes; (void)n_in; (void)out_size; (void)in_dtypes; (void)out_dtype; (void)d_out_all;
    if (nranks != NR || ws_bytes < WS_NEEDED || rank < 0 || rank >= NR) {
        fprintf(stderr, "unexpected configuration\n");
        return;
    }
    static bool attr_set[NR] = {};
    if (!attr_set[rank]) {
        HIP_CHECK(hipFuncSetAttribute((const void*)ag_gemm_fused, hipFuncAttributeMaxDynamicSharedMemorySize, LDS_BYTES));
        attr_set[rank] = true;
    }
    static const Sched sched = make_sched();
    Params p;
    p.x = (const uint8_t*)d_in[0];
    p.w = (const uint8_t*)d_in[1];
    p.sx = (const float*)d_in[2];
    p.sw = (const float*)d_in[3];
    p.out = (float*)d_out;
    for (int i = 0; i < NR; ++i) p.ws[i] = (uint8_t*)d_ws_all[i];
    p.rank = rank;
    p.pad = 0;
    p.sched = sched;
    hipLaunchKernelGGL(ag_gemm_fused, dim3(NWG), dim3(512), LDS_BYTES, stream, p);
    HIP_CHECK(hipGetLastError());
}
```

```cpp
#include <hip/hip_runtime.h>
#include <rccl/rccl.h>
#include <cstdio>
#include <cstdint>

#define HIP_CHECK(expr) do { hipError_t _e = (expr); if (_e != hipSuccess) { \
    fprintf(stderr, "HIP error %s at %s:%d\n", hipGetErrorString(_e), __FILE__, __LINE__); \
    return; } } while (0)

namespace agk {

#define LAS __attribute__((address_space(3)))
typedef int i32x8 __attribute__((ext_vector_type(8)));
typedef int i32x4 __attribute__((ext_vector_type(4)));
typedef int i32x2 __attribute__((ext_vector_type(2)));
typedef unsigned u32x4 __attribute__((ext_vector_type(4)));
typedef float f32x4 __attribute__((ext_vector_type(4)));

constexpr int NR = 8;
constexpr int KT = 4096, NL = 2048, ML = 512;
constexpr int BM = 128, BN = 128, BK = 128;
constexpr int NSTEP = KT / BK;
constexpr int XB = BM * BK;
constexpr int WB = BK * BN;
#define NXS 5
#define NWS 4
constexpr int LDS_W = NXS * XB;
constexpr int LDS_CTRL = LDS_W + NWS * WB;
constexpr int LDS_PARK = LDS_CTRL + 2048;
constexpr int LDS_BYTES = LDS_PARK + 12288;
static_assert(LDS_BYTES <= 163840, "LDS");
constexpr int NWG = 256;
constexpr unsigned SHARD_BYTES = ML * KT;
constexpr unsigned W_BYTES = KT * NL;

constexpr size_t WS_CNT = 0;
constexpr size_t WS_TMO = WS_CNT + NWG * 128;
constexpr size_t WS_PROG = WS_TMO + 128;
constexpr size_t WS_DBG = 512u << 10;
constexpr size_t WS_STAGE = 1u << 20;
constexpr size_t WS_NEEDED = WS_STAGE + 2ull * NR * SHARD_BYTES;
constexpr unsigned long long DEADLINE_TICKS = 20000000ull;

#define SK_START {0, 8, 1024, 1032}
#define SK_RATE {16, 16, 16, 16}
struct Sched {
    unsigned char ord[4][32];
    unsigned char pos[128];
};
constexpr Sched make_sched() {
    Sched s{};
    const int start[4] = SK_START;
    const int rate[4] = SK_RATE;
    int nxt[4] = {0, 0, 0, 0};
    for (int p = 0; p < 128; ++p) {
        int best = -1, bk = 0;
        for (int g = 0; g < 4; ++g) {
            if (nxt[g] >= 32) continue;
            const int key = start[g] + nxt[g] * rate[g];
            if (best < 0 || key < bk) { best = g; bk = key; }
        }
        s.ord[p & 3][p >> 2] = (unsigned char)(best * 32 + nxt[best]);
        s.pos[best * 32 + nxt[best]] = (unsigned char)p;
        ++nxt[best];
    }
    return s;
}

struct Params {
    const uint8_t* x;
    const uint8_t* w;
    const float* sx;
    const float* sw;
    float* out;
    uint8_t* ws[NR];
    int rank;
    int pad;
    Sched sched;
};

#define WAIT_V(n) asm volatile("s_waitcnt vmcnt(" #n ")" ::: "memory")
#define WAIT_L0() do { asm volatile("" ::: "memory"); __builtin_amdgcn_s_waitcnt(0xC07F); asm volatile("" ::: "memory"); } while (0)
#define BAR() do { asm volatile("" ::: "memory"); __builtin_amdgcn_s_barrier(); asm volatile("" ::: "memory"); } while (0)
#define SCHED() __builtin_amdgcn_sched_barrier(0)

__device__ __forceinline__ i32x4 make_rsrc(const void* p, unsigned bytes) {
    const unsigned long long a = (unsigned long long)p;
    i32x4 r;
    r.x = __builtin_amdgcn_readfirstlane((int)(unsigned)a);
    r.y = __builtin_amdgcn_readfirstlane((int)((unsigned)(a >> 32) & 0xffffu));
    r.z = __builtin_amdgcn_readfirstlane((int)bytes);
    r.w = 0x00020000;
    return r;
}
__device__ __forceinline__ void dma16(unsigned lds, unsigned voff, i32x4 rsrc, unsigned soff) {
    asm volatile("s_mov_b32 m0, %0\n\ts_nop 0\n\tbuffer_load_dwordx4 %1, %2, %3 offen lds"
                 :: "s"(lds), "v"(voff), "s"(rsrc), "s"(soff) : "memory");
}
__device__ __forceinline__ void dma16x4(unsigned lds, unsigned v0, unsigned v1, unsigned v2, unsigned v3, i32x4 rsrc, unsigned soff) {
    asm volatile("s_mov_b32 m0, %0\n\ts_nop 0\n\t"
                 "buffer_load_dwordx4 %1, %5, %6 offen lds\n\t"
                 "buffer_load_dwordx4 %2, %5, %6 offen offset:1024 lds\n\t"
                 "buffer_load_dwordx4 %3, %5, %6 offen offset:2048 lds\n\t"
                 "buffer_load_dwordx4 %4, %5, %6 offen offset:3072 lds"
                 :: "s"(lds), "v"(v0), "v"(v1), "v"(v2), "v"(v3), "s"(rsrc), "s"(soff) : "memory");
}
__device__ __forceinline__ void dma4_sys(unsigned lds, unsigned voff, i32x4 rsrc) {
    asm volatile("s_mov_b32 m0, %0\n\ts_nop 0\n\tbuffer_load_dword %1, %2, 0 offen sc0 sc1 lds"
                 :: "s"(lds), "v"(voff), "s"(rsrc) : "memory");
}
__device__ __forceinline__ void ld16_asm(u32x4& d, unsigned voff, i32x4 rsrc, unsigned soff) {
    asm volatile("buffer_load_dwordx4 %0, %1, %2, %3 offen" : "=v"(d) : "v"(voff), "s"(rsrc), "s"(soff) : "memory");
}
__device__ __forceinline__ void st16_sys(const u32x4& d, unsigned voff, i32x4 rsrc, unsigned soff) {
    asm volatile("buffer_store_dwordx4 %0, %1, %2, %3 offen sc0 sc1\n\ts_nop 1"
                 :: "v"(d), "v"(voff), "s"(rsrc), "s"(soff) : "memory");
}
__device__ __forceinline__ void wait_vm(int n) {
    if (n >= 12) {
        if (n >= 20) { WAIT_V(20); } else if (n >= 16) { WAIT_V(16); } else { WAIT_V(12); }
    } else {
        if (n >= 8) { WAIT_V(8); } else if (n >= 4) { WAIT_V(4); } else { WAIT_V(0); }
    }
}

struct Frags { i32x8 x[4]; i32x8 w[4]; };
__device__ __forceinline__ void put4(i32x8& d, int i, i32x4 v) { d[4 * i] = v.x; d[4 * i + 1] = v.y; d[4 * i + 2] = v.z; d[4 * i + 3] = v.w; }
__device__ __forceinline__ void put2(i32x8& d, int i, i32x2 v) { d[2 * i] = v.x; d[2 * i + 1] = v.y; }

__device__ __forceinline__ void read_w1(unsigned aw, Frags& f, int nb, int q) {
    put2(f.w[nb], q, __builtin_amdgcn_ds_read_tr8_b64_v2i32((LAS i32x2*)(size_t)((aw ^ (unsigned)(nb << 4)) + ((q >> 1) * 64 + (q & 1) * 8) * BN)));
}
__device__ __forceinline__ void read_x1(unsigned ax, Frags& f, int mb, int q) {
    put4(f.x[mb], q, *(LAS i32x4*)(size_t)((q ? (ax ^ 64u) : ax) + mb * 2048));
}
__device__ __forceinline__ void read_nth(unsigned ax, unsigned aw, Frags& f, int r) {
    if (r < 4) read_w1(aw, f, 0, r);
    else if (r < 12) read_x1(ax, f, (r - 4) >> 1, (r - 4) & 1);
    else read_w1(aw, f, 1 + ((r - 12) >> 2), (r - 12) & 3);
}
__device__ __forceinline__ void mma1(f32x4 (&acc)[4][4], const Frags& f, int nb, int mb) {
    acc[nb][mb] = __builtin_amdgcn_mfma_scale_f32_16x16x128_f8f6f4(f.w[nb], f.x[mb], acc[nb][mb], 1, 1, 0, 0, 0, 0);
}
__device__ __forceinline__ unsigned next_x(unsigned soff) { return soff == (NXS - 1) * XB ? 0u : soff + XB; }
__device__ __forceinline__ unsigned next_w(unsigned soff) { return soff == (NWS - 1) * WB ? 0u : soff + WB; }

__device__ __forceinline__ f32x4 epi(f32x4 a, float scale, bool bad) {
    f32x4 v = a * scale;
    v.x = v.x < 0.f ? 0.f : v.x;
    v.y = v.y < 0.f ? 0.f : v.y;
    v.z = v.z < 0.f ? 0.f : v.z;
    v.w = v.w < 0.f ? 0.f : v.w;
    const float qnan = __builtin_nanf("");
    if (bad) v = (f32x4){qnan, qnan, qnan, qnan};
    return v;
}
#define MID_ST 0
__device__ __forceinline__ void st_out(float* dst, f32x4 v) {
    asm volatile("global_store_dwordx4 %0, %1, off\n\ts_nop 1" :: "v"(dst), "v"(v) : "memory");
}
__device__ __forceinline__ float* store_tile(f32x4 (&acc)[4][4], f32x4 (&pend)[8], unsigned park, float* out, int row0, int col0,
                                             float scale, bool bad) {
    int tz = threadIdx.x;
    asm volatile("" : "+v"(tz));
    const int ln = tz & 63, wv = tz >> 6;
    const int wm = wv & 1, wn = wv >> 1;
    float* o = out + (size_t)(row0 + wm * 64 + (ln & 15)) * NL + (col0 + wn * 64 + 4 * (ln >> 4));
#pragma unroll
    for (int mb = 0; mb < 4; ++mb)
#pragma unroll
        for (int nb = 0; nb < 4; ++nb) {
            const f32x4 v = epi(acc[nb][mb], scale, bad);
            if (mb == 0 || (mb == 1 && nb == 3)) st_out(o + (size_t)mb * 16 * NL + nb * 16, v);
            else if (mb == 1) *(LAS f32x4*)(size_t)(park + nb * 1024) = v;
            else pend[(mb - 2) * 4 + nb] = v;
            acc[nb][mb] = (f32x4){0.f, 0.f, 0.f, 0.f};
        }
    return o;
}

#define PRE_BAR 2
#define AGK_MODE 0
#define OUT_ST 2

__global__ void __launch_bounds__(512, 2) ag_gemm_fused(Params p) {
    extern __shared__ __attribute__((aligned(16))) char smem[];
    constexpr bool dbg_nowait = (AGK_MODE & 1) != 0, dbg_nopush = (AGK_MODE & 2) != 0;
    constexpr bool dbg_nodma = (AGK_MODE & 4) != 0, dbg_nolocal = (AGK_MODE & 16) != 0;
    if (AGK_MODE & 64) return;
    const int tid = threadIdx.x, lane = tid & 63;
    const int wid = __builtin_amdgcn_readfirstlane(tid >> 6);
    const int b = blockIdx.x;
    const int ct = b & 7, rt = b >> 3;
    const int rank = p.rank;
    uint8_t* wsl = p.ws[rank];
    unsigned* cntp = (unsigned*)(wsl + WS_CNT + (size_t)b * 128);

    if (rt < 4) {
        if (ct == NR - 1 || dbg_nopush) return;
        const int peer = (rank + 1 + ct) & (NR - 1);
        uint8_t* wsp = p.ws[peer];
        unsigned cnt_old;
        asm volatile("global_load_dword %0, %1, off sc1" : "=v"(cnt_old) : "v"(cntp) : "memory");
        const i32x4 rs = make_rsrc(p.x, SHARD_BYTES);
        const unsigned vs = (wid * 16 + (lane >> 3)) * KT + (lane & 7) * 16;
        const unsigned vd0 = (wid * 16 + (lane >> 3)) * BK + (((unsigned)lane & 7u) ^ ((unsigned)lane >> 4)) * 16;
        const unsigned vd1 = (wid * 16 + 8 + (lane >> 3)) * BK + (((unsigned)lane & 7u) ^ (4u + ((unsigned)lane >> 4))) * 16;
        const unsigned* ow = (const unsigned*)p.sched.ord[rt];
        u32x4 pb[4][2];
        auto push_load = [&](unsigned u, u32x4 (&d)[2]) {
            const unsigned so = (u >> 5) * (BM * KT) + (u & 31u) * BK;
            ld16_asm(d[0], vs, rs, so);
            ld16_asm(d[1], vs, rs, so + 8 * KT);
        };
        unsigned wc = ow[0];
        push_load(wc & 255u, pb[0]);
        push_load((wc >> 8) & 255u, pb[1]);
        push_load((wc >> 16) & 255u, pb[2]);
        push_load(wc >> 24, pb[3]);
        WAIT_V(8);
        asm volatile("" : "+v"(cnt_old));
        const unsigned call = (unsigned)__builtin_amdgcn_readfirstlane(cnt_old) + 1u;
        const unsigned pbase = call * 64u;
        const i32x4 rd = make_rsrc(wsp + WS_STAGE + ((size_t)(call & 1u) * NR + rank) * SHARD_BYTES, SHARD_BYTES);
        unsigned* flag = (unsigned*)(wsp + WS_PROG + (size_t)rank * 128) + rt;
        auto publish = [&](int done) {
            BAR();
            if (tid == 0) __hip_atomic_store(flag, pbase + (unsigned)done, __ATOMIC_RELAXED, __HIP_MEMORY_SCOPE_SYSTEM);
        };
        auto push_store = [&](unsigned u, u32x4 (&d)[2]) {
            st16_sys(d[0], vd0, rd, u * XB);
            st16_sys(d[1], vd1, rd, u * XB);
        };
#define PUSH_D 1
        {
            const unsigned wn = ow[1];
            WAIT_V(6); push_store(wc & 255u, pb[0]); push_load(wn & 255u, pb[0]);
            WAIT_V(8); push_store((wc >> 8) & 255u, pb[1]); push_load((wn >> 8) & 255u, pb[1]);
            WAIT_V(6); publish(1);
            WAIT_V(10); push_store((wc >> 16) & 255u, pb[2]); push_load((wn >> 16) & 255u, pb[2]);
            WAIT_V(6); publish(2);
            WAIT_V(12); push_store(wc >> 24, pb[3]); push_load(wn >> 24, pb[3]);
            WAIT_V(6); publish(3);
            wc = wn;
        }
#pragma unroll 1
        for (int t = 4; t < NSTEP - 4; t += 4) {
            const unsigned wn = ow[(t >> 2) + 1];
#pragma unroll
            for (int i = 0; i < 4; ++i) {
                push_store((wc >> (8 * i)) & 255u, pb[i]);
                push_load((wn >> (8 * i)) & 255u, pb[i]);
                WAIT_V(6); publish(t + i);
            }
            wc = wn;
        }
        push_store(wc & 255u, pb[0]);          WAIT_V(8); publish(27);
        push_store((wc >> 8) & 255u, pb[1]);   WAIT_V(6); publish(28);
        push_store((wc >> 16) & 255u, pb[2]);  WAIT_V(4); publish(29);
        push_store(wc >> 24, pb[3]);           WAIT_V(4); publish(30);
        WAIT_V(2); publish(31);
        WAIT_V(0); publish(32);
        if (tid == 0) __hip_atomic_store(cntp, call, __ATOMIC_RELAXED, __HIP_MEMORY_SCOPE_AGENT);
        return;
    }

    const int idx = rt - 4;
    const int sh = 1 + (idx >> 2), gg = (idx >> 1) & 1, half = idx & 1;
    const int src = (rank + sh) & (NR - 1);
    const int nloc = (idx < 16 && !dbg_nolocal) ? NSTEP : 0;
    const int lrow = idx * 32 & ~63;
    const int NS = nloc + 2 * NSTEP;
    const int col0 = ct * 256 + half * BN;
    const unsigned lds0 = (unsigned)(uintptr_t)(LAS char*)smem;
    volatile LAS unsigned* ctrl = (volatile LAS unsigned*)(LAS char*)(smem + LDS_CTRL);

    const i32x4 rw = make_rsrc(p.w, W_BYTES);
    unsigned* prog = (unsigned*)(wsl + WS_PROG + (size_t)src * 128);
    unsigned* tmo = (unsigned*)(wsl + WS_TMO);
    const int row_last = src * ML + (gg + 2) * BM;
    float scale = 0.f;

    if (wid < 4) {
        scale = __builtin_bit_cast(float, __builtin_amdgcn_readfirstlane(__builtin_bit_cast(int, p.sx[0] * p.sw[0])));
        if (idx >= 16) {
            const i32x4 rxp = make_rsrc(p.x, SHARD_BYTES);
            const unsigned vw = (unsigned)(tid >> 4) * NL + ct * 256 + (tid & 15) * 16;
            const unsigned vx = (unsigned)(tid >> 3) * KT + (tid & 7) * 16;
            u32x4 d0, d1, d2, d3;
            for (int ks = idx - 16; ks < NSTEP; ks += 12) {
                const unsigned so = (unsigned)ks * (BK * NL), sx = (unsigned)ks * BK;
#pragma unroll
                for (int r = 0; r < 8; r += 4) {
                    ld16_asm(d0, vw, rw, so + (r + 0) * 16 * NL); ld16_asm(d1, vw, rw, so + (r + 1) * 16 * NL);
                    ld16_asm(d2, vw, rw, so + (r + 2) * 16 * NL); ld16_asm(d3, vw, rw, so + (r + 3) * 16 * NL);
                }
#pragma unroll
                for (int r = 0; r < 16; r += 4) {
                    ld16_asm(d0, vx, rxp, sx + (r + 0) * 32 * KT); ld16_asm(d1, vx, rxp, sx + (r + 1) * 32 * KT);
                    ld16_asm(d2, vx, rxp, sx + (r + 2) * 32 * KT); ld16_asm(d3, vx, rxp, sx + (r + 3) * 32 * KT);
                }
            }
            WAIT_V(0);
            asm volatile("" :: "v"(d0), "v"(d1), "v"(d2), "v"(d3));
        }
        unsigned axl, awl;
        {
            const int wm = wid & 1, wn = wid >> 1;
            const unsigned m = lane & 15, gq = lane >> 4, q = m >> 1, h = m & 1;
            axl = lds0 + (wm * 64 + m) * BK + 16 * (gq ^ (m >> 1));
            awl = lds0 + LDS_W + (16 * gq + q) * BN + 16 * ((unsigned)(wn * 4) ^ ((q >> 1) | ((gq & 1) << 2))) + 8 * h;
        }
        f32x4 acc[4][4];
#pragma unroll
        for (int i = 0; i < 4; ++i)
#pragma unroll
            for (int j = 0; j < 4; ++j) acc[i][j] = (f32x4){0.f, 0.f, 0.f, 0.f};
        Frags fa, fb;
        f32x4 pend[8];
#pragma unroll
        for (int i = 0; i < 8; ++i) pend[i] = (f32x4){0.f, 0.f, 0.f, 0.f};
        float* pend_o = nullptr;
        const unsigned park = lds0 + LDS_PARK + wid * 3072 + lane * 16;
        BAR();
        BAR();
        unsigned cx = 0, cw = 0;
        if (nloc) {
            const unsigned axs = axl - (wid & 1) * (32 * BK);
            auto lread = [&](unsigned ax, unsigned aw, Frags& f, int r) {
                if (r < 4) read_w1(aw, f, 0, r);
                else if (r < 8) read_x1(ax, f, (r - 4) >> 1, (r - 4) & 1);
                else read_w1(aw, f, 1 + ((r - 8) >> 2), (r - 8) & 3);
            };
#pragma unroll
            for (int r = 0; r < 20; ++r) lread(axs, awl, fa, r);
            auto lstep = [&](int s, Frags& fc, Frags& fn) {
                cx = next_x(cx); cw = next_w(cw);
                const unsigned ax = axs + cx, aw = awl + cw;
                WAIT_L0();
                BAR();
                SCHED();
#pragma unroll
                for (int i = 0; i < 8; ++i) {
                    mma1(acc, fc, i >> 1, i & 1);
#pragma unroll
                    for (int r = 0; r < 3; ++r) if (3 * i + r < 20) lread(ax, aw, fn, 3 * i + r);
                    SCHED();
                }
            };
#pragma unroll 1
            for (int s = 0; s < NSTEP; s += 2) {
                lstep(s, fa, fb);
                lstep(s + 1, fb, fa);
            }
            int tz = threadIdx.x;
            asm volatile("" : "+v"(tz));
            const int ln = tz & 63, wv = tz >> 6;
            float* o = p.out + (size_t)(rank * ML + lrow + (wv & 1) * 32 + (ln & 15)) * NL + (col0 + (wv >> 1) * 64 + 4 * (ln >> 4));
#pragma unroll
            for (int mb = 0; mb < 2; ++mb)
#pragma unroll
                for (int nb = 0; nb < 4; ++nb) {
                    pend[mb * 4 + nb] = epi(acc[nb][mb], scale, false);
                    acc[nb][mb] = (f32x4){0.f, 0.f, 0.f, 0.f};
                }
            pend_o = o;
            WAIT_L0();
        }
#pragma unroll
        for (int r = 0; r < 24; ++r) read_nth(axl + cx, awl + cw, fa, r);
        auto step = [&](int s, Frags& fc, Frags& fn) {
            cx = next_x(cx); cw = next_w(cw);
            const unsigned ax = axl + cx, aw = awl + cw;
            WAIT_L0();
            SCHED();
#pragma unroll
            for (int i = 0; i < PRE_BAR; ++i) { mma1(acc, fc, i >> 2, i & 3); SCHED(); }
            BAR();
            SCHED();
#pragma unroll
            for (int i = PRE_BAR; i < 16; ++i) {
                mma1(acc, fc, i >> 2, i & 3);
                if (2 * (i - PRE_BAR) < 24) { read_nth(ax, aw, fn, 2 * (i - PRE_BAR)); read_nth(ax, aw, fn, 2 * (i - PRE_BAR) + 1); }
                SCHED();
            }
            if (s == nloc + NSTEP - 1) {
                const bool bad = __builtin_amdgcn_readfirstlane((int)ctrl[4]) != 0;
                pend_o = store_tile(acc, pend, park, p.out, src * ML + gg * BM, col0, scale, bad);
            }
        };
#pragma unroll 1
        for (int s = nloc; s < NS; s += 2) {
            step(s, fa, fb);
            const int k1 = s - nloc, k2 = k1 - NSTEP;
            const int k = (nloc && k1 < NSTEP) ? k1 : k2;
            if (k >= 2 && k <= 12) {
                float* po = k2 >= 0 ? pend_o + (size_t)32 * NL : pend_o;
                if (k == 2) { st_out(po, pend[0]); st_out(po + 16, pend[1]); }
                else if (k == 4) { st_out(po + 32, pend[2]); st_out(po + 48, pend[3]); }
                else if (k == 6) { st_out(po + 16 * NL, pend[4]); st_out(po + 16 * NL + 16, pend[5]); }
                else if (k == 8) { st_out(po + 16 * NL + 32, pend[6]); st_out(po + 16 * NL + 48, pend[7]); }
                else if (k2 >= 0) {
                    if (k == 10) {
                        const f32x4 v0 = *(LAS f32x4*)(size_t)(park), v1 = *(LAS f32x4*)(size_t)(park + 1024);
                        st_out(pend_o + 16 * NL, v0);
                        st_out(pend_o + 16 * NL + 16, v1);
                    } else {
                        const f32x4 v2 = *(LAS f32x4*)(size_t)(park + 2048);
                        st_out(pend_o + 16 * NL + 32, v2);
                    }
                    WAIT_L0();
                }
            }
            step(s + 1, fb, fa);
        }
        const bool bad = __builtin_amdgcn_readfirstlane((int)ctrl[4]) != 0;
        const int wm = wid & 1, wn = wid >> 1;
#pragma unroll
        for (int mb = 0; mb < 4; ++mb) {
            const unsigned row = wm * 64 + mb * 16 + (lane & 15);
#pragma unroll
            for (int nb = 0; nb < 4; ++nb) {
                const unsigned chunk = wn * 16 + nb * 4 + (lane >> 4);
                *(LAS f32x4*)(size_t)(lds0 + row * 512 + ((chunk ^ (row & 7u)) << 4)) = epi(acc[nb][mb], scale, bad);
            }
        }
    } else {
        unsigned vw[8];
        if (wid < 6) {
            const unsigned p8 = lane & 7, f0 = (unsigned)(lane >> 4) & 3u;
#pragma unroll
            for (int i = 0; i < 8; ++i) {
                const unsigned pi = (wid - 4) * 8 + i;
                vw[i] = (pi * 8 + (lane >> 3)) * NL + col0 + 16 * (p8 ^ (f0 | (((pi >> 1) & 1u) << 2))) - (i & 3) * 1024;
            }
        }
        const unsigned ldw = lds0 + LDS_W + (wid - 4) * 8192;
        auto issue_w = [&](int j, unsigned slot) {
            if (dbg_nodma) return;
            const unsigned so = (unsigned)(j & 31) * (BK * NL);
            dma16x4(ldw + slot, vw[0], vw[1], vw[2], vw[3], rw, so);
            dma16x4(ldw + slot + 4096, vw[4], vw[5], vw[6], vw[7], rw, so);
        };
        auto wait_w = [&](int s) {
            if (NWS == 4) { if (s + 3 < NS) { WAIT_V(16); } else if (s + 2 < NS) { WAIT_V(8); } else { WAIT_V(0); } }
            else { if (s + 2 < NS) { WAIT_V(8); } else { WAIT_V(0); } }
        };
        auto first_w = [&]() {
#pragma unroll
            for (int j = 0; j < NWS; ++j) issue_w(j, j * WB);
        };
        if (wid == 5) {
            first_w();
            BAR();
            if (NWS == 4) { WAIT_V(24); } else { WAIT_V(16); }
            BAR();
            unsigned cur = 0;
#pragma unroll 1
            for (int s = 0; s < NS; ++s) {
                wait_w(s);
                BAR();
                if (s + NWS < NS) issue_w(s + NWS, cur);
                cur = next_w(cur);
            }
        } else {
            const int xw = wid & 1;
            int nix = 0;
            unsigned isl = 0;
            const i32x4 rxl = make_rsrc(p.x + (size_t)lrow * KT, 64 * KT);
            unsigned vl[8];
            {
                const unsigned p8 = lane & 7, f0 = (unsigned)(lane >> 4);
#pragma unroll
                for (int i = 0; i < 8; ++i)
                    vl[i] = (i * 8 + (lane >> 3)) * KT + 16 * (p8 ^ ((f0 + 4u * (i & 1)) & 7u)) - (i & 3) * 1024;
            }
            auto issue_local = [&]() {
                const unsigned so = (unsigned)nix * BK;
                if (!dbg_nodma && wid == 6) {
                    dma16x4(lds0 + isl, vl[0], vl[1], vl[2], vl[3], rxl, so);
                    dma16x4(lds0 + isl + 4096, vl[4], vl[5], vl[6], vl[7], rxl, so);
                }
                isl = next_x(isl);
                ++nix;
            };
            unsigned cnt_old;
            asm volatile("global_load_dword %0, %1, off sc1" : "=v"(cnt_old) : "v"(cntp) : "memory");
            const unsigned posv = p.sched.pos[(gg + 2 * (lane >> 5)) * 32 + (lane & 31)];
            if (wid == 4) first_w();
            else if (nloc) {
#pragma unroll 1
                for (int j = 0; j < NXS - 1; ++j) issue_local();
            }
            WAIT_V(0);
            asm volatile("" : "+v"(cnt_old));
            const unsigned call = (unsigned)__builtin_amdgcn_readfirstlane(cnt_old) + 1u;
            const unsigned pbase = call * 64u;
            const unsigned pj = posv & 3u;
            const unsigned need = pbase + (posv >> 2) + 1u;
            auto ready_count = [&](unsigned v) -> int {
                const unsigned long long m = __builtin_amdgcn_ballot_w64((int)(v - need) >= 0);
                return nloc + (m == ~0ull ? 64 : __builtin_ctzll(~m));
            };
            int R = dbg_nowait ? NS : nloc;
            auto refresh = [&]() {
                const int r = ready_count(ctrl[pj]);
                if (r > R) R = r;
            };
            if (wid == 4) {
                __builtin_amdgcn_fence(__ATOMIC_ACQUIRE, "");
                if (lane < 8) ctrl[lane] = lane < 4 ? pbase : 0u;
                const i32x4 rf = make_rsrc(prog, 16);
                auto watch = [&](int st) {
                    if (R < NS) refresh();
                    if (R > st || st >= NS) return;
                    unsigned long long t0 = __builtin_amdgcn_s_memrealtime();
                    for (unsigned it = 1;; ++it) {
                        const unsigned v = __hip_atomic_load(prog + pj, __ATOMIC_RELAXED, __HIP_MEMORY_SCOPE_SYSTEM);
                        const int r = ready_count(v);
                        if (r > R) { R = r; ctrl[pj] = v; }
                        if (R > st) break;
                        if ((it & 15u) == 0u) {
                            const unsigned tv = (unsigned)__builtin_amdgcn_readfirstlane(
                                __hip_atomic_load(tmo, __ATOMIC_RELAXED, __HIP_MEMORY_SCOPE_AGENT));
                            if (tv == call || __builtin_amdgcn_s_memrealtime() - t0 > DEADLINE_TICKS) {
                                if (lane == 0) __hip_atomic_store(tmo, call, __ATOMIC_RELAXED, __HIP_MEMORY_SCOPE_AGENT);
                                if (lane == 0) ctrl[4] = 1u;
                                if (lane < 4) ctrl[lane] = pbase + 63u;
                                R = NS;
                                break;
                            }
                        }
                        __builtin_amdgcn_s_sleep(1);
                    }
                };
                WAIT_V(0);
                WAIT_L0();
                BAR();
                unsigned cur = 0;
#pragma unroll 1
                for (int s = -1; s < NS; ++s) {
                    watch(s + 1);
                    if (s >= 0) wait_w(s);
                    WAIT_L0();
                    BAR();
                    if (R < NS) { if (lane < 4) dma4_sys(lds0 + LDS_CTRL, (unsigned)lane * 4u, rf); }
                    if (s >= 0) {
                        if (s + NWS < NS) issue_w(s + NWS, cur);
                        cur = next_w(cur);
                    }
                }
                if (lane == 0) __hip_atomic_store(cntp, call, __ATOMIC_RELAXED, __HIP_MEMORY_SCOPE_AGENT);
            } else {
                const i32x4 rxr = make_rsrc(wsl + WS_STAGE + ((size_t)(call & 1u) * NR + src) * SHARD_BYTES, SHARD_BYTES);
                const unsigned vr = xw * 8192 + lane * 16;
                const unsigned ldx = lds0 + xw * 8192;
                auto issue_x = [&]() {
                    if (nix < nloc) { issue_local(); return; }
                    if (!dbg_nodma) {
                        const int jr = nix - nloc;
                        const unsigned so = (unsigned)((gg + 2 * (jr >> 5)) * 32 + (jr & 31)) * XB;
                        dma16x4(ldx + isl, vr, vr, vr, vr, rxr, so);
                        dma16x4(ldx + isl + 4096, vr, vr, vr, vr, rxr, so + 4096);
                    }
                    isl = next_x(isl);
                    ++nix;
                };
                BAR();
#pragma unroll 1
                for (int s = -1; s < NS; ++s) {
                    const int st = s + 1;
                    if (st < NS) {
                        if (R < NS) refresh();
                        while (R <= st) { __builtin_amdgcn_s_sleep(1); refresh(); }
                        const int lim = s + NXS < NS ? s + NXS : NS;
                        const int l = R < lim ? R : lim;
                        while (nix < l) issue_x();
                        const int ahead = nix - 1 - st;
                        if (ahead >= 3) {
                            if (ahead >= 4) { WAIT_V(32); } else { WAIT_V(24); }
                        } else if (ahead == 2) { WAIT_V(16); } else if (ahead == 1) { WAIT_V(8); } else { WAIT_V(0); }
                    }
                    WAIT_L0();
                    BAR();
                    if (nix < R && nix < NS && nix <= s + NXS) issue_x();
                }
            }
        }
    }
    WAIT_L0();
    BAR();
    {
        int tz = threadIdx.x;
        asm volatile("" : "+v"(tz));
        const int ln = tz & 63, wv = tz >> 6;
        float* o = p.out + (size_t)(row_last + wv * 16 + (ln >> 5)) * NL + col0 + (ln & 31) * 4;
        f32x4 v[8];
#pragma unroll
        for (int i = 0; i < 8; ++i) {
            const unsigned row = wv * 16 + 2 * i + (ln >> 5);
            v[i] = *(LAS f32x4*)(size_t)(lds0 + row * 512 + (((unsigned)(ln & 31) ^ (row & 7u)) << 4));
        }
#pragma unroll
        for (int i = 0; i < 8; ++i) {
            float* dst = o + (size_t)i * 2 * NL;
            asm volatile("global_store_dwordx4 %0, %1, off sc1\n\ts_nop 1" :: "v"(dst), "v"(v[i]) : "memory");
        }
    }
}

}

extern "C" void kernel_launch(int rank, int nranks, ncclComm_t comm,
                              hipStream_t stream,
                              void* const* d_in, const int* in_sizes, int n_in,
                              void* d_out, int out_size,
                              const int* in_dtypes, int out_dtype,
                              void* const* d_out_all,
                              void* const* d_ws_all, size_t ws_bytes) {
    using namespace agk;
    (void)comm; (void)in_sizes; (void)n_in; (void)out_size; (void)in_dtypes; (void)out_dtype; (void)d_out_all;
    if (nranks != NR || ws_bytes < WS_NEEDED || rank < 0 || rank >= NR) {
        fprintf(stderr, "unexpected configuration\n");
        return;
    }
    static bool attr_set[NR] = {};
    if (!attr_set[rank]) {
        HIP_CHECK(hipFuncSetAttribute((const void*)ag_gemm_fused, hipFuncAttributeMaxDynamicSharedMemorySize, LDS_BYTES));
        attr_set[rank] = true;
    }
    static const Sched sched = make_sched();
    Params p;
    p.x = (const uint8_t*)d_in[0];
    p.w = (const uint8_t*)d_in[1];
    p.sx = (const float*)d_in[2];
    p.sw = (const float*)d_in[3];
    p.out = (float*)d_out;
    for (int i = 0; i < NR; ++i) p.ws[i] = (uint8_t*)d_ws_all[i];
    p.rank = rank;
    p.pad = 0;
    p.sched = sched;
    hipLaunchKernelGGL(ag_gemm_fused, dim3(NWG), dim3(512), LDS_BYTES, stream, p);
    HIP_CHECK(hipGetLastError());
}
```

```cpp
#include <hip/hip_runtime.h>
#include <rccl/rccl.h>
#include <cstdio>
#include <cstdint>

#define HIP_CHECK(expr) do { hipError_t _e = (expr); if (_e != hipSuccess) { \
    fprintf(stderr, "HIP error %s at %s:%d\n", hipGetErrorString(_e), __FILE__, __LINE__); \
    return; } } while (0)

namespace agk {

#define LAS __attribute__((address_space(3)))
typedef int i32x8 __attribute__((ext_vector_type(8)));
typedef int i32x4 __attribute__((ext_vector_type(4)));
typedef int i32x2 __attribute__((ext_vector_type(2)));
typedef unsigned u32x4 __attribute__((ext_vector_type(4)));
typedef float f32x4 __attribute__((ext_vector_type(4)));

constexpr int NR = 8;
constexpr int KT = 4096, NL = 2048, ML = 512;
constexpr int BM = 128, BN = 128, BK = 128;
constexpr int NSTEP = KT / BK;
constexpr int XB = BM * BK;
constexpr int WB = BK * BN;
#define NXS 5
#define NWS 4
constexpr int LDS_W = NXS * XB;
constexpr int LDS_CTRL = LDS_W + NWS * WB;
constexpr int LDS_PARK = LDS_CTRL + 2048;
constexpr int LDS_BYTES = LDS_PARK + 12288;
static_assert(LDS_BYTES <= 163840, "LDS");
constexpr int NWG = 256;
constexpr unsigned SHARD_BYTES = ML * KT;
constexpr unsigned W_BYTES = KT * NL;

constexpr size_t WS_CNT = 0;
constexpr size_t WS_TMO = WS_CNT + NWG * 128;
constexpr size_t WS_PROG = WS_TMO + 128;
constexpr size_t WS_DBG = 512u << 10;
constexpr size_t WS_STAGE = 1u << 20;
constexpr size_t WS_NEEDED = WS_STAGE + 2ull * NR * SHARD_BYTES;
constexpr unsigned long long DEADLINE_TICKS = 20000000ull;

#define SK_START {0, 8, 1024, 1032}
#define SK_RATE {16, 16, 16, 16}
struct Sched {
    unsigned char ord[4][32];
    unsigned char pos[128];
};
constexpr Sched make_sched() {
    Sched s{};
    const int start[4] = SK_START;
    const int rate[4] = SK_RATE;
    int nxt[4] = {0, 0, 0, 0};
    for (int p = 0; p < 128; ++p) {
        int best = -1, bk = 0;
        for (int g = 0; g < 4; ++g) {
            if (nxt[g] >= 32) continue;
            const int key = start[g] + nxt[g] * rate[g];
            if (best < 0 || key < bk) { best = g; bk = key; }
        }
        s.ord[p & 3][p >> 2] = (unsigned char)(best * 32 + nxt[best]);
        s.pos[best * 32 + nxt[best]] = (unsigned char)p;
        ++nxt[best];
    }
    return s;
}

struct Params {
    const uint8_t* x;
    const uint8_t* w;
    const float* sx;
    const float* sw;
    float* out;
    uint8_t* ws[NR];
    int rank;
    int pad;
    Sched sched;
};

#define WAIT_V(n) asm volatile("s_waitcnt vmcnt(" #n ")" ::: "memory")
#define WAIT_L0() do { asm volatile("" ::: "memory"); __builtin_amdgcn_s_waitcnt(0xC07F); asm volatile("" ::: "memory"); } while (0)
#define BAR() do { asm volatile("" ::: "memory"); __builtin_amdgcn_s_barrier(); asm volatile("" ::: "memory"); } while (0)
#define SCHED() __builtin_amdgcn_sched_barrier(0)

__device__ __forceinline__ i32x4 make_rsrc(const void* p, unsigned bytes) {
    const unsigned long long a = (unsigned long long)p;
    i32x4 r;
    r.x = __builtin_amdgcn_readfirstlane((int)(unsigned)a);
    r.y = __builtin_amdgcn_readfirstlane((int)((unsigned)(a >> 32) & 0xffffu));
    r.z = __builtin_amdgcn_readfirstlane((int)bytes);
    r.w = 0x00020000;
    return r;
}
__device__ __forceinline__ void dma16(unsigned lds, unsigned voff, i32x4 rsrc, unsigned soff) {
    asm volatile("s_mov_b32 m0, %0\n\ts_nop 0\n\tbuffer_load_dwordx4 %1, %2, %3 offen lds"
                 :: "s"(lds), "v"(voff), "s"(rsrc), "s"(soff) : "memory");
}
__device__ __forceinline__ void dma16x4(unsigned lds, unsigned v0, unsigned v1, unsigned v2, unsigned v3, i32x4 rsrc, unsigned soff) {
    asm volatile("s_mov_b32 m0, %0\n\ts_nop 0\n\t"
                 "buffer_load_dwordx4 %1, %5, %6 offen lds\n\t"
                 "buffer_load_dwordx4 %2, %5, %6 offen offset:1024 lds\n\t"
                 "buffer_load_dwordx4 %3, %5, %6 offen offset:2048 lds\n\t"
                 "buffer_load_dwordx4 %4, %5, %6 offen offset:3072 lds"
                 :: "s"(lds), "v"(v0), "v"(v1), "v"(v2), "v"(v3), "s"(rsrc), "s"(soff) : "memory");
}
__device__ __forceinline__ void dma4_sys(unsigned lds, unsigned voff, i32x4 rsrc) {
    asm volatile("s_mov_b32 m0, %0\n\ts_nop 0\n\tbuffer_load_dword %1, %2, 0 offen sc0 sc1 lds"
                 :: "s"(lds), "v"(voff), "s"(rsrc) : "memory");
}
__device__ __forceinline__ void ld16_asm(u32x4& d, unsigned voff, i32x4 rsrc, unsigned soff) {
    asm volatile("buffer_load_dwordx4 %0, %1, %2, %3 offen" : "=v"(d) : "v"(voff), "s"(rsrc), "s"(soff) : "memory");
}
__device__ __forceinline__ void st16_sys(const u32x4& d, unsigned voff, i32x4 rsrc, unsigned soff) {
    asm volatile("buffer_store_dwordx4 %0, %1, %2, %3 offen sc0 sc1\n\ts_nop 1"
                 :: "v"(d), "v"(voff), "s"(rsrc), "s"(soff) : "memory");
}
__device__ __forceinline__ void wait_vm(int n) {
    if (n >= 12) {
        if (n >= 20) { WAIT_V(20); } else if (n >= 16) { WAIT_V(16); } else { WAIT_V(12); }
    } else {
        if (n >= 8) { WAIT_V(8); } else if (n >= 4) { WAIT_V(4); } else { WAIT_V(0); }
    }
}

struct Frags { i32x8 x[4]; i32x8 w[4]; };
__device__ __forceinline__ void put4(i32x8& d, int i, i32x4 v) { d[4 * i] = v.x; d[4 * i + 1] = v.y; d[4 * i + 2] = v.z; d[4 * i + 3] = v.w; }
__device__ __forceinline__ void put2(i32x8& d, int i, i32x2 v) { d[2 * i] = v.x; d[2 * i + 1] = v.y; }

__device__ __forceinline__ void read_w1(unsigned aw, Frags& f, int nb, int q) {
    put2(f.w[nb], q, __builtin_amdgcn_ds_read_tr8_b64_v2i32((LAS i32x2*)(size_t)((aw ^ (unsigned)(nb << 4)) + ((q >> 1) * 64 + (q & 1) * 8) * BN)));
}
__device__ __forceinline__ void read_x1(unsigned ax, Frags& f, int mb, int q) {
    put4(f.x[mb], q, *(LAS i32x4*)(size_t)((q ? (ax ^ 64u) : ax) + mb * 2048));
}
__device__ __forceinline__ void read_nth(unsigned ax, unsigned aw, Frags& f, int r) {
    if (r < 4) read_w1(aw, f, 0, r);
    else if (r < 12) read_x1(ax, f, (r - 4) >> 1, (r - 4) & 1);
    else read_w1(aw, f, 1 + ((r - 12) >> 2), (r - 12) & 3);
}
__device__ __forceinline__ void mma1(f32x4 (&acc)[4][4], const Frags& f, int nb, int mb) {
    acc[nb][mb] = __builtin_amdgcn_mfma_scale_f32_16x16x128_f8f6f4(f.w[nb], f.x[mb], acc[nb][mb], 1, 1, 0, 0, 0, 0);
}
__device__ __forceinline__ unsigned next_x(unsigned soff) { return soff == (NXS - 1) * XB ? 0u : soff + XB; }
__device__ __forceinline__ unsigned next_w(unsigned soff) { return soff == (NWS - 1) * WB ? 0u : soff + WB; }

__device__ __forceinline__ f32x4 epi(f32x4 a, float scale, bool bad) {
    f32x4 v = a * scale;
    v.x = v.x < 0.f ? 0.f : v.x;
    v.y = v.y < 0.f ? 0.f : v.y;
    v.z = v.z < 0.f ? 0.f : v.z;
    v.w = v.w < 0.f ? 0.f : v.w;
    const float qnan = __builtin_nanf("");
    if (bad) v = (f32x4){qnan, qnan, qnan, qnan};
    return v;
}
#define MID_ST 0
__device__ __forceinline__ void st_out(float* dst, f32x4 v) {
    asm volatile("global_store_dwordx4 %0, %1, off\n\ts_nop 1" :: "v"(dst), "v"(v) : "memory");
}
__device__ __forceinline__ float* store_tile(f32x4 (&acc)[4][4], f32x4 (&pend)[8], unsigned park, float* out, int row0, int col0,
                                             float scale, bool bad) {
    int tz = threadIdx.x;
    asm volatile("" : "+v"(tz));
    const int ln = tz & 63, wv = tz >> 6;
    const int wm = wv & 1, wn = wv >> 1;
    float* o = out + (size_t)(row0 + wm * 64 + (ln & 15)) * NL + (col0 + wn * 64 + 4 * (ln >> 4));
#pragma unroll
    for (int mb = 0; mb < 4; ++mb)
#pragma unroll
        for (int nb = 0; nb < 4; ++nb) {
            const f32x4 v = epi(acc[nb][mb], scale, bad);
            if (mb == 0 || (mb == 1 && nb == 3)) st_out(o + (size_t)mb * 16 * NL + nb * 16, v);
            else if (mb == 1) *(LAS f32x4*)(size_t)(park + nb * 1024) = v;
            else pend[(mb - 2) * 4 + nb] = v;
            acc[nb][mb] = (f32x4){0.f, 0.f, 0.f, 0.f};
        }
    return o;
}

#define PRE_BAR 2
#define AGK_MODE 0
#define OUT_ST 2

__global__ void __launch_bounds__(512, 2) ag_gemm_fused(Params p) {
    extern __shared__ __attribute__((aligned(16))) char smem[];
    constexpr bool dbg_nowait = (AGK_MODE & 1) != 0, dbg_nopush = (AGK_MODE & 2) != 0;
    constexpr bool dbg_nodma = (AGK_MODE & 4) != 0, dbg_nolocal = (AGK_MODE & 16) != 0;
    if (AGK_MODE & 64) return;
    const int tid = threadIdx.x, lane = tid & 63;
    const int wid = __builtin_amdgcn_readfirstlane(tid >> 6);
    const int b = blockIdx.x;
    const int ct = b & 7, rt = b >> 3;
    const int rank = p.rank;
    uint8_t* wsl = p.ws[rank];
    unsigned* cntp = (unsigned*)(wsl + WS_CNT + (size_t)b * 128);

    if (rt < 4) {
        if (ct == NR - 1 || dbg_nopush) return;
        const int peer = (rank + 1 + ct) & (NR - 1);
        uint8_t* wsp = p.ws[peer];
        unsigned cnt_old;
        asm volatile("global_load_dword %0, %1, off sc1" : "=v"(cnt_old) : "v"(cntp) : "memory");
        const i32x4 rs = make_rsrc(p.x, SHARD_BYTES);
        const unsigned vs = (wid * 16 + (lane >> 3)) * KT + (lane & 7) * 16;
        const unsigned vd0 = (wid * 16 + (lane >> 3)) * BK + (((unsigned)lane & 7u) ^ ((unsigned)lane >> 4)) * 16;
        const unsigned vd1 = (wid * 16 + 8 + (lane >> 3)) * BK + (((unsigned)lane & 7u) ^ (4u + ((unsigned)lane >> 4))) * 16;
        const unsigned* ow = (const unsigned*)p.sched.ord[rt];
        u32x4 pb[4][2];
        auto push_load = [&](unsigned u, u32x4 (&d)[2]) {
            const unsigned so = (u >> 5) * (BM * KT) + (u & 31u) * BK;
            ld16_asm(d[0], vs, rs, so);
            ld16_asm(d[1], vs, rs, so + 8 * KT);
        };
        unsigned wc = ow[0];
        push_load(wc & 255u, pb[0]);
        push_load((wc >> 8) & 255u, pb[1]);
        push_load((wc >> 16) & 255u, pb[2]);
        push_load(wc >> 24, pb[3]);
        WAIT_V(8);
        asm volatile("" : "+v"(cnt_old));
        const unsigned call = (unsigned)__builtin_amdgcn_readfirstlane(cnt_old) + 1u;
        const unsigned pbase = call * 64u;
        const i32x4 rd = make_rsrc(wsp + WS_STAGE + ((size_t)(call & 1u) * NR + rank) * SHARD_BYTES, SHARD_BYTES);
        unsigned* flag = (unsigned*)(wsp + WS_PROG + (size_t)rank * 128) + rt;
        auto publish = [&](int done) {
            BAR();
            if (tid == 0) __hip_atomic_store(flag, pbase + (unsigned)done, __ATOMIC_RELAXED, __HIP_MEMORY_SCOPE_SYSTEM);
        };
        auto push_store = [&](unsigned u, u32x4 (&d)[2]) {
            st16_sys(d[0], vd0, rd, u * XB);
            st16_sys(d[1], vd1, rd, u * XB);
        };
#define PUSH_D 1
        {
            const unsigned wn = ow[1];
            WAIT_V(6); push_store(wc & 255u, pb[0]); push_load(wn & 255u, pb[0]);
            WAIT_V(8); push_store((wc >> 8) & 255u, pb[1]); push_load((wn >> 8) & 255u, pb[1]);
            WAIT_V(6); publish(1);
            WAIT_V(10); push_store((wc >> 16) & 255u, pb[2]); push_load((wn >> 16) & 255u, pb[2]);
            WAIT_V(6); publish(2);
            WAIT_V(12); push_store(wc >> 24, pb[3]); push_load(wn >> 24, pb[3]);
            WAIT_V(6); publish(3);
            wc = wn;
        }
#pragma unroll 1
        for (int t = 4; t < NSTEP - 4; t += 4) {
            const unsigned wn = ow[(t >> 2) + 1];
#pragma unroll
            for (int i = 0; i < 4; ++i) {
                push_store((wc >> (8 * i)) & 255u, pb[i]);
                push_load((wn >> (8 * i)) & 255u, pb[i]);
                WAIT_V(6); publish(t + i);
            }
            wc = wn;
        }
        push_store(wc & 255u, pb[0]);          WAIT_V(4); publish(28);
        push_store((wc >> 8) & 255u, pb[1]);   WAIT_V(2); publish(29);
        push_store((wc >> 16) & 255u, pb[2]);  WAIT_V(2); publish(30);
        push_store(wc >> 24, pb[3]);           WAIT_V(2); publish(31);
        WAIT_V(0); publish(32);
        if (tid == 0) __hip_atomic_store(cntp, call, __ATOMIC_RELAXED, __HIP_MEMORY_SCOPE_AGENT);
        return;
    }

    const int idx = rt - 4;
    const int sh = 1 + (idx >> 2), gg = (idx >> 1) & 1, half = idx & 1;
    const int src = (rank + sh) & (NR - 1);
    const int nloc = (idx < 16 && !dbg_nolocal) ? NSTEP : 0;
    const int lrow = idx * 32 & ~63;
    const int NS = nloc + 2 * NSTEP;
    const int col0 = ct * 256 + half * BN;
    const unsigned lds0 = (unsigned)(uintptr_t)(LAS char*)smem;
    volatile LAS unsigned* ctrl = (volatile LAS unsigned*)(LAS char*)(smem + LDS_CTRL);

    const i32x4 rw = make_rsrc(p.w, W_BYTES);
    unsigned* prog = (unsigned*)(wsl + WS_PROG + (size_t)src * 128);
    unsigned* tmo = (unsigned*)(wsl + WS_TMO);
    const int row_last = src * ML + (gg + 2) * BM;
    float scale = 0.f;

    if (wid < 4) {
        scale = __builtin_bit_cast(float, __builtin_amdgcn_readfirstlane(__builtin_bit_cast(int, p.sx[0] * p.sw[0])));
        if (idx >= 16) {
            const i32x4 rxp = make_rsrc(p.x, SHARD_BYTES);
            const unsigned vw = (unsigned)(tid >> 4) * NL + ct * 256 + (tid & 15) * 16;
            const unsigned vx = (unsigned)(tid >> 3) * KT + (tid & 7) * 16;
            u32x4 d0, d1, d2, d3;
            for (int ks = idx - 16; ks < NSTEP; ks += 12) {
                const unsigned so = (unsigned)ks * (BK * NL), sx = (unsigned)ks * BK;
#pragma unroll
                for (int r = 0; r < 8; r += 4) {
                    ld16_asm(d0, vw, rw, so + (r + 0) * 16 * NL); ld16_asm(d1, vw, rw, so + (r + 1) * 16 * NL);
                    ld16_asm(d2, vw, rw, so + (r + 2) * 16 * NL); ld16_asm(d3, vw, rw, so + (r + 3) * 16 * NL);
                }
#pragma unroll
                for (int r = 0; r < 16; r += 4) {
                    ld16_asm(d0, vx, rxp, sx + (r + 0) * 32 * KT); ld16_asm(d1, vx, rxp, sx + (r + 1) * 32 * KT);
                    ld16_asm(d2, vx, rxp, sx + (r + 2) * 32 * KT); ld16_asm(d3, vx, rxp, sx + (r + 3) * 32 * KT);
                }
            }
            WAIT_V(0);
            asm volatile("" :: "v"(d0), "v"(d1), "v"(d2), "v"(d3));
        }
        unsigned axl, awl;
        {
            const int wm = wid & 1, wn = wid >> 1;
            const unsigned m = lane & 15, gq = lane >> 4, q = m >> 1, h = m & 1;
            axl = lds0 + (wm * 64 + m) * BK + 16 * (gq ^ (m >> 1));
            awl = lds0 + LDS_W + (16 * gq + q) * BN + 16 * ((unsigned)(wn * 4) ^ ((q >> 1) | ((gq & 1) << 2))) + 8 * h;
        }
        f32x4 acc[4][4];
#pragma unroll
        for (int i = 0; i < 4; ++i)
#pragma unroll
            for (int j = 0; j < 4; ++j) acc[i][j] = (f32x4){0.f, 0.f, 0.f, 0.f};
        Frags fa, fb;
        f32x4 pend[8];
#pragma unroll
        for (int i = 0; i < 8; ++i) pend[i] = (f32x4){0.f, 0.f, 0.f, 0.f};
        float* pend_o = nullptr;
        const unsigned park = lds0 + LDS_PARK + wid * 3072 + lane * 16;
        BAR();
        BAR();
        unsigned cx = 0, cw = 0;
        if (nloc) {
            const unsigned axs = axl - (wid & 1) * (32 * BK);
            auto lread = [&](unsigned ax, unsigned aw, Frags& f, int r) {
                if (r < 4) read_w1(aw, f, 0, r);
                else if (r < 8) read_x1(ax, f, (r - 4) >> 1, (r - 4) & 1);
                else read_w1(aw, f, 1 + ((r - 8) >> 2), (r - 8) & 3);
            };
#pragma unroll
            for (int r = 0; r < 20; ++r) lread(axs, awl, fa, r);
            auto lstep = [&](int s, Frags& fc, Frags& fn) {
                cx = next_x(cx); cw = next_w(cw);
                const unsigned ax = axs + cx, aw = awl + cw;
                WAIT_L0();
                BAR();
                SCHED();
#pragma unroll
                for (int i = 0; i < 8; ++i) {
                    mma1(acc, fc, i >> 1, i & 1);
#pragma unroll
                    for (int r = 0; r < 3; ++r) if (3 * i + r < 20) lread(ax, aw, fn, 3 * i + r);
                    SCHED();
                }
            };
#pragma unroll 1
            for (int s = 0; s < NSTEP; s += 2) {
                lstep(s, fa, fb);
                lstep(s + 1, fb, fa);
            }
            int tz = threadIdx.x;
            asm volatile("" : "+v"(tz));
            const int ln = tz & 63, wv = tz >> 6;
            float* o = p.out + (size_t)(rank * ML + lrow + (wv & 1) * 32 + (ln & 15)) * NL + (col0 + (wv >> 1) * 64 + 4 * (ln >> 4));
#pragma unroll
            for (int mb = 0; mb < 2; ++mb)
#pragma unroll
                for (int nb = 0; nb < 4; ++nb) {
                    pend[mb * 4 + nb] = epi(acc[nb][mb], scale, false);
                    acc[nb][mb] = (f32x4){0.f, 0.f, 0.f, 0.f};
                }
            pend_o = o;
            WAIT_L0();
        }
#pragma unroll
        for (int r = 0; r < 24; ++r) read_nth(axl + cx, awl + cw, fa, r);
        auto step = [&](int s, Frags& fc, Frags& fn) {
            cx = next_x(cx); cw = next_w(cw);
            const unsigned ax = axl + cx, aw = awl + cw;
            WAIT_L0();
            SCHED();
#pragma unroll
            for (int i = 0; i < PRE_BAR; ++i) { mma1(acc, fc, i >> 2, i & 3); SCHED(); }
            BAR();
            SCHED();
#pragma unroll
            for (int i = PRE_BAR; i < 16; ++i) {
                mma1(acc, fc, i >> 2, i & 3);
                if (2 * (i - PRE_BAR) < 24) { read_nth(ax, aw, fn, 2 * (i - PRE_BAR)); read_nth(ax, aw, fn, 2 * (i - PRE_BAR) + 1); }
                SCHED();
            }
            if (s == nloc + NSTEP - 1) {
                const bool bad = __builtin_amdgcn_readfirstlane((int)ctrl[4]) != 0;
                pend_o = store_tile(acc, pend, park, p.out, src * ML + gg * BM, col0, scale, bad);
            }
        };
#pragma unroll 1
        for (int s = nloc; s < NS; s += 2) {
            step(s, fa, fb);
            const int k1 = s - nloc, k2 = k1 - NSTEP;
            const int k = (nloc && k1 < NSTEP) ? k1 : k2;
            if (k >= 2 && k <= 12) {
                float* po = k2 >= 0 ? pend_o + (size_t)32 * NL : pend_o;
                if (k == 2) { st_out(po, pend[0]); st_out(po + 16, pend[1]); }
                else if (k == 4) { st_out(po + 32, pend[2]); st_out(po + 48, pend[3]); }
                else if (k == 6) { st_out(po + 16 * NL, pend[4]); st_out(po + 16 * NL + 16, pend[5]); }
                else if (k == 8) { st_out(po + 16 * NL + 32, pend[6]); st_out(po + 16 * NL + 48, pend[7]); }
                else if (k2 >= 0) {
                    if (k == 10) {
                        const f32x4 v0 = *(LAS f32x4*)(size_t)(park), v1 = *(LAS f32x4*)(size_t)(park + 1024);
                        st_out(pend_o + 16 * NL, v0);
                        st_out(pend_o + 16 * NL + 16, v1);
                    } else {
                        const f32x4 v2 = *(LAS f32x4*)(size_t)(park + 2048);
                        st_out(pend_o + 16 * NL + 32, v2);
                    }
                    WAIT_L0();
                }
            }
            step(s + 1, fb, fa);
        }
        const bool bad = __builtin_amdgcn_readfirstlane((int)ctrl[4]) != 0;
        const int wm = wid & 1, wn = wid >> 1;
#pragma unroll
        for (int mb = 0; mb < 4; ++mb) {
            const unsigned row = wm * 64 + mb * 16 + (lane & 15);
#pragma unroll
            for (int nb = 0; nb < 4; ++nb) {
                const unsigned chunk = wn * 16 + nb * 4 + (lane >> 4);
                *(LAS f32x4*)(size_t)(lds0 + row * 512 + ((chunk ^ (row & 7u)) << 4)) = epi(acc[nb][mb], scale, bad);
            }
        }
    } else {
        unsigned vw[8];
        if (wid < 6) {
            const unsigned p8 = lane & 7, f0 = (unsigned)(lane >> 4) & 3u;
#pragma unroll
            for (int i = 0; i < 8; ++i) {
                const unsigned pi = (wid - 4) * 8 + i;
                vw[i] = (pi * 8 + (lane >> 3)) * NL + col0 + 16 * (p8 ^ (f0 | (((pi >> 1) & 1u) << 2))) - (i & 3) * 1024;
            }
        }
        const unsigned ldw = lds0 + LDS_W + (wid - 4) * 8192;
        auto issue_w = [&](int j, unsigned slot) {
            if (dbg_nodma) return;
            const unsigned so = (unsigned)(j & 31) * (BK * NL);
            dma16x4(ldw + slot, vw[0], vw[1], vw[2], vw[3], rw, so);
            dma16x4(ldw + slot + 4096, vw[4], vw[5], vw[6], vw[7], rw, so);
        };
        auto wait_w = [&](int s) {
            if (NWS == 4) { if (s + 3 < NS) { WAIT_V(16); } else if (s + 2 < NS) { WAIT_V(8); } else { WAIT_V(0); } }
            else { if (s + 2 < NS) { WAIT_V(8); } else { WAIT_V(0); } }
        };
        auto first_w = [&]() {
#pragma unroll
            for (int j = 0; j < NWS; ++j) issue_w(j, j * WB);
        };
        if (wid == 5) {
            first_w();
            BAR();
            if (NWS == 4) { WAIT_V(24); } else { WAIT_V(16); }
            BAR();
            unsigned cur = 0;
#pragma unroll 1
            for (int s = 0; s < NS; ++s) {
                wait_w(s);
                BAR();
                if (s + NWS < NS) issue_w(s + NWS, cur);
                cur = next_w(cur);
            }
        } else {
            const int xw = wid & 1;
            int nix = 0;
            unsigned isl = 0;
            const i32x4 rxl = make_rsrc(p.x + (size_t)lrow * KT, 64 * KT);
            unsigned vl[8];
            {
                const unsigned p8 = lane & 7, f0 = (unsigned)(lane >> 4);
#pragma unroll
                for (int i = 0; i < 8; ++i)
                    vl[i] = (i * 8 + (lane >> 3)) * KT + 16 * (p8 ^ ((f0 + 4u * (i & 1)) & 7u)) - (i & 3) * 1024;
            }
            auto issue_local = [&]() {
                const unsigned so = (unsigned)nix * BK;
                if (!dbg_nodma && wid == 6) {
                    dma16x4(lds0 + isl, vl[0], vl[1], vl[2], vl[3], rxl, so);
                    dma16x4(lds0 + isl + 4096, vl[4], vl[5], vl[6], vl[7], rxl, so);
                }
                isl = next_x(isl);
                ++nix;
            };
            unsigned cnt_old;
            asm volatile("global_load_dword %0, %1, off sc1" : "=v"(cnt_old) : "v"(cntp) : "memory");
            const unsigned posv = p.sched.pos[(gg + 2 * (lane >> 5)) * 32 + (lane & 31)];
            if (wid == 4) first_w();
            else if (nloc) {
#pragma unroll 1
                for (int j = 0; j < NXS - 1; ++j) issue_local();
            }
            WAIT_V(0);
            asm volatile("" : "+v"(cnt_old));
            const unsigned call = (unsigned)__builtin_amdgcn_readfirstlane(cnt_old) + 1u;
            const unsigned pbase = call * 64u;
            const unsigned pj = posv & 3u;
            const unsigned need = pbase + (posv >> 2) + 1u;
            auto ready_count = [&](unsigned v) -> int {
                const unsigned long long m = __builtin_amdgcn_ballot_w64((int)(v - need) >= 0);
                return nloc + (m == ~0ull ? 64 : __builtin_ctzll(~m));
            };
            int R = dbg_nowait ? NS : nloc;
            auto refresh = [&]() {
                const int r = ready_count(ctrl[pj]);
                if (r > R) R = r;
            };
            if (wid == 4) {
                __builtin_amdgcn_fence(__ATOMIC_ACQUIRE, "");
                if (lane < 8) ctrl[lane] = lane < 4 ? pbase : 0u;
                const i32x4 rf = make_rsrc(prog, 16);
                auto watch = [&](int st) {
                    if (R < NS) refresh();
                    if (R > st || st >= NS) return;
                    unsigned long long t0 = __builtin_amdgcn_s_memrealtime();
                    for (unsigned it = 1;; ++it) {
                        const unsigned v = __hip_atomic_load(prog + pj, __ATOMIC_RELAXED, __HIP_MEMORY_SCOPE_SYSTEM);
                        const int r = ready_count(v);
                        if (r > R) { R = r; ctrl[pj] = v; }
                        if (R > st) break;
                        if ((it & 15u) == 0u) {
                            const unsigned tv = (unsigned)__builtin_amdgcn_readfirstlane(
                                __hip_atomic_load(tmo, __ATOMIC_RELAXED, __HIP_MEMORY_SCOPE_AGENT));
                            if (tv == call || __builtin_amdgcn_s_memrealtime() - t0 > DEADLINE_TICKS) {
                                if (lane == 0) __hip_atomic_store(tmo, call, __ATOMIC_RELAXED, __HIP_MEMORY_SCOPE_AGENT);
                                if (lane == 0) ctrl[4] = 1u;
                                if (lane < 4) ctrl[lane] = pbase + 63u;
                                R = NS;
                                break;
                            }
                        }
                        __builtin_amdgcn_s_sleep(1);
                    }
                };
                WAIT_V(0);
                WAIT_L0();
                BAR();
                unsigned cur = 0;
#pragma unroll 1
                for (int s = -1; s < NS; ++s) {
                    watch(s + 1);
                    if (s >= 0) wait_w(s);
                    WAIT_L0();
                    BAR();
                    if (R < NS) { if (lane < 4) dma4_sys(lds0 + LDS_CTRL, (unsigned)lane * 4u, rf); }
                    if (s >= 0) {
                        if (s + NWS < NS) issue_w(s + NWS, cur);
                        cur = next_w(cur);
                    }
                }
                if (lane == 0) __hip_atomic_store(cntp, call, __ATOMIC_RELAXED, __HIP_MEMORY_SCOPE_AGENT);
            } else {
                const i32x4 rxr = make_rsrc(wsl + WS_STAGE + ((size_t)(call & 1u) * NR + src) * SHARD_BYTES, SHARD_BYTES);
                const unsigned vr = xw * 8192 + lane * 16;
                const unsigned ldx = lds0 + xw * 8192;
                auto issue_x = [&]() {
                    if (nix < nloc) { issue_local(); return; }
                    if (!dbg_nodma) {
                        const int jr = nix - nloc;
                        const unsigned so = (unsigned)((gg + 2 * (jr >> 5)) * 32 + (jr & 31)) * XB;
                        dma16x4(ldx + isl, vr, vr, vr, vr, rxr, so);
                        dma16x4(ldx + isl + 4096, vr, vr, vr, vr, rxr, so + 4096);
                    }
                    isl = next_x(isl);
                    ++nix;
                };
                BAR();
#pragma unroll 1
                for (int s = -1; s < NS; ++s) {
                    const int st = s + 1;
                    if (st < NS) {
                        if (R < NS) refresh();
                        while (R <= st) { __builtin_amdgcn_s_sleep(1); refresh(); }
                        const int lim = s + NXS < NS ? s + NXS : NS;
                        const int l = R < lim ? R : lim;
                        while (nix < l) issue_x();
                        const int ahead = nix - 1 - st;
                        if (ahead >= 3) {
                            if (ahead >= 4) { WAIT_V(32); } else { WAIT_V(24); }
                        } else if (ahead == 2) { WAIT_V(16); } else if (ahead == 1) { WAIT_V(8); } else { WAIT_V(0); }
                    }
                    WAIT_L0();
                    BAR();
                    if (nix < R && nix < NS && nix <= s + NXS) issue_x();
                }
            }
        }
    }
    WAIT_L0();
    BAR();
    {
        int tz = threadIdx.x;
        asm volatile("" : "+v"(tz));
        const int ln = tz & 63, wv = tz >> 6;
        float* o = p.out + (size_t)(row_last + wv * 16 + (ln >> 5)) * NL + col0 + (ln & 31) * 4;
        f32x4 v[8];
#pragma unroll
        for (int i = 0; i < 8; ++i) {
            const unsigned row = wv * 16 + 2 * i + (ln >> 5);
            v[i] = *(LAS f32x4*)(size_t)(lds0 + row * 512 + (((unsigned)(ln & 31) ^ (row & 7u)) << 4));
        }
#pragma unroll
        for (int i = 0; i < 8; ++i) {
            float* dst = o + (size_t)i * 2 * NL;
            asm volatile("global_store_dwordx4 %0, %1, off sc1\n\ts_nop 1" :: "v"(dst), "v"(v[i]) : "memory");
        }
    }
}

}

extern "C" void kernel_launch(int rank, int nranks, ncclComm_t comm,
                              hipStream_t stream,
                              void* const* d_in, const int* in_sizes, int n_in,
                              void* d_out, int out_size,
                              const int* in_dtypes, int out_dtype,
                              void* const* d_out_all,
                              void* const* d_ws_all, size_t ws_bytes) {
    using namespace agk;
    (void)comm; (void)in_sizes; (void)n_in; (void)out_size; (void)in_dtypes; (void)out_dtype; (void)d_out_all;
    if (nranks != NR || ws_bytes < WS_NEEDED || rank < 0 || rank >= NR) {
        fprintf(stderr, "unexpected configuration\n");
        return;
    }
    static bool attr_set[NR] = {};
    if (!attr_set[rank]) {
        HIP_CHECK(hipFuncSetAttribute((const void*)ag_gemm_fused, hipFuncAttributeMaxDynamicSharedMemorySize, LDS_BYTES));
        attr_set[rank] = true;
    }
    static const Sched sched = make_sched();
    Params p;
    p.x = (const uint8_t*)d_in[0];
    p.w = (const uint8_t*)d_in[1];
    p.sx = (const float*)d_in[2];
    p.sw = (const float*)d_in[3];
    p.out = (float*)d_out;
    for (int i = 0; i < NR; ++i) p.ws[i] = (uint8_t*)d_ws_all[i];
    p.rank = rank;
    p.pad = 0;
    p.sched = sched;
    hipLaunchKernelGGL(ag_gemm_fused, dim3(NWG), dim3(512), LDS_BYTES, stream, p);
    HIP_CHECK(hipGetLastError());
}
```

```cpp
#include <hip/hip_runtime.h>
#include <rccl/rccl.h>
#include <cstdio>
#include <cstdint>

#define HIP_CHECK(expr) do { hipError_t _e = (expr); if (_e != hipSuccess) { \
    fprintf(stderr, "HIP error %s at %s:%d\n", hipGetErrorString(_e), __FILE__, __LINE__); \
    return; } } while (0)

namespace agk {

#define LAS __attribute__((address_space(3)))
typedef int i32x8 __attribute__((ext_vector_type(8)));
typedef int i32x4 __attribute__((ext_vector_type(4)));
typedef int i32x2 __attribute__((ext_vector_type(2)));
typedef unsigned u32x4 __attribute__((ext_vector_type(4)));
typedef float f32x4 __attribute__((ext_vector_type(4)));

constexpr int NR = 8;
constexpr int KT = 4096, NL = 2048, ML = 512;
constexpr int BM = 128, BN = 128, BK = 128;
constexpr int NSTEP = KT / BK;
constexpr int XB = BM * BK;
constexpr int WB = BK * BN;
constexpr int NXS = 4, NWS = 3;
constexpr int XLB = 64 * BK;
constexpr int LDS_W = NXS * XB;
constexpr int LDS_XL = LDS_W + NWS * WB;
constexpr int LDS_CTRL = LDS_XL + NWS * XLB;
constexpr int LDS_PARK = LDS_CTRL + 2048;
constexpr int LDS_BYTES = LDS_PARK + 12288;
static_assert(LDS_BYTES <= 163840, "LDS");
constexpr int NWG = 256;
constexpr unsigned SHARD_BYTES = ML * KT;
constexpr unsigned W_BYTES = KT * NL;

constexpr size_t WS_CNT = 0;
constexpr size_t WS_TMO = WS_CNT + NWG * 128;
constexpr size_t WS_PROG = WS_TMO + 128;
constexpr size_t WS_DBG = 512u << 10;
constexpr size_t WS_STAGE = 1u << 20;
constexpr size_t WS_NEEDED = WS_STAGE + 2ull * NR * SHARD_BYTES;
constexpr unsigned long long DEADLINE_TICKS = 20000000ull;

#define SK_START {0, 8, 1024, 1032}
#define SK_RATE {16, 16, 16, 16}
struct Sched {
    unsigned char ord[4][32];
    unsigned char pos[128];
};
constexpr Sched make_sched() {
    Sched s{};
    const int start[4] = SK_START;
    const int rate[4] = SK_RATE;
    int nxt[4] = {0, 0, 0, 0};
    for (int p = 0; p < 128; ++p) {
        int best = -1, bk = 0;
        for (int g = 0; g < 4; ++g) {
            if (nxt[g] >= 32) continue;
            const int key = start[g] + nxt[g] * rate[g];
            if (best < 0 || key < bk) { best = g; bk = key; }
        }
        s.ord[p & 3][p >> 2] = (unsigned char)(best * 32 + nxt[best]);
        s.pos[best * 32 + nxt[best]] = (unsigned char)p;
        ++nxt[best];
    }
    return s;
}

struct Params {
    const uint8_t* x;
    const uint8_t* w;
    const float* sx;
    const float* sw;
    float* out;
    uint8_t* ws[NR];
    int rank;
    int pad;
    Sched sched;
};

#define WAIT_V(n) asm volatile("s_waitcnt vmcnt(" #n ")" ::: "memory")
#define WAIT_L0() do { asm volatile("" ::: "memory"); __builtin_amdgcn_s_waitcnt(0xC07F); asm volatile("" ::: "memory"); } while (0)
#define BAR() do { asm volatile("" ::: "memory"); __builtin_amdgcn_s_barrier(); asm volatile("" ::: "memory"); } while (0)
#define SCHED() __builtin_amdgcn_sched_barrier(0)

__device__ __forceinline__ i32x4 make_rsrc(const void* p, unsigned bytes) {
    const unsigned long long a = (unsigned long long)p;
    i32x4 r;
    r.x = __builtin_amdgcn_readfirstlane((int)(unsigned)a);
    r.y = __builtin_amdgcn_readfirstlane((int)((unsigned)(a >> 32) & 0xffffu));
    r.z = __builtin_amdgcn_readfirstlane((int)bytes);
    r.w = 0x00020000;
    return r;
}
__device__ __forceinline__ void dma16(unsigned lds, unsigned voff, i32x4 rsrc, unsigned soff) {
    asm volatile("s_mov_b32 m0, %0\n\ts_nop 0\n\tbuffer_load_dwordx4 %1, %2, %3 offen lds"
                 :: "s"(lds), "v"(voff), "s"(rsrc), "s"(soff) : "memory");
}
__device__ __forceinline__ void dma16x4(unsigned lds, unsigned v0, unsigned v1, unsigned v2, unsigned v3, i32x4 rsrc, unsigned soff) {
    asm volatile("s_mov_b32 m0, %0\n\ts_nop 0\n\t"
                 "buffer_load_dwordx4 %1, %5, %6 offen lds\n\t"
                 "buffer_load_dwordx4 %2, %5, %6 offen offset:1024 lds\n\t"
                 "buffer_load_dwordx4 %3, %5, %6 offen offset:2048 lds\n\t"
                 "buffer_load_dwordx4 %4, %5, %6 offen offset:3072 lds"
                 :: "s"(lds), "v"(v0), "v"(v1), "v"(v2), "v"(v3), "s"(rsrc), "s"(soff) : "memory");
}
__device__ __forceinline__ void dma4_sys(unsigned lds, unsigned voff, i32x4 rsrc) {
    asm volatile("s_mov_b32 m0, %0\n\ts_nop 0\n\tbuffer_load_dword %1, %2, 0 offen sc0 sc1 lds"
                 :: "s"(lds), "v"(voff), "s"(rsrc) : "memory");
}
__device__ __forceinline__ void ld16_asm(u32x4& d, unsigned voff, i32x4 rsrc, unsigned soff) {
    asm volatile("buffer_load_dwordx4 %0, %1, %2, %3 offen" : "=v"(d) : "v"(voff), "s"(rsrc), "s"(soff) : "memory");
}
__device__ __forceinline__ void st16_sys(const u32x4& d, unsigned voff, i32x4 rsrc, unsigned soff) {
    asm volatile("buffer_store_dwordx4 %0, %1, %2, %3 offen sc0 sc1\n\ts_nop 1"
                 :: "v"(d), "v"(voff), "s"(rsrc), "s"(soff) : "memory");
}
__device__ __forceinline__ void wait_vm(int n) {
    if (n >= 12) {
        if (n >= 20) { WAIT_V(20); } else if (n >= 16) { WAIT_V(16); } else { WAIT_V(12); }
    } else {
        if (n >= 8) { WAIT_V(8); } else if (n >= 4) { WAIT_V(4); } else { WAIT_V(0); }
    }
}

struct Frags { i32x8 x[4]; i32x8 w[4]; };
__device__ __forceinline__ void put4(i32x8& d, int i, i32x4 v) { d[4 * i] = v.x; d[4 * i + 1] = v.y; d[4 * i + 2] = v.z; d[4 * i + 3] = v.w; }
__device__ __forceinline__ void put2(i32x8& d, int i, i32x2 v) { d[2 * i] = v.x; d[2 * i + 1] = v.y; }

__device__ __forceinline__ void read_w1(unsigned aw, Frags& f, int nb, int q) {
    put2(f.w[nb], q, __builtin_amdgcn_ds_read_tr8_b64_v2i32((LAS i32x2*)(size_t)((aw ^ (unsigned)(nb << 4)) + ((q >> 1) * 64 + (q & 1) * 8) * BN)));
}
__device__ __forceinline__ void read_x1(unsigned ax, Frags& f, int mb, int q) {
    put4(f.x[mb], q, *(LAS i32x4*)(size_t)((q ? (ax ^ 64u) : ax) + mb * 2048));
}
__device__ __forceinline__ void read_nth(unsigned ax, unsigned aw, Frags& f, int r) {
    if (r < 4) read_w1(aw, f, 0, r);
    else if (r < 12) read_x1(ax, f, (r - 4) >> 1, (r - 4) & 1);
    else read_w1(aw, f, 1 + ((r - 12) >> 2), (r - 12) & 3);
}
__device__ __forceinline__ void mma1(f32x4 (&acc)[4][4], const Frags& f, int nb, int mb) {
    acc[nb][mb] = __builtin_amdgcn_mfma_scale_f32_16x16x128_f8f6f4(f.w[nb], f.x[mb], acc[nb][mb], 1, 1, 0, 0, 0, 0);
}
__device__ __forceinline__ unsigned next_x(unsigned soff) { return soff == (NXS - 1) * XB ? 0u : soff + XB; }
__device__ __forceinline__ unsigned next_w(unsigned soff) { return soff == (NWS - 1) * WB ? 0u : soff + WB; }

__device__ __forceinline__ f32x4 epi(f32x4 a, float scale, bool bad) {
    f32x4 v = a * scale;
    v.x = v.x < 0.f ? 0.f : v.x;
    v.y = v.y < 0.f ? 0.f : v.y;
    v.z = v.z < 0.f ? 0.f : v.z;
    v.w = v.w < 0.f ? 0.f : v.w;
    const float qnan = __builtin_nanf("");
    if (bad) v = (f32x4){qnan, qnan, qnan, qnan};
    return v;
}
#define MID_ST 0
__device__ __forceinline__ void st_out(float* dst, f32x4 v) {
    asm volatile("global_store_dwordx4 %0, %1, off\n\ts_nop 1" :: "v"(dst), "v"(v) : "memory");
}
__device__ __forceinline__ float* store_tile(f32x4 (&acc)[4][4], f32x4 (&pend)[8], unsigned park, float* out, int row0, int col0,
                                             float scale, bool bad) {
    int tz = threadIdx.x;
    asm volatile("" : "+v"(tz));
    const int ln = tz & 63, wv = tz >> 6;
    const int wm = wv & 1, wn = wv >> 1;
    float* o = out + (size_t)(row0 + wm * 64 + (ln & 15)) * NL + (col0 + wn * 64 + 4 * (ln >> 4));
#pragma unroll
    for (int mb = 0; mb < 4; ++mb)
#pragma unroll
        for (int nb = 0; nb < 4; ++nb) {
            const f32x4 v = epi(acc[nb][mb], scale, bad);
            if (mb == 0 || (mb == 1 && nb == 3)) st_out(o + (size_t)mb * 16 * NL + nb * 16, v);
            else if (mb == 1) *(LAS f32x4*)(size_t)(park + nb * 1024) = v;
            else pend[(mb - 2) * 4 + nb] = v;
            acc[nb][mb] = (f32x4){0.f, 0.f, 0.f, 0.f};
        }
    return o;
}

#define PRE_BAR 2
#define AGK_MODE 0
#define OUT_ST 2

#define TRACE_STEP(s)

__global__ void __launch_bounds__(512, 2) ag_gemm_fused(Params p) {
    extern __shared__ __attribute__((aligned(16))) char smem[];
    constexpr bool dbg_nowait = (AGK_MODE & 1) != 0, dbg_nopush = (AGK_MODE & 2) != 0;
    constexpr bool dbg_nodma = (AGK_MODE & 4) != 0, dbg_nolocal = (AGK_MODE & 16) != 0;
    if (AGK_MODE & 64) return;
    const int tid = threadIdx.x, lane = tid & 63;
    const int wid = __builtin_amdgcn_readfirstlane(tid >> 6);
    const int b = blockIdx.x;
    const int ct = b & 7, rt = b >> 3;
    const int rank = p.rank;
    uint8_t* wsl = p.ws[rank];
    unsigned* cntp = (unsigned*)(wsl + WS_CNT + (size_t)b * 128);

    if (rt < 4) {
        if (ct == NR - 1 || dbg_nopush) return;
        const int peer = (rank + 1 + ct) & (NR - 1);
        uint8_t* wsp = p.ws[peer];
        unsigned cnt_old;
        asm volatile("global_load_dword %0, %1, off sc1" : "=v"(cnt_old) : "v"(cntp) : "memory");
        const i32x4 rs = make_rsrc(p.x, SHARD_BYTES);
        const unsigned vs = (wid * 16 + (lane >> 3)) * KT + (lane & 7) * 16;
        const unsigned vd0 = (wid * 16 + (lane >> 3)) * BK + (((unsigned)lane & 7u) ^ ((unsigned)lane >> 4)) * 16;
        const unsigned vd1 = (wid * 16 + 8 + (lane >> 3)) * BK + (((unsigned)lane & 7u) ^ (4u + ((unsigned)lane >> 4))) * 16;
        const unsigned* ow = (const unsigned*)p.sched.ord[rt];
        u32x4 pb[4][2];
        auto push_load = [&](unsigned u, u32x4 (&d)[2]) {
            const unsigned so = (u >> 5) * (BM * KT) + (u & 31u) * BK;
            ld16_asm(d[0], vs, rs, so);
            ld16_asm(d[1], vs, rs, so + 8 * KT);
        };
        unsigned wc = ow[0];
        push_load(wc & 255u, pb[0]);
        push_load((wc >> 8) & 255u, pb[1]);
        push_load((wc >> 16) & 255u, pb[2]);
        push_load(wc >> 24, pb[3]);
        WAIT_V(8);
        asm volatile("" : "+v"(cnt_old));
        const unsigned call = (unsigned)__builtin_amdgcn_readfirstlane(cnt_old) + 1u;
        const unsigned pbase = call * 64u;
        const i32x4 rd = make_rsrc(wsp + WS_STAGE + ((size_t)(call & 1u) * NR + rank) * SHARD_BYTES, SHARD_BYTES);
        unsigned* flag = (unsigned*)(wsp + WS_PROG + (size_t)rank * 128) + rt;
        auto publish = [&](int done) {
            BAR();
            if (tid == 0) __hip_atomic_store(flag, pbase + (unsigned)done, __ATOMIC_RELAXED, __HIP_MEMORY_SCOPE_SYSTEM);
        };
        auto push_store = [&](unsigned u, u32x4 (&d)[2]) {
            st16_sys(d[0], vd0, rd, u * XB);
            st16_sys(d[1], vd1, rd, u * XB);
        };
#define PUSH_D 1
        {
            const unsigned wn = ow[1];
            WAIT_V(6); push_store(wc & 255u, pb[0]); push_load(wn & 255u, pb[0]);
            WAIT_V(8); push_store((wc >> 8) & 255u, pb[1]); push_load((wn >> 8) & 255u, pb[1]);
            WAIT_V(6); publish(1);
            WAIT_V(10); push_store((wc >> 16) & 255u, pb[2]); push_load((wn >> 16) & 255u, pb[2]);
            WAIT_V(6); publish(2);
            WAIT_V(12); push_store(wc >> 24, pb[3]); push_load(wn >> 24, pb[3]);
            WAIT_V(6); publish(3);
            wc = wn;
        }
#pragma unroll 1
        for (int t = 4; t < NSTEP - 4; t += 4) {
            const unsigned wn = ow[(t >> 2) + 1];
#pragma unroll
            for (int i = 0; i < 4; ++i) {
                push_store((wc >> (8 * i)) & 255u, pb[i]);
                push_load((wn >> (8 * i)) & 255u, pb[i]);
                WAIT_V(6); publish(t + i);
            }
            wc = wn;
        }
        push_store(wc & 255u, pb[0]);          WAIT_V(4); publish(28);
        push_store((wc >> 8) & 255u, pb[1]);   WAIT_V(2); publish(29);
        push_store((wc >> 16) & 255u, pb[2]);  WAIT_V(2); publish(30);
        push_store(wc >> 24, pb[3]);           WAIT_V(2); publish(31);
        WAIT_V(0); publish(32);
        if (tid == 0) __hip_atomic_store(cntp, call, __ATOMIC_RELAXED, __HIP_MEMORY_SCOPE_AGENT);
        return;
    }

    const int idx = rt - 4;
    const int sh = 1 + (idx >> 2), gg = (idx >> 1) & 1, half = idx & 1;
    const int src = (rank + sh) & (NR - 1);
    const bool loc = idx < 16 && !dbg_nolocal;
    const int lrow = idx * 32 & ~63;
    constexpr int NS = 2 * NSTEP;
    const int col0 = ct * 256 + half * BN;
    const unsigned lds0 = (unsigned)(uintptr_t)(LAS char*)smem;
    volatile LAS unsigned* ctrl = (volatile LAS unsigned*)(LAS char*)(smem + LDS_CTRL);

    const i32x4 rw = make_rsrc(p.w, W_BYTES);
    unsigned* prog = (unsigned*)(wsl + WS_PROG + (size_t)src * 128);
    unsigned* tmo = (unsigned*)(wsl + WS_TMO);
    const int row_last = src * ML + (gg + 2) * BM;
    float scale = 0.f;

    if (wid < 4) {
        scale = __builtin_bit_cast(float, __builtin_amdgcn_readfirstlane(__builtin_bit_cast(int, p.sx[0] * p.sw[0])));
        {
            const i32x4 rxp = make_rsrc(p.x, SHARD_BYTES);
            const unsigned vw = (unsigned)(tid >> 4) * NL + ct * 256 + (tid & 15) * 16;
            const unsigned vx = (unsigned)(tid >> 3) * KT + (tid & 7) * 16;
            u32x4 d0, d1, d2, d3;
            for (int ks = idx; ks < NSTEP; ks += 28) {
                const unsigned so = (unsigned)ks * (BK * NL), sx = (unsigned)ks * BK;
#pragma unroll
                for (int r = 0; r < 8; r += 4) {
                    ld16_asm(d0, vw, rw, so + (r + 0) * 16 * NL); ld16_asm(d1, vw, rw, so + (r + 1) * 16 * NL);
                    ld16_asm(d2, vw, rw, so + (r + 2) * 16 * NL); ld16_asm(d3, vw, rw, so + (r + 3) * 16 * NL);
                }
#pragma unroll
                for (int r = 0; r < 16; r += 4) {
                    ld16_asm(d0, vx, rxp, sx + (r + 0) * 32 * KT); ld16_asm(d1, vx, rxp, sx + (r + 1) * 32 * KT);
                    ld16_asm(d2, vx, rxp, sx + (r + 2) * 32 * KT); ld16_asm(d3, vx, rxp, sx + (r + 3) * 32 * KT);
                }
            }
            WAIT_V(0);
            asm volatile("" :: "v"(d0), "v"(d1), "v"(d2), "v"(d3));
        }
        const int wm = wid & 1, wn = wid >> 1;
        unsigned axl, awl, axs;
        {
            const unsigned m = lane & 15, gq = lane >> 4, q = m >> 1, h = m & 1;
            axl = lds0 + (wm * 64 + m) * BK + 16 * (gq ^ (m >> 1));
            awl = lds0 + LDS_W + (16 * gq + q) * BN + 16 * ((unsigned)(wn * 4) ^ ((q >> 1) | ((gq & 1) << 2))) + 8 * h;
            axs = lds0 + LDS_XL + (wm * 32 + m) * BK + 16 * (gq ^ (m >> 1));
        }
        f32x4 acc[4][4];
#pragma unroll
        for (int i = 0; i < 4; ++i)
#pragma unroll
            for (int j = 0; j < 4; ++j) acc[i][j] = (f32x4){0.f, 0.f, 0.f, 0.f};
        Frags fa, fb;
        f32x4 pend[8];
        float* pend_o;
        BAR();
        BAR();
        unsigned cx = 0, cw = 0;
        auto step = [&](int s, Frags& fc, Frags& fn) {
            cx = next_x(cx); cw = next_w(cw);
            const unsigned ax = axl + cx, aw = awl + cw;
            WAIT_L0();
            SCHED();
#pragma unroll
            for (int i = 0; i < PRE_BAR; ++i) { mma1(acc, fc, i >> 2, i & 3); SCHED(); }
            BAR();
            TRACE_STEP(s);
            SCHED();
#pragma unroll
            for (int i = PRE_BAR; i < 16; ++i) {
                mma1(acc, fc, i >> 2, i & 3);
                if (2 * (i - PRE_BAR) < 24) { read_nth(ax, aw, fn, 2 * (i - PRE_BAR)); read_nth(ax, aw, fn, 2 * (i - PRE_BAR) + 1); }
                SCHED();
            }
        };
        if (loc) {
            f32x4 accl[4][2];
#pragma unroll
            for (int i = 0; i < 4; ++i) accl[i][0] = accl[i][1] = (f32x4){0.f, 0.f, 0.f, 0.f};
            i32x8 xl[2];
            auto read_xl = [&](unsigned a, int mb, int q) {
                put4(xl[mb], q, *(LAS i32x4*)(size_t)((q ? (a ^ 64u) : a) + mb * 2048));
            };
#pragma unroll
            for (int r = 0; r < 24; ++r) read_nth(axl, awl, fa, r);
            read_xl(axs, 0, 0); read_xl(axs, 0, 1); read_xl(axs, 1, 0); read_xl(axs, 1, 1);
            auto cstep = [&](int s, Frags& fc, Frags& fn) {
                cx = next_x(cx); cw = next_w(cw);
                const unsigned ax = axl + cx, aw = awl + cw, al = axs + (cw >> 1);
                WAIT_L0();
                SCHED();
#pragma unroll
                for (int i = 0; i < PRE_BAR; ++i) { mma1(acc, fc, i >> 2, i & 3); SCHED(); }
                BAR();
                TRACE_STEP(s);
                SCHED();
#pragma unroll
                for (int g2 = 0; g2 < 8; ++g2) {
                    accl[g2 >> 1][g2 & 1] = __builtin_amdgcn_mfma_scale_f32_16x16x128_f8f6f4(fc.w[g2 >> 1], xl[g2 & 1], accl[g2 >> 1][g2 & 1], 1, 1, 0, 0, 0, 0);
                    read_nth(ax, aw, fn, 2 * g2); read_nth(ax, aw, fn, 2 * g2 + 1);
                    SCHED();
                }
#pragma unroll
                for (int i = PRE_BAR; i < 16; ++i) {
                    mma1(acc, fc, i >> 2, i & 3);
                    const int g2 = i - PRE_BAR;
                    if (g2 < 4) { read_nth(ax, aw, fn, 16 + 2 * g2); read_nth(ax, aw, fn, 17 + 2 * g2); }
                    else if (g2 < 6) { read_xl(al, g2 - 4, 0); read_xl(al, g2 - 4, 1); }
                    SCHED();
                }
            };
#pragma unroll 1
            for (int s = 0; s < NSTEP; s += 2) {
                cstep(s, fa, fb);
                cstep(s + 1, fb, fa);
            }
            int tz = threadIdx.x;
            asm volatile("" : "+v"(tz));
            const int ln = tz & 63;
            float* o = p.out + (size_t)(rank * ML + lrow + wm * 32 + (ln & 15)) * NL + (col0 + wn * 64 + 4 * (ln >> 4));
#pragma unroll
            for (int mb = 0; mb < 2; ++mb)
#pragma unroll
                for (int nb = 0; nb < 4; ++nb) st_out(o + (size_t)mb * 16 * NL + nb * 16, epi(accl[nb][mb], scale, false));
        } else {
#pragma unroll
            for (int r = 0; r < 24; ++r) read_nth(axl, awl, fa, r);
#pragma unroll 1
            for (int s = 0; s < NSTEP; s += 2) {
                step(s, fa, fb);
                step(s + 1, fb, fa);
            }
        }
        int tp = threadIdx.x;
        asm volatile("" : "+v"(tp));
        const unsigned park = lds0 + LDS_PARK + (tp >> 6) * 3072 + (tp & 63) * 16;
        {
            const bool bad = __builtin_amdgcn_readfirstlane((int)ctrl[4]) != 0;
            pend_o = store_tile(acc, pend, park, p.out, src * ML + gg * BM, col0, scale, bad);
        }
#pragma unroll 1
        for (int s = NSTEP; s < NS; s += 2) {
            step(s, fa, fb);
            const int k = s - NSTEP;
            if (k >= 2 && k <= 12) {
                float* po = pend_o + (size_t)32 * NL;
                if (k == 2) { st_out(po, pend[0]); st_out(po + 16, pend[1]); }
                else if (k == 4) { st_out(po + 32, pend[2]); st_out(po + 48, pend[3]); }
                else if (k == 6) { st_out(po + 16 * NL, pend[4]); st_out(po + 16 * NL + 16, pend[5]); }
                else if (k == 8) { st_out(po + 16 * NL + 32, pend[6]); st_out(po + 16 * NL + 48, pend[7]); }
                else {
                    if (k == 10) {
                        const f32x4 v0 = *(LAS f32x4*)(size_t)(park), v1 = *(LAS f32x4*)(size_t)(park + 1024);
                        st_out(pend_o + 16 * NL, v0);
                        st_out(pend_o + 16 * NL + 16, v1);
                    } else {
                        const f32x4 v2 = *(LAS f32x4*)(size_t)(park + 2048);
                        st_out(pend_o + 16 * NL + 32, v2);
                    }
                    WAIT_L0();
                }
            }
            step(s + 1, fb, fa);
        }
        const bool bad = __builtin_amdgcn_readfirstlane((int)ctrl[4]) != 0;
        int tq = threadIdx.x;
        asm volatile("" : "+v"(tq));
        const unsigned lq = tq & 63;
#pragma unroll
        for (int mb = 0; mb < 4; ++mb) {
            const unsigned row = wm * 64 + mb * 16 + (lq & 15);
#pragma unroll
            for (int nb = 0; nb < 4; ++nb) {
                const unsigned chunk = wn * 16 + nb * 4 + (lq >> 4);
                *(LAS f32x4*)(size_t)(lds0 + row * 512 + ((chunk ^ (row & 7u)) << 4)) = epi(acc[nb][mb], scale, bad);
            }
        }
    } else {
        unsigned vw[8];
        if (wid < 6) {
            const unsigned p8 = lane & 7, f0 = (unsigned)(lane >> 4) & 3u;
#pragma unroll
            for (int i = 0; i < 8; ++i) {
                const unsigned pi = (wid - 4) * 8 + i;
                vw[i] = (pi * 8 + (lane >> 3)) * NL + col0 + 16 * (p8 ^ (f0 | (((pi >> 1) & 1u) << 2))) - (i & 3) * 1024;
            }
        }
        const unsigned ldw = lds0 + LDS_W + (wid - 4) * 8192;
        const i32x4 rxl = make_rsrc(p.x + (size_t)lrow * KT, 64 * KT);
        unsigned vl[4];
        {
            const unsigned p8 = lane & 7, f0 = (unsigned)(lane >> 4);
#pragma unroll
            for (int i = 0; i < 4; ++i)
                vl[i] = (((wid & 1) * 4 + i) * 8 + (lane >> 3)) * KT + 16 * (p8 ^ ((f0 + 4u * (i & 1)) & 7u)) - i * 1024;
        }
        auto issue_w = [&](int j, unsigned slot) {
            if (dbg_nodma) return;
            const unsigned so = (unsigned)(j & 31) * (BK * NL);
            dma16x4(ldw + slot, vw[0], vw[1], vw[2], vw[3], rw, so);
            dma16x4(ldw + slot + 4096, vw[4], vw[5], vw[6], vw[7], rw, so);
            if (loc && j < NSTEP)
                dma16x4(lds0 + LDS_XL + (slot >> 1) + (wid & 1) * 4096, vl[0], vl[1], vl[2], vl[3], rxl, (unsigned)j * BK);
        };
        auto wait_w = [&](int s) {
            if (s + 2 < NS) { if (loc && s + 2 < NSTEP) { WAIT_V(12); } else { WAIT_V(8); } } else { WAIT_V(0); }
        };
        auto first_w = [&]() { issue_w(0, 0); issue_w(1, WB); issue_w(2, 2 * WB); };
        if (wid == 5) {
            first_w();
            BAR();
            WAIT_V(16);
            BAR();
            unsigned cur = 0;
#pragma unroll 1
            for (int s = 0; s < NS; ++s) {
                wait_w(s);
                BAR();
                if (s + NWS < NS) issue_w(s + NWS, cur);
                cur = next_w(cur);
            }
        } else {
            const int xw = wid & 1;
            unsigned cnt_old;
            asm volatile("global_load_dword %0, %1, off sc1" : "=v"(cnt_old) : "v"(cntp) : "memory");
            const unsigned posv = p.sched.pos[(gg + 2 * (lane >> 5)) * 32 + (lane & 31)];
            if (wid == 4) first_w();
            WAIT_V(0);
            asm volatile("" : "+v"(cnt_old));
            const unsigned call = (unsigned)__builtin_amdgcn_readfirstlane(cnt_old) + 1u;
            const unsigned pbase = call * 64u;
            const unsigned pj = posv & 3u;
            const unsigned need = pbase + (posv >> 2) + 1u;
            auto ready_count = [&](unsigned v) -> int {
                const unsigned long long m = __builtin_amdgcn_ballot_w64((int)(v - need) >= 0);
                return m == ~0ull ? NS : __builtin_ctzll(~m);
            };
            int R = dbg_nowait ? NS : 0;
            auto refresh = [&]() {
                const int r = ready_count(ctrl[pj]);
                if (r > R) R = r;
            };
            if (wid == 4) {
                __builtin_amdgcn_fence(__ATOMIC_ACQUIRE, "");
                if (lane < 8) ctrl[lane] = lane < 4 ? pbase : 0u;
                const i32x4 rf = make_rsrc(prog, 16);
                auto watch = [&](int st) {
                    if (R < NS) refresh();
                    if (R > st || st >= NS) return;
                    unsigned long long t0 = __builtin_amdgcn_s_memrealtime();
                    for (unsigned it = 1;; ++it) {
                        const unsigned v = __hip_atomic_load(prog + pj, __ATOMIC_RELAXED, __HIP_MEMORY_SCOPE_SYSTEM);
                        const int r = ready_count(v);
                        if (r > R) { R = r; ctrl[pj] = v; }
                        if (R > st) break;
                        if ((it & 15u) == 0u) {
                            const unsigned tv = (unsigned)__builtin_amdgcn_readfirstlane(
                                __hip_atomic_load(tmo, __ATOMIC_RELAXED, __HIP_MEMORY_SCOPE_AGENT));
                            if (tv == call || __builtin_amdgcn_s_memrealtime() - t0 > DEADLINE_TICKS) {
                                if (lane == 0) __hip_atomic_store(tmo, call, __ATOMIC_RELAXED, __HIP_MEMORY_SCOPE_AGENT);
                                if (lane == 0) ctrl[4] = 1u;
                                if (lane < 4) ctrl[lane] = pbase + 63u;
                                R = NS;
                                break;
                            }
                        }
                        __builtin_amdgcn_s_sleep(1);
                    }
                };
                WAIT_V(0);
                WAIT_L0();
                BAR();
                unsigned cur = 0;
#pragma unroll 1
                for (int s = -1; s < NS; ++s) {
                    watch(s + 1);
                    if (s >= 0) wait_w(s);
                    WAIT_L0();
                    BAR();
                    if (R < NS) { if (lane < 4) dma4_sys(lds0 + LDS_CTRL, (unsigned)lane * 4u, rf); }
                    if (s >= 0) {
                        if (s + NWS < NS) issue_w(s + NWS, cur);
                        cur = next_w(cur);
                    }
                }
                if (lane == 0) __hip_atomic_store(cntp, call, __ATOMIC_RELAXED, __HIP_MEMORY_SCOPE_AGENT);
            } else {
                i32x4 rxr = make_rsrc(wsl + WS_STAGE + ((size_t)(call & 1u) * NR + src) * SHARD_BYTES, SHARD_BYTES);
                if (AGK_MODE & 128) rxr.z = 0;
                const unsigned vr = xw * 8192 + lane * 16;
                const unsigned ldx = lds0 + xw * 8192;
                int nix = 0;
                unsigned isl = 0;
                auto issue_x = [&]() {
                    if (!dbg_nodma) {
                        const unsigned so = (unsigned)((gg + 2 * (nix >> 5)) * 32 + (nix & 31)) * XB;
                        dma16x4(ldx + isl, vr, vr, vr, vr, rxr, so);
                        dma16x4(ldx + isl + 4096, vr, vr, vr, vr, rxr, so + 4096);
                    }
                    isl = next_x(isl);
                    ++nix;
                };
                BAR();
#pragma unroll 1
                for (int s = -1; s < NS; ++s) {
                    const int st = s + 1;
                    if (st < NS) {
                        if (R < NS) refresh();
                        while (R <= st) { __builtin_amdgcn_s_sleep(1); refresh(); }
                        const int lim = s + NXS < NS ? s + NXS : NS;
                        const int l = R < lim ? R : lim;
                        while (nix < l) issue_x();
                        const int ahead = nix - 1 - st;
                        if (ahead >= 3) { WAIT_V(24); } else if (ahead == 2) { WAIT_V(16); } else if (ahead == 1) { WAIT_V(8); } else { WAIT_V(0); }
                    }
                    WAIT_L0();
                    BAR();
                    if (nix < R && nix < NS && nix <= s + NXS) issue_x();
                }
            }
        }
    }
    WAIT_L0();
    BAR();
    {
        int tz = threadIdx.x;
        asm volatile("" : "+v"(tz));
        const int ln = tz & 63, wv = tz >> 6;
        float* o = p.out + (size_t)(row_last + wv * 16 + (ln >> 5)) * NL + col0 + (ln & 31) * 4;
        f32x4 v[8];
#pragma unroll
        for (int i = 0; i < 8; ++i) {
            const unsigned row = wv * 16 + 2 * i + (ln >> 5);
            v[i] = *(LAS f32x4*)(size_t)(lds0 + row * 512 + (((unsigned)(ln & 31) ^ (row & 7u)) << 4));
        }
#pragma unroll
        for (int i = 0; i < 8; ++i) {
            float* dst = o + (size_t)i * 2 * NL;
            asm volatile("global_store_dwordx4 %0, %1, off sc1\n\ts_nop 1" :: "v"(dst), "v"(v[i]) : "memory");
        }
    }
}

}

extern "C" void kernel_launch(int rank, int nranks, ncclComm_t comm,
                              hipStream_t stream,
                              void* const* d_in, const int* in_sizes, int n_in,
                              void* d_out, int out_size,
                              const int* in_dtypes, int out_dtype,
                              void* const* d_out_all,
                              void* const* d_ws_all, size_t ws_bytes) {
    using namespace agk;
    (void)comm; (void)in_sizes; (void)n_in; (void)out_size; (void)in_dtypes; (void)out_dtype; (void)d_out_all;
    if (nranks != NR || ws_bytes < WS_NEEDED || rank < 0 || rank >= NR) {
        fprintf(stderr, "unexpected configuration\n");
        return;
    }
    static bool attr_set[NR] = {};
    if (!attr_set[rank]) {
        HIP_CHECK(hipFuncSetAttribute((const void*)ag_gemm_fused, hipFuncAttributeMaxDynamicSharedMemorySize, LDS_BYTES));
        attr_set[rank] = true;
    }
    static const Sched sched = make_sched();
    Params p;
    p.x = (const uint8_t*)d_in[0];
    p.w = (const uint8_t*)d_in[1];
    p.sx = (const float*)d_in[2];
    p.sw = (const float*)d_in[3];
    p.out = (float*)d_out;
    for (int i = 0; i < NR; ++i) p.ws[i] = (uint8_t*)d_ws_all[i];
    p.rank = rank;
    p.pad = 0;
    p.sched = sched;
    hipLaunchKernelGGL(ag_gemm_fused, dim3(NWG), dim3(512), LDS_BYTES, stream, p);
    HIP_CHECK(hipGetLastError());
}
```

```cpp
#include <hip/hip_runtime.h>
#include <rccl/rccl.h>
#include <cstdio>
#include <cstdint>

#define HIP_CHECK(expr) do { hipError_t _e = (expr); if (_e != hipSuccess) { \
    fprintf(stderr, "HIP error %s at %s:%d\n", hipGetErrorString(_e), __FILE__, __LINE__); \
    return; } } while (0)

namespace agk {

#define LAS __attribute__((address_space(3)))
typedef int i32x8 __attribute__((ext_vector_type(8)));
typedef int i32x4 __attribute__((ext_vector_type(4)));
typedef int i32x2 __attribute__((ext_vector_type(2)));
typedef unsigned u32x4 __attribute__((ext_vector_type(4)));
typedef float f32x4 __attribute__((ext_vector_type(4)));

constexpr int NR = 8;
constexpr int KT = 4096, NL = 2048, ML = 512;
constexpr int BM = 128, BN = 128, BK = 128;
constexpr int NSTEP = KT / BK;
constexpr int XB = BM * BK;
constexpr int WB = BK * BN;
constexpr int NXS = 4, NWS = 3;
constexpr int XLB = 64 * BK;
constexpr int LDS_W = NXS * XB;
constexpr int LDS_XL = LDS_W + NWS * WB;
constexpr int LDS_CTRL = LDS_XL + NWS * XLB;
constexpr int LDS_PARK = LDS_CTRL + 2048;
constexpr int LDS_BYTES = LDS_PARK + 12288;
static_assert(LDS_BYTES <= 163840, "LDS");
constexpr int NWG = 256;
constexpr unsigned SHARD_BYTES = ML * KT;
constexpr unsigned W_BYTES = KT * NL;

constexpr size_t WS_CNT = 0;
constexpr size_t WS_TMO = WS_CNT + NWG * 128;
constexpr size_t WS_PROG = WS_TMO + 128;
constexpr size_t WS_DBG = 512u << 10;
constexpr size_t WS_STAGE = 1u << 20;
constexpr size_t WS_NEEDED = WS_STAGE + 2ull * NR * SHARD_BYTES;
constexpr unsigned long long DEADLINE_TICKS = 20000000ull;

#define SK_START {0, 8, 1024, 1032}
#define SK_RATE {16, 16, 16, 16}
struct Sched {
    unsigned char ord[4][32];
    unsigned char pos[128];
};
constexpr Sched make_sched() {
    Sched s{};
    const int start[4] = SK_START;
    const int rate[4] = SK_RATE;
    int nxt[4] = {0, 0, 0, 0};
    for (int p = 0; p < 128; ++p) {
        int best = -1, bk = 0;
        for (int g = 0; g < 4; ++g) {
            if (nxt[g] >= 32) continue;
            const int key = start[g] + nxt[g] * rate[g];
            if (best < 0 || key < bk) { best = g; bk = key; }
        }
        s.ord[p & 3][p >> 2] = (unsigned char)(best * 32 + nxt[best]);
        s.pos[best * 32 + nxt[best]] = (unsigned char)p;
        ++nxt[best];
    }
    return s;
}

struct Params {
    const uint8_t* x;
    const uint8_t* w;
    const float* sx;
    const float* sw;
    float* out;
    uint8_t* ws[NR];
    int rank;
    int pad;
    Sched sched;
};

#define WAIT_V(n) asm volatile("s_waitcnt vmcnt(" #n ")" ::: "memory")
#define WAIT_L0() do { asm volatile("" ::: "memory"); __builtin_amdgcn_s_waitcnt(0xC07F); asm volatile("" ::: "memory"); } while (0)
#define BAR() do { asm volatile("" ::: "memory"); __builtin_amdgcn_s_barrier(); asm volatile("" ::: "memory"); } while (0)
#define SCHED() __builtin_amdgcn_sched_barrier(0)

__device__ __forceinline__ i32x4 make_rsrc(const void* p, unsigned bytes) {
    const unsigned long long a = (unsigned long long)p;
    i32x4 r;
    r.x = __builtin_amdgcn_readfirstlane((int)(unsigned)a);
    r.y = __builtin_amdgcn_readfirstlane((int)((unsigned)(a >> 32) & 0xffffu));
    r.z = __builtin_amdgcn_readfirstlane((int)bytes);
    r.w = 0x00020000;
    return r;
}
__device__ __forceinline__ void dma16(unsigned lds, unsigned voff, i32x4 rsrc, unsigned soff) {
    asm volatile("s_mov_b32 m0, %0\n\ts_nop 0\n\tbuffer_load_dwordx4 %1, %2, %3 offen lds"
                 :: "s"(lds), "v"(voff), "s"(rsrc), "s"(soff) : "memory");
}
__device__ __forceinline__ void dma16x4(unsigned lds, unsigned v0, unsigned v1, unsigned v2, unsigned v3, i32x4 rsrc, unsigned soff) {
    asm volatile("s_mov_b32 m0, %0\n\ts_nop 0\n\t"
                 "buffer_load_dwordx4 %1, %5, %6 offen lds\n\t"
                 "buffer_load_dwordx4 %2, %5, %6 offen offset:1024 lds\n\t"
                 "buffer_load_dwordx4 %3, %5, %6 offen offset:2048 lds\n\t"
                 "buffer_load_dwordx4 %4, %5, %6 offen offset:3072 lds"
                 :: "s"(lds), "v"(v0), "v"(v1), "v"(v2), "v"(v3), "s"(rsrc), "s"(soff) : "memory");
}
#define X_POL ""
__device__ __forceinline__ void dma16x4_x(unsigned lds, unsigned v0, i32x4 rsrc, unsigned soff) {
    asm volatile("s_mov_b32 m0, %0\n\ts_nop 0\n\t"
                 "buffer_load_dwordx4 %1, %2, %3 offen " X_POL " lds\n\t"
                 "buffer_load_dwordx4 %1, %2, %3 offen offset:1024 " X_POL " lds\n\t"
                 "buffer_load_dwordx4 %1, %2, %3 offen offset:2048 " X_POL " lds\n\t"
                 "buffer_load_dwordx4 %1, %2, %3 offen offset:3072 " X_POL " lds"
                 :: "s"(lds), "v"(v0), "s"(rsrc), "s"(soff) : "memory");
}
__device__ __forceinline__ void dma4_sys(unsigned lds, unsigned voff, i32x4 rsrc) {
    asm volatile("s_mov_b32 m0, %0\n\ts_nop 0\n\tbuffer_load_dword %1, %2, 0 offen sc0 sc1 lds"
                 :: "s"(lds), "v"(voff), "s"(rsrc) : "memory");
}
__device__ __forceinline__ void ld16_asm(u32x4& d, unsigned voff, i32x4 rsrc, unsigned soff) {
    asm volatile("buffer_load_dwordx4 %0, %1, %2, %3 offen" : "=v"(d) : "v"(voff), "s"(rsrc), "s"(soff) : "memory");
}
__device__ __forceinline__ void st16_sys(const u32x4& d, unsigned voff, i32x4 rsrc, unsigned soff) {
    asm volatile("buffer_store_dwordx4 %0, %1, %2, %3 offen sc0 sc1\n\ts_nop 1"
                 :: "v"(d), "v"(voff), "s"(rsrc), "s"(soff) : "memory");
}
__device__ __forceinline__ void wait_vm(int n) {
    if (n >= 12) {
        if (n >= 20) { WAIT_V(20); } else if (n >= 16) { WAIT_V(16); } else { WAIT_V(12); }
    } else {
        if (n >= 8) { WAIT_V(8); } else if (n >= 4) { WAIT_V(4); } else { WAIT_V(0); }
    }
}

struct Frags { i32x8 x[4]; i32x8 w[4]; };
__device__ __forceinline__ void put4(i32x8& d, int i, i32x4 v) { d[4 * i] = v.x; d[4 * i + 1] = v.y; d[4 * i + 2] = v.z; d[4 * i + 3] = v.w; }
__device__ __forceinline__ void put2(i32x8& d, int i, i32x2 v) { d[2 * i] = v.x; d[2 * i + 1] = v.y; }

__device__ __forceinline__ void read_w1(unsigned aw, Frags& f, int nb, int q) {
    put2(f.w[nb], q, __builtin_amdgcn_ds_read_tr8_b64_v2i32((LAS i32x2*)(size_t)((aw ^ (unsigned)(nb << 4)) + ((q >> 1) * 64 + (q & 1) * 8) * BN)));
}
__device__ __forceinline__ void read_x1(unsigned ax, Frags& f, int mb, int q) {
    put4(f.x[mb], q, *(LAS i32x4*)(size_t)((q ? (ax ^ 64u) : ax) + mb * 2048));
}
__device__ __forceinline__ void read_nth(unsigned ax, unsigned aw, Frags& f, int r) {
    if (r < 4) read_w1(aw, f, 0, r);
    else if (r < 12) read_x1(ax, f, (r - 4) >> 1, (r - 4) & 1);
    else read_w1(aw, f, 1 + ((r - 12) >> 2), (r - 12) & 3);
}
__device__ __forceinline__ void mma1(f32x4 (&acc)[4][4], const Frags& f, int nb, int mb) {
    acc[nb][mb] = __builtin_amdgcn_mfma_scale_f32_16x16x128_f8f6f4(f.w[nb], f.x[mb], acc[nb][mb], 1, 1, 0, 0, 0, 0);
}
__device__ __forceinline__ unsigned next_x(unsigned soff) { return soff == (NXS - 1) * XB ? 0u : soff + XB; }
__device__ __forceinline__ unsigned next_w(unsigned soff) { return soff == (NWS - 1) * WB ? 0u : soff + WB; }

__device__ __forceinline__ f32x4 epi(f32x4 a, float scale, bool bad) {
    const float sc = bad ? __builtin_nanf("") : scale;
    return __builtin_elementwise_maximum(a * sc, (f32x4){0.f, 0.f, 0.f, 0.f});
}
#define MID_ST 0
__device__ __forceinline__ void st_out(float* dst, f32x4 v) {
    asm volatile("global_store_dwordx4 %0, %1, off\n\ts_nop 1" :: "v"(dst), "v"(v) : "memory");
}
__device__ __forceinline__ float* store_tile(f32x4 (&acc)[4][4], f32x4 (&pend)[8], unsigned park, float* out, int row0, int col0,
                                             float scale, bool bad) {
    int tz = threadIdx.x;
    asm volatile("" : "+v"(tz));
    const int ln = tz & 63, wv = tz >> 6;
    const int wm = wv & 1, wn = wv >> 1;
    float* o = out + (size_t)(row0 + wm * 64 + (ln & 15)) * NL + (col0 + wn * 64 + 4 * (ln >> 4));
#pragma unroll
    for (int mb = 0; mb < 4; ++mb)
#pragma unroll
        for (int nb = 0; nb < 4; ++nb) {
            const f32x4 v = epi(acc[nb][mb], scale, bad);
            if (mb == 0 || (mb == 1 && nb == 3)) st_out(o + (size_t)mb * 16 * NL + nb * 16, v);
            else if (mb == 1) *(LAS f32x4*)(size_t)(park + nb * 1024) = v;
            else pend[(mb - 2) * 4 + nb] = v;
            acc[nb][mb] = (f32x4){0.f, 0.f, 0.f, 0.f};
        }
    return o;
}

#define PRE_BAR 2
#define AGK_MODE 0
#define OUT_ST 2

#define TRACE_STEP(s)

__global__ void __launch_bounds__(512, 2) ag_gemm_fused(Params p) {
    extern __shared__ __attribute__((aligned(16))) char smem[];
    constexpr bool dbg_nowait = (AGK_MODE & 1) != 0, dbg_nopush = (AGK_MODE & 2) != 0;
    constexpr bool dbg_nodma = (AGK_MODE & 4) != 0, dbg_nolocal = (AGK_MODE & 16) != 0;
    if (AGK_MODE & 64) return;
    const int tid = threadIdx.x, lane = tid & 63;
    const int wid = __builtin_amdgcn_readfirstlane(tid >> 6);
    const int b = blockIdx.x;
    const int ct = b & 7, rt = b >> 3;
    const int rank = p.rank;
    uint8_t* wsl = p.ws[rank];
    unsigned* cntp = (unsigned*)(wsl + WS_CNT + (size_t)b * 128);

    if (rt < 4) {
        if (ct == NR - 1 || dbg_nopush) return;
        const int peer = (rank + 1 + ct) & (NR - 1);
        uint8_t* wsp = p.ws[peer];
        unsigned cnt_old;
        asm volatile("global_load_dword %0, %1, off sc1" : "=v"(cnt_old) : "v"(cntp) : "memory");
        const i32x4 rs = make_rsrc(p.x, SHARD_BYTES);
        const unsigned vs = (wid * 16 + (lane >> 3)) * KT + (lane & 7) * 16;
        const unsigned vd0 = (wid * 16 + (lane >> 3)) * BK + (((unsigned)lane & 7u) ^ ((unsigned)lane >> 4)) * 16;
        const unsigned vd1 = (wid * 16 + 8 + (lane >> 3)) * BK + (((unsigned)lane & 7u) ^ (4u + ((unsigned)lane >> 4))) * 16;
        const unsigned* ow = (const unsigned*)p.sched.ord[rt];
        u32x4 pb[4][2];
        auto push_load = [&](unsigned u, u32x4 (&d)[2]) {
            const unsigned so = (u >> 5) * (BM * KT) + (u & 31u) * BK;
            ld16_asm(d[0], vs, rs, so);
            ld16_asm(d[1], vs, rs, so + 8 * KT);
        };
        unsigned wc = ow[0];
        push_load(wc & 255u, pb[0]);
        push_load((wc >> 8) & 255u, pb[1]);
        push_load((wc >> 16) & 255u, pb[2]);
        push_load(wc >> 24, pb[3]);
        WAIT_V(8);
        asm volatile("" : "+v"(cnt_old));
        const unsigned call = (unsigned)__builtin_amdgcn_readfirstlane(cnt_old) + 1u;
        const unsigned pbase = call * 64u;
        const i32x4 rd = make_rsrc(wsp + WS_STAGE + ((size_t)(call & 1u) * NR + rank) * SHARD_BYTES, SHARD_BYTES);
        unsigned* flag = (unsigned*)(wsp + WS_PROG + (size_t)rank * 128) + rt;
        auto publish = [&](int done) {
            BAR();
            if (tid == 0) __hip_atomic_store(flag, pbase + (unsigned)done, __ATOMIC_RELAXED, __HIP_MEMORY_SCOPE_SYSTEM);
        };
        auto push_store = [&](unsigned u, u32x4 (&d)[2]) {
            st16_sys(d[0], vd0, rd, u * XB);
            st16_sys(d[1], vd1, rd, u * XB);
        };
#define PUSH_D 1
        {
            const unsigned wn = ow[1];
            WAIT_V(6); push_store(wc & 255u, pb[0]); push_load(wn & 255u, pb[0]);
            WAIT_V(8); push_store((wc >> 8) & 255u, pb[1]); push_load((wn >> 8) & 255u, pb[1]);
            WAIT_V(6); publish(1);
            WAIT_V(10); push_store((wc >> 16) & 255u, pb[2]); push_load((wn >> 16) & 255u, pb[2]);
            WAIT_V(6); publish(2);
            WAIT_V(12); push_store(wc >> 24, pb[3]); push_load(wn >> 24, pb[3]);
            WAIT_V(6); publish(3);
            wc = wn;
        }
#pragma unroll 1
        for (int t = 4; t < NSTEP - 4; t += 4) {
            const unsigned wn = ow[(t >> 2) + 1];
#pragma unroll
            for (int i = 0; i < 4; ++i) {
                push_store((wc >> (8 * i)) & 255u, pb[i]);
                push_load((wn >> (8 * i)) & 255u, pb[i]);
                WAIT_V(6); publish(t + i);
            }
            wc = wn;
        }
        push_store(wc & 255u, pb[0]);          WAIT_V(4); publish(28);
        push_store((wc >> 8) & 255u, pb[1]);   WAIT_V(2); publish(29);
        push_store((wc >> 16) & 255u, pb[2]);  WAIT_V(2); publish(30);
        push_store(wc >> 24, pb[3]);           WAIT_V(2); publish(31);
        WAIT_V(0); publish(32);
        if (tid == 0) __hip_atomic_store(cntp, call, __ATOMIC_RELAXED, __HIP_MEMORY_SCOPE_AGENT);
        return;
    }

    const int idx = rt - 4;
    const int sh = 1 + (idx >> 2), gg = (idx >> 1) & 1, half = idx & 1;
    const int src = (rank + sh) & (NR - 1);
    const bool loc = idx < 16 && !dbg_nolocal;
    const int lrow = idx * 32 & ~63;
    constexpr int NS = 2 * NSTEP;
    const int col0 = ct * 256 + half * BN;
    const unsigned lds0 = (unsigned)(uintptr_t)(LAS char*)smem;
    volatile LAS unsigned* ctrl = (volatile LAS unsigned*)(LAS char*)(smem + LDS_CTRL);

    const i32x4 rw = make_rsrc(p.w, W_BYTES);
    unsigned* prog = (unsigned*)(wsl + WS_PROG + (size_t)src * 128);
    unsigned* tmo = (unsigned*)(wsl + WS_TMO);
    const int row_last = src * ML + (gg + 2) * BM;
    float scale = 0.f;

    if (wid < 4) {
        scale = __builtin_bit_cast(float, __builtin_amdgcn_readfirstlane(__builtin_bit_cast(int, p.sx[0] * p.sw[0])));
        {
            const i32x4 rxp = make_rsrc(p.x, SHARD_BYTES);
            const unsigned vw = (unsigned)(tid >> 4) * NL + ct * 256 + (tid & 15) * 16;
            const unsigned vx = (unsigned)(tid >> 3) * KT + (tid & 7) * 16;
            u32x4 d0, d1, d2, d3;
            for (int ks = idx; ks < NSTEP; ks += 28) {
                const unsigned so = (unsigned)ks * (BK * NL), sx = (unsigned)ks * BK;
#pragma unroll
                for (int r = 0; r < 8; r += 4) {
                    ld16_asm(d0, vw, rw, so + (r + 0) * 16 * NL); ld16_asm(d1, vw, rw, so + (r + 1) * 16 * NL);
                    ld16_asm(d2, vw, rw, so + (r + 2) * 16 * NL); ld16_asm(d3, vw, rw, so + (r + 3) * 16 * NL);
                }
#pragma unroll
                for (int r = 0; r < 16; r += 4) {
                    ld16_asm(d0, vx, rxp, sx + (r + 0) * 32 * KT); ld16_asm(d1, vx, rxp, sx + (r + 1) * 32 * KT);
                    ld16_asm(d2, vx, rxp, sx + (r + 2) * 32 * KT); ld16_asm(d3, vx, rxp, sx + (r + 3) * 32 * KT);
                }
            }
            WAIT_V(0);
            asm volatile("" :: "v"(d0), "v"(d1), "v"(d2), "v"(d3));
        }
        const int wm = wid & 1, wn = wid >> 1;
        unsigned axl, awl, axs;
        {
            const unsigned m = lane & 15, gq = lane >> 4, q = m >> 1, h = m & 1;
            axl = lds0 + (wm * 64 + m) * BK + 16 * (gq ^ (m >> 1));
            awl = lds0 + LDS_W + (16 * gq + q) * BN + 16 * ((unsigned)(wn * 4) ^ ((q >> 1) | ((gq & 1) << 2))) + 8 * h;
            axs = lds0 + LDS_XL + (wm * 32 + m) * BK + 16 * (gq ^ (m >> 1));
        }
        f32x4 acc[4][4];
#pragma unroll
        for (int i = 0; i < 4; ++i)
#pragma unroll
            for (int j = 0; j < 4; ++j) acc[i][j] = (f32x4){0.f, 0.f, 0.f, 0.f};
        Frags fa, fb;
        f32x4 pend[8];
        float* pend_o;
        BAR();
        BAR();
        unsigned cx = 0, cw = 0;
        auto step = [&](int s, Frags& fc, Frags& fn) {
            cx = next_x(cx); cw = next_w(cw);
            const unsigned ax = axl + cx, aw = awl + cw;
            WAIT_L0();
            SCHED();
#pragma unroll
            for (int i = 0; i < PRE_BAR; ++i) { mma1(acc, fc, i >> 2, i & 3); SCHED(); }
            BAR();
            TRACE_STEP(s);
            SCHED();
#pragma unroll
            for (int i = PRE_BAR; i < 16; ++i) {
                mma1(acc, fc, i >> 2, i & 3);
                if (2 * (i - PRE_BAR) < 24) { read_nth(ax, aw, fn, 2 * (i - PRE_BAR)); read_nth(ax, aw, fn, 2 * (i - PRE_BAR) + 1); }
                SCHED();
            }
        };
        if (loc) {
            f32x4 accl[4][2];
#pragma unroll
            for (int i = 0; i < 4; ++i) accl[i][0] = accl[i][1] = (f32x4){0.f, 0.f, 0.f, 0.f};
            i32x8 xl[2];
            auto read_xl = [&](unsigned a, int mb, int q) {
                put4(xl[mb], q, *(LAS i32x4*)(size_t)((q ? (a ^ 64u) : a) + mb * 2048));
            };
#pragma unroll
            for (int r = 0; r < 24; ++r) read_nth(axl, awl, fa, r);
            read_xl(axs, 0, 0); read_xl(axs, 0, 1); read_xl(axs, 1, 0); read_xl(axs, 1, 1);
            auto cstep = [&](int s, Frags& fc, Frags& fn) {
                cx = next_x(cx); cw = next_w(cw);
                const unsigned ax = axl + cx, aw = awl + cw, al = axs + (cw >> 1);
                WAIT_L0();
                SCHED();
#pragma unroll
                for (int i = 0; i < PRE_BAR; ++i) { mma1(acc, fc, i >> 2, i & 3); SCHED(); }
                BAR();
                TRACE_STEP(s);
                SCHED();
#pragma unroll
                for (int g2 = 0; g2 < 8; ++g2) {
                    accl[g2 >> 1][g2 & 1] = __builtin_amdgcn_mfma_scale_f32_16x16x128_f8f6f4(fc.w[g2 >> 1], xl[g2 & 1], accl[g2 >> 1][g2 & 1], 1, 1, 0, 0, 0, 0);
                    read_nth(ax, aw, fn, 2 * g2); read_nth(ax, aw, fn, 2 * g2 + 1);
                    SCHED();
                }
#pragma unroll
                for (int i = PRE_BAR; i < 16; ++i) {
                    mma1(acc, fc, i >> 2, i & 3);
                    const int g2 = i - PRE_BAR;
                    if (g2 < 4) { read_nth(ax, aw, fn, 16 + 2 * g2); read_nth(ax, aw, fn, 17 + 2 * g2); }
                    else if (g2 < 6) { read_xl(al, g2 - 4, 0); read_xl(al, g2 - 4, 1); }
                    SCHED();
                }
            };
#pragma unroll 1
            for (int s = 0; s < NSTEP; s += 2) {
                cstep(s, fa, fb);
                cstep(s + 1, fb, fa);
            }
            int tz = threadIdx.x;
            asm volatile("" : "+v"(tz));
            const int ln = tz & 63;
            float* o = p.out + (size_t)(rank * ML + lrow + wm * 32 + (ln & 15)) * NL + (col0 + wn * 64 + 4 * (ln >> 4));
            const unsigned pk = lds0 + LDS_XL + (tz >> 6) * 6144 + ln * 16;
#pragma unroll
            for (int i = 0; i < 8; ++i) {
                const f32x4 v = epi(accl[i & 3][i >> 2], scale, false);
                if (i < 2) st_out(o + i * 16, v);
                else *(LAS f32x4*)(size_t)(pk + (i - 2) * 1024) = v;
            }
        } else {
#pragma unroll
            for (int r = 0; r < 24; ++r) read_nth(axl, awl, fa, r);
#pragma unroll 1
            for (int s = 0; s < NSTEP; s += 2) {
                step(s, fa, fb);
                step(s + 1, fb, fa);
            }
        }
        int tp = threadIdx.x;
        asm volatile("" : "+v"(tp));
        const unsigned park = lds0 + LDS_PARK + (tp >> 6) * 3072 + (tp & 63) * 16;
        {
            const bool bad = __builtin_amdgcn_readfirstlane((int)ctrl[4]) != 0;
            pend_o = store_tile(acc, pend, park, p.out, src * ML + gg * BM, col0, scale, bad);
        }
#pragma unroll 1
        for (int s = NSTEP; s < NS; s += 2) {
            step(s, fa, fb);
            const int k = s - NSTEP;
            if (k >= 2 && k <= 12) {
                float* po = pend_o + (size_t)32 * NL;
                if (k == 2) { st_out(po, pend[0]); st_out(po + 16, pend[1]); }
                else if (k == 4) { st_out(po + 32, pend[2]); st_out(po + 48, pend[3]); }
                else if (k == 6) { st_out(po + 16 * NL, pend[4]); st_out(po + 16 * NL + 16, pend[5]); }
                else if (k == 8) { st_out(po + 16 * NL + 32, pend[6]); st_out(po + 16 * NL + 48, pend[7]); }
                else {
                    if (k == 10) {
                        const f32x4 v0 = *(LAS f32x4*)(size_t)(park), v1 = *(LAS f32x4*)(size_t)(park + 1024);
                        st_out(pend_o + 16 * NL, v0);
                        st_out(pend_o + 16 * NL + 16, v1);
                    } else {
                        const f32x4 v2 = *(LAS f32x4*)(size_t)(park + 2048);
                        st_out(pend_o + 16 * NL + 32, v2);
                    }
                    WAIT_L0();
                }
            } else if (loc && k >= 14 && k <= 18) {
                int tz = threadIdx.x;
                asm volatile("" : "+v"(tz));
                const int ln = tz & 63, i = k - 12;
                float* o = p.out + (size_t)(rank * ML + lrow + wm * 32 + (i >> 2) * 16 + (ln & 15)) * NL
                                 + (col0 + wn * 64 + (i & 3) * 16 + 4 * (ln >> 4));
                const unsigned pk = lds0 + LDS_XL + (tz >> 6) * 6144 + ln * 16 + (i - 2) * 1024;
                const f32x4 v0 = *(LAS f32x4*)(size_t)(pk), v1 = *(LAS f32x4*)(size_t)(pk + 1024);
                st_out(o, v0);
                st_out(o + 16, v1);
                WAIT_L0();
            }
            step(s + 1, fb, fa);
        }
        const bool bad = __builtin_amdgcn_readfirstlane((int)ctrl[4]) != 0;
        int tq = threadIdx.x;
        asm volatile("" : "+v"(tq));
        const unsigned lq = tq & 63;
#pragma unroll
        for (int mb = 0; mb < 4; ++mb) {
            const unsigned row = wm * 64 + mb * 16 + (lq & 15);
#pragma unroll
            for (int nb = 0; nb < 4; ++nb) {
                const unsigned chunk = wn * 16 + nb * 4 + (lq >> 4);
                *(LAS f32x4*)(size_t)(lds0 + row * 512 + ((chunk ^ (row & 7u)) << 4)) = epi(acc[nb][mb], scale, bad);
            }
        }
    } else {
        unsigned vw[8];
        if (wid < 6) {
            const unsigned p8 = lane & 7, f0 = (unsigned)(lane >> 4) & 3u;
#pragma unroll
            for (int i = 0; i < 8; ++i) {
                const unsigned pi = (wid - 4) * 8 + i;
                vw[i] = (pi * 8 + (lane >> 3)) * NL + col0 + 16 * (p8 ^ (f0 | (((pi >> 1) & 1u) << 2))) - (i & 3) * 1024;
            }
        }
        const unsigned ldw = lds0 + LDS_W + (wid - 4) * 8192;
        const i32x4 rxl = make_rsrc(p.x + (size_t)lrow * KT, 64 * KT);
        unsigned vl[4];
        {
            const unsigned p8 = lane & 7, f0 = (unsigned)(lane >> 4);
#pragma unroll
            for (int i = 0; i < 4; ++i)
                vl[i] = (((wid & 1) * 4 + i) * 8 + (lane >> 3)) * KT + 16 * (p8 ^ ((f0 + 4u * (i & 1)) & 7u)) - i * 1024;
        }
        auto issue_w = [&](int j, unsigned slot) {
            if (dbg_nodma) return;
            const unsigned so = (unsigned)(j & 31) * (BK * NL);
            dma16x4(ldw + slot, vw[0], vw[1], vw[2], vw[3], rw, so);
            dma16x4(ldw + slot + 4096, vw[4], vw[5], vw[6], vw[7], rw, so);
            if (loc && j < NSTEP)
                dma16x4(lds0 + LDS_XL + (slot >> 1) + (wid & 1) * 4096, vl[0], vl[1], vl[2], vl[3], rxl, (unsigned)j * BK);
        };
        auto wait_w = [&](int s) {
            if (s + 2 < NS) { if (loc && s + 2 < NSTEP) { WAIT_V(12); } else { WAIT_V(8); } } else { WAIT_V(0); }
        };
        auto first_w = [&]() { issue_w(0, 0); issue_w(1, WB); issue_w(2, 2 * WB); };
        if (wid == 5) {
            first_w();
            BAR();
            WAIT_V(16);
            BAR();
            unsigned cur = 0;
#pragma unroll 1
            for (int s = 0; s < NS; ++s) {
                wait_w(s);
                BAR();
                if (s + NWS < NS) issue_w(s + NWS, cur);
                cur = next_w(cur);
            }
        } else {
            const int xw = wid & 1;
            unsigned cnt_old;
            asm volatile("global_load_dword %0, %1, off sc1" : "=v"(cnt_old) : "v"(cntp) : "memory");
            const unsigned posv = p.sched.pos[(gg + 2 * (lane >> 5)) * 32 + (lane & 31)];
            if (wid == 4) first_w();
            WAIT_V(0);
            asm volatile("" : "+v"(cnt_old));
            const unsigned call = (unsigned)__builtin_amdgcn_readfirstlane(cnt_old) + 1u;
            const unsigned pbase = call * 64u;
            const unsigned pj = posv & 3u;
            const unsigned need = pbase + (posv >> 2) + 1u;
            auto ready_count = [&](unsigned v) -> int {
                const unsigned long long m = __builtin_amdgcn_ballot_w64((int)(v - need) >= 0);
                return m == ~0ull ? NS : __builtin_ctzll(~m);
            };
            int R = dbg_nowait ? NS : 0;
            auto refresh = [&]() {
                const int r = ready_count(ctrl[pj]);
                if (r > R) R = r;
            };
            if (wid == 4) {
                __builtin_amdgcn_fence(__ATOMIC_ACQUIRE, "");
                if (lane < 8) ctrl[lane] = lane < 4 ? pbase : 0u;
                const i32x4 rf = make_rsrc(prog, 16);
                auto watch = [&](int st) {
                    if (R < NS) refresh();
                    if (R > st || st >= NS) return;
                    unsigned long long t0 = __builtin_amdgcn_s_memrealtime();
                    for (unsigned it = 1;; ++it) {
                        const unsigned v = __hip_atomic_load(prog + pj, __ATOMIC_RELAXED, __HIP_MEMORY_SCOPE_SYSTEM);
                        const int r = ready_count(v);
                        if (r > R) { R = r; ctrl[pj] = v; }
                        if (R > st) break;
                        if ((it & 15u) == 0u) {
                            const unsigned tv = (unsigned)__builtin_amdgcn_readfirstlane(
                                __hip_atomic_load(tmo, __ATOMIC_RELAXED, __HIP_MEMORY_SCOPE_AGENT));
                            if (tv == call || __builtin_amdgcn_s_memrealtime() - t0 > DEADLINE_TICKS) {
                                if (lane == 0) __hip_atomic_store(tmo, call, __ATOMIC_RELAXED, __HIP_MEMORY_SCOPE_AGENT);
                                if (lane == 0) ctrl[4] = 1u;
                                if (lane < 4) ctrl[lane] = pbase + 63u;
                                R = NS;
                                break;
                            }
                        }
                        __builtin_amdgcn_s_sleep(1);
                    }
                };
                WAIT_V(0);
                WAIT_L0();
                BAR();
                unsigned cur = 0;
#pragma unroll 1
                for (int s = -1; s < NS; ++s) {
                    watch(s + 1);
                    if (s >= 0) wait_w(s);
                    WAIT_L0();
                    BAR();
                    if (R < NS) { if (lane < 4) dma4_sys(lds0 + LDS_CTRL, (unsigned)lane * 4u, rf); }
                    if (s >= 0) {
                        if (s + NWS < NS) issue_w(s + NWS, cur);
                        cur = next_w(cur);
                    }
                }
                if (lane == 0) __hip_atomic_store(cntp, call, __ATOMIC_RELAXED, __HIP_MEMORY_SCOPE_AGENT);
            } else {
                i32x4 rxr = make_rsrc(wsl + WS_STAGE + ((size_t)(call & 1u) * NR + src) * SHARD_BYTES, SHARD_BYTES);
                if (AGK_MODE & 128) rxr.z = 0;
                const unsigned vr = xw * 8192 + lane * 16;
                const unsigned ldx = lds0 + xw * 8192;
                int nix = 0;
                unsigned isl = 0;
                auto issue_x = [&]() {
                    if (!dbg_nodma) {
                        const unsigned so = (unsigned)((gg + 2 * (nix >> 5)) * 32 + (nix & 31)) * XB;
                        dma16x4_x(ldx + isl, vr, rxr, so);
                        dma16x4_x(ldx + isl + 4096, vr, rxr, so + 4096);
                    }
                    isl = next_x(isl);
                    ++nix;
                };
                BAR();
#pragma unroll 1
                for (int s = -1; s < NS; ++s) {
                    const int st = s + 1;
                    if (st < NS) {
                        if (R < NS) refresh();
                        while (R <= st) { __builtin_amdgcn_s_sleep(1); refresh(); }
                        const int lim = s + NXS < NS ? s + NXS : NS;
                        const int l = R < lim ? R : lim;
                        while (nix < l) issue_x();
                        const int ahead = nix - 1 - st;
                        if (ahead >= 3) { WAIT_V(24); } else if (ahead == 2) { WAIT_V(16); } else if (ahead == 1) { WAIT_V(8); } else { WAIT_V(0); }
                    }
                    WAIT_L0();
                    BAR();
                    if (nix < R && nix < NS && nix <= s + NXS) issue_x();
                }
            }
        }
    }
    WAIT_L0();
    BAR();
    {
        int tz = threadIdx.x;
        asm volatile("" : "+v"(tz));
        const int ln = tz & 63, wv = tz >> 6;
        float* o = p.out + (size_t)(row_last + wv * 16 + (ln >> 5)) * NL + col0 + (ln & 31) * 4;
        f32x4 v[8];
#pragma unroll
        for (int i = 0; i < 8; ++i) {
            const unsigned row = wv * 16 + 2 * i + (ln >> 5);
            v[i] = *(LAS f32x4*)(size_t)(lds0 + row * 512 + (((unsigned)(ln & 31) ^ (row & 7u)) << 4));
        }
#pragma unroll
        for (int i = 0; i < 8; ++i) {
            float* dst = o + (size_t)i * 2 * NL;
            asm volatile("global_store_dwordx4 %0, %1, off sc1\n\ts_nop 1" :: "v"(dst), "v"(v[i]) : "memory");
        }
    }
}

}

extern "C" void kernel_launch(int rank, int nranks, ncclComm_t comm,
                              hipStream_t stream,
                              void* const* d_in, const int* in_sizes, int n_in,
                              void* d_out, int out_size,
                              const int* in_dtypes, int out_dtype,
                              void* const* d_out_all,
                              void* const* d_ws_all, size_t ws_bytes) {
    using namespace agk;
    (void)comm; (void)in_sizes; (void)n_in; (void)out_size; (void)in_dtypes; (void)out_dtype; (void)d_out_all;
    if (nranks != NR || ws_bytes < WS_NEEDED || rank < 0 || rank >= NR) {
        fprintf(stderr, "unexpected configuration\n");
        return;
    }
    static bool attr_set[NR] = {};
    if (!attr_set[rank]) {
        HIP_CHECK(hipFuncSetAttribute((const void*)ag_gemm_fused, hipFuncAttributeMaxDynamicSharedMemorySize, LDS_BYTES));
        attr_set[rank] = true;
    }
    static const Sched sched = make_sched();
    Params p;
    p.x = (const uint8_t*)d_in[0];
    p.w = (const uint8_t*)d_in[1];
    p.sx = (const float*)d_in[2];
    p.sw = (const float*)d_in[3];
    p.out = (float*)d_out;
    for (int i = 0; i < NR; ++i) p.ws[i] = (uint8_t*)d_ws_all[i];
    p.rank = rank;
    p.pad = 0;
    p.sched = sched;
    hipLaunchKernelGGL(ag_gemm_fused, dim3(NWG), dim3(512), LDS_BYTES, stream, p);
    HIP_CHECK(hipGetLastError());
}
```

```cpp
#include <hip/hip_runtime.h>
#include <rccl/rccl.h>
#include <cstdio>
#include <cstdint>

#define HIP_CHECK(expr) do { hipError_t _e = (expr); if (_e != hipSuccess) { \
    fprintf(stderr, "HIP error %s at %s:%d\n", hipGetErrorString(_e), __FILE__, __LINE__); \
    return; } } while (0)

namespace agk {

#define LAS __attribute__((address_space(3)))
typedef int i32x8 __attribute__((ext_vector_type(8)));
typedef int i32x4 __attribute__((ext_vector_type(4)));
typedef int i32x2 __attribute__((ext_vector_type(2)));
typedef unsigned u32x4 __attribute__((ext_vector_type(4)));
typedef float f32x4 __attribute__((ext_vector_type(4)));

constexpr int NR = 8;
constexpr int KT = 4096, NL = 2048, ML = 512;
constexpr int BM = 128, BN = 128, BK = 128;
constexpr int NSTEP = KT / BK;
constexpr int XB = BM * BK;
constexpr int WB = BK * BN;
constexpr int NXS = 4, NWS = 3;
constexpr int XLB = 64 * BK;
constexpr int LDS_W = NXS * XB;
constexpr int LDS_XL = LDS_W + NWS * WB;
constexpr int LDS_CTRL = LDS_XL + NWS * XLB;
constexpr int LDS_PARK = LDS_CTRL + 2048;
constexpr int LDS_BYTES = LDS_PARK + 12288;
static_assert(LDS_BYTES <= 163840, "LDS");
constexpr int NWG = 256;
constexpr unsigned SHARD_BYTES = ML * KT;
constexpr unsigned W_BYTES = KT * NL;

constexpr size_t WS_CNT = 0;
constexpr size_t WS_TMO = WS_CNT + NWG * 128;
constexpr size_t WS_PROG = WS_TMO + 128;
constexpr size_t WS_DBG = 512u << 10;
constexpr size_t WS_STAGE = 1u << 20;
constexpr size_t WS_NEEDED = WS_STAGE + 2ull * NR * SHARD_BYTES;
constexpr unsigned long long DEADLINE_TICKS = 20000000ull;

#define SK_START {0, 8, 1024, 1032}
#define SK_RATE {16, 16, 16, 16}
struct Sched {
    unsigned char ord[4][32];
    unsigned char pos[128];
};
constexpr Sched make_sched() {
    Sched s{};
    const int start[4] = SK_START;
    const int rate[4] = SK_RATE;
    int nxt[4] = {0, 0, 0, 0};
    for (int p = 0; p < 128; ++p) {
        int best = -1, bk = 0;
        for (int g = 0; g < 4; ++g) {
            if (nxt[g] >= 32) continue;
            const int key = start[g] + nxt[g] * rate[g];
            if (best < 0 || key < bk) { best = g; bk = key; }
        }
        s.ord[p & 3][p >> 2] = (unsigned char)(best * 32 + nxt[best]);
        s.pos[best * 32 + nxt[best]] = (unsigned char)p;
        ++nxt[best];
    }
    return s;
}

struct Params {
    const uint8_t* x;
    const uint8_t* w;
    const float* sx;
    const float* sw;
    float* out;
    uint8_t* ws[NR];
    int rank;
    int pad;
    Sched sched;
};

#define WAIT_V(n) asm volatile("s_waitcnt vmcnt(" #n ")" ::: "memory")
#define WAIT_L0() do { asm volatile("" ::: "memory"); __builtin_amdgcn_s_waitcnt(0xC07F); asm volatile("" ::: "memory"); } while (0)
#define BAR() do { asm volatile("" ::: "memory"); __builtin_amdgcn_s_barrier(); asm volatile("" ::: "memory"); } while (0)
#define SCHED() __builtin_amdgcn_sched_barrier(0)

__device__ __forceinline__ i32x4 make_rsrc(const void* p, unsigned bytes) {
    const unsigned long long a = (unsigned long long)p;
    i32x4 r;
    r.x = __builtin_amdgcn_readfirstlane((int)(unsigned)a);
    r.y = __builtin_amdgcn_readfirstlane((int)((unsigned)(a >> 32) & 0xffffu));
    r.z = __builtin_amdgcn_readfirstlane((int)bytes);
    r.w = 0x00020000;
    return r;
}
__device__ __forceinline__ void dma16x4(unsigned lds, unsigned v0, unsigned v1, unsigned v2, unsigned v3, i32x4 rsrc, unsigned soff) {
    asm volatile("s_mov_b32 m0, %0\n\ts_nop 0\n\t"
                 "buffer_load_dwordx4 %1, %5, %6 offen lds\n\t"
                 "buffer_load_dwordx4 %2, %5, %6 offen offset:1024 lds\n\t"
                 "buffer_load_dwordx4 %3, %5, %6 offen offset:2048 lds\n\t"
                 "buffer_load_dwordx4 %4, %5, %6 offen offset:3072 lds"
                 :: "s"(lds), "v"(v0), "v"(v1), "v"(v2), "v"(v3), "s"(rsrc), "s"(soff) : "memory");
}
#define X_POL ""
__device__ __forceinline__ void dma16x4_x(unsigned lds, unsigned v0, i32x4 rsrc, unsigned soff) {
    asm volatile("s_mov_b32 m0, %0\n\ts_nop 0\n\t"
                 "buffer_load_dwordx4 %1, %2, %3 offen " X_POL " lds\n\t"
                 "buffer_load_dwordx4 %1, %2, %3 offen offset:1024 " X_POL " lds\n\t"
                 "buffer_load_dwordx4 %1, %2, %3 offen offset:2048 " X_POL " lds\n\t"
                 "buffer_load_dwordx4 %1, %2, %3 offen offset:3072 " X_POL " lds"
                 :: "s"(lds), "v"(v0), "s"(rsrc), "s"(soff) : "memory");
}
__device__ __forceinline__ void dma4_sys(unsigned lds, unsigned voff, i32x4 rsrc) {
    asm volatile("s_mov_b32 m0, %0\n\ts_nop 0\n\tbuffer_load_dword %1, %2, 0 offen sc0 sc1 lds"
                 :: "s"(lds), "v"(voff), "s"(rsrc) : "memory");
}
__device__ __forceinline__ void ld16_asm(u32x4& d, unsigned voff, i32x4 rsrc, unsigned soff) {
    asm volatile("buffer_load_dwordx4 %0, %1, %2, %3 offen" : "=v"(d) : "v"(voff), "s"(rsrc), "s"(soff) : "memory");
}
__device__ __forceinline__ void st16_sys(const u32x4& d, unsigned voff, i32x4 rsrc, unsigned soff) {
    asm volatile("buffer_store_dwordx4 %0, %1, %2, %3 offen sc0 sc1\n\ts_nop 1"
                 :: "v"(d), "v"(voff), "s"(rsrc), "s"(soff) : "memory");
}
struct Frags { i32x8 x[4]; i32x8 w[4]; };
__device__ __forceinline__ void put4(i32x8& d, int i, i32x4 v) { d[4 * i] = v.x; d[4 * i + 1] = v.y; d[4 * i + 2] = v.z; d[4 * i + 3] = v.w; }
__device__ __forceinline__ void put2(i32x8& d, int i, i32x2 v) { d[2 * i] = v.x; d[2 * i + 1] = v.y; }

__device__ __forceinline__ void read_w1(unsigned aw, Frags& f, int nb, int q) {
    put2(f.w[nb], q, __builtin_amdgcn_ds_read_tr8_b64_v2i32((LAS i32x2*)(size_t)((aw ^ (unsigned)(nb << 4)) + ((q >> 1) * 64 + (q & 1) * 8) * BN)));
}
__device__ __forceinline__ void read_x1(unsigned ax, Frags& f, int mb, int q) {
    put4(f.x[mb], q, *(LAS i32x4*)(size_t)((q ? (ax ^ 64u) : ax) + mb * 2048));
}
__device__ __forceinline__ void read_nth(unsigned ax, unsigned aw, Frags& f, int r) {
    if (r < 4) read_w1(aw, f, 0, r);
    else if (r < 12) read_x1(ax, f, (r - 4) >> 1, (r - 4) & 1);
    else read_w1(aw, f, 1 + ((r - 12) >> 2), (r - 12) & 3);
}
__device__ __forceinline__ void mma1(f32x4 (&acc)[4][4], const Frags& f, int nb, int mb) {
    acc[nb][mb] = __builtin_amdgcn_mfma_scale_f32_16x16x128_f8f6f4(f.w[nb], f.x[mb], acc[nb][mb], 1, 1, 0, 0, 0, 0);
}
__device__ __forceinline__ unsigned next_x(unsigned soff) { return soff == (NXS - 1) * XB ? 0u : soff + XB; }
__device__ __forceinline__ unsigned next_w(unsigned soff) { return soff == (NWS - 1) * WB ? 0u : soff + WB; }

__device__ __forceinline__ f32x4 epi(f32x4 a, float scale, bool bad) {
    const float sc = bad ? __builtin_nanf("") : scale;
    return __builtin_elementwise_maximum(a * sc, (f32x4){0.f, 0.f, 0.f, 0.f});
}
#define MID_ST 0
__device__ __forceinline__ void st_out(float* dst, f32x4 v) {
    asm volatile("global_store_dwordx4 %0, %1, off\n\ts_nop 1" :: "v"(dst), "v"(v) : "memory");
}
__device__ __forceinline__ float* store_tile(f32x4 (&acc)[4][4], f32x4 (&pend)[8], unsigned park, float* out, int row0, int col0,
                                             float scale, bool bad) {
    int tz = threadIdx.x;
    asm volatile("" : "+v"(tz));
    const int ln = tz & 63, wv = tz >> 6;
    const int wm = wv & 1, wn = wv >> 1;
    float* o = out + (size_t)(row0 + wm * 64 + (ln & 15)) * NL + (col0 + wn * 64 + 4 * (ln >> 4));
#pragma unroll
    for (int mb = 0; mb < 4; ++mb)
#pragma unroll
        for (int nb = 0; nb < 4; ++nb) {
            const f32x4 v = epi(acc[nb][mb], scale, bad);
            if (mb == 0 || (mb == 1 && nb == 3)) st_out(o + (size_t)mb * 16 * NL + nb * 16, v);
            else if (mb == 1) *(LAS f32x4*)(size_t)(park + nb * 1024) = v;
            else pend[(mb - 2) * 4 + nb] = v;
            acc[nb][mb] = (f32x4){0.f, 0.f, 0.f, 0.f};
        }
    return o;
}

#define PRE_BAR 2
#define AGK_MODE 0
#define OUT_ST 2

#define TRACE_STEP(s)

__global__ void __launch_bounds__(512, 2) ag_gemm_fused(Params p) {
    extern __shared__ __attribute__((aligned(16))) char smem[];
    constexpr bool dbg_nowait = (AGK_MODE & 1) != 0, dbg_nopush = (AGK_MODE & 2) != 0;
    constexpr bool dbg_nodma = (AGK_MODE & 4) != 0, dbg_nolocal = (AGK_MODE & 16) != 0;
    if (AGK_MODE & 64) return;
    const int tid = threadIdx.x, lane = tid & 63;
    const int wid = __builtin_amdgcn_readfirstlane(tid >> 6);
    const int b = blockIdx.x;
    const int ct = b & 7, rt = b >> 3;
    const int rank = p.rank;
    uint8_t* wsl = p.ws[rank];
    unsigned* cntp = (unsigned*)(wsl + WS_CNT + (size_t)b * 128);

    if (rt < 4) {
#define PEER_ROT 0
        const int pq = (ct + PEER_ROT * rt) & 7;
        if (pq == NR - 1 || dbg_nopush) return;
        const int peer = (rank + 1 + pq) & (NR - 1);
        uint8_t* wsp = p.ws[peer];
        unsigned cnt_old;
        asm volatile("global_load_dword %0, %1, off sc1" : "=v"(cnt_old) : "v"(cntp) : "memory");
        const i32x4 rs = make_rsrc(p.x, SHARD_BYTES);
        const unsigned vs = (wid * 16 + (lane >> 3)) * KT + (lane & 7) * 16;
        const unsigned vd0 = (wid * 16 + (lane >> 3)) * BK + (((unsigned)lane & 7u) ^ ((unsigned)lane >> 4)) * 16;
        const unsigned vd1 = (wid * 16 + 8 + (lane >> 3)) * BK + (((unsigned)lane & 7u) ^ (4u + ((unsigned)lane >> 4))) * 16;
        const unsigned* ow = (const unsigned*)p.sched.ord[rt];
        u32x4 pb[4][2];
        auto push_load = [&](unsigned u, u32x4 (&d)[2]) {
            const unsigned so = (u >> 5) * (BM * KT) + (u & 31u) * BK;
            ld16_asm(d[0], vs, rs, so);
            ld16_asm(d[1], vs, rs, so + 8 * KT);
        };
        unsigned wc = ow[0];
        push_load(wc & 255u, pb[0]);
        push_load((wc >> 8) & 255u, pb[1]);
        push_load((wc >> 16) & 255u, pb[2]);
        push_load(wc >> 24, pb[3]);
        WAIT_V(8);
        asm volatile("" : "+v"(cnt_old));
        const unsigned call = (unsigned)__builtin_amdgcn_readfirstlane(cnt_old) + 1u;
        const unsigned pbase = call * 64u;
        const i32x4 rd = make_rsrc(wsp + WS_STAGE + ((size_t)(call & 1u) * NR + rank) * SHARD_BYTES, SHARD_BYTES);
        unsigned* flag = (unsigned*)(wsp + WS_PROG + (size_t)rank * 128) + rt;
        auto publish = [&](int done) {
            BAR();
            if (tid == 0) __hip_atomic_store(flag, pbase + (unsigned)done, __ATOMIC_RELAXED, __HIP_MEMORY_SCOPE_SYSTEM);
        };
        auto push_store = [&](unsigned u, u32x4 (&d)[2]) {
            st16_sys(d[0], vd0, rd, u * XB);
            st16_sys(d[1], vd1, rd, u * XB);
        };
#define PUSH_D 1
        {
            const unsigned wn = ow[1];
            WAIT_V(6); push_store(wc & 255u, pb[0]); push_load(wn & 255u, pb[0]);
            WAIT_V(8); push_store((wc >> 8) & 255u, pb[1]); push_load((wn >> 8) & 255u, pb[1]);
            WAIT_V(6); publish(1);
            WAIT_V(10); push_store((wc >> 16) & 255u, pb[2]); push_load((wn >> 16) & 255u, pb[2]);
            WAIT_V(6); publish(2);
            WAIT_V(12); push_store(wc >> 24, pb[3]); push_load(wn >> 24, pb[3]);
            WAIT_V(6); publish(3);
            wc = wn;
        }
#pragma unroll 1
        for (int t = 4; t < NSTEP - 4; t += 4) {
            const unsigned wn = ow[(t >> 2) + 1];
#pragma unroll
            for (int i = 0; i < 4; ++i) {
                push_store((wc >> (8 * i)) & 255u, pb[i]);
                push_load((wn >> (8 * i)) & 255u, pb[i]);
                WAIT_V(6); publish(t + i);
            }
            wc = wn;
        }
        push_store(wc & 255u, pb[0]);          WAIT_V(4); publish(28);
        push_store((wc >> 8) & 255u, pb[1]);   WAIT_V(2); publish(29);
        push_store((wc >> 16) & 255u, pb[2]);  WAIT_V(2); publish(30);
        push_store(wc >> 24, pb[3]);           WAIT_V(2); publish(31);
        WAIT_V(0); publish(32);
        if (tid == 0) __hip_atomic_store(cntp, call, __ATOMIC_RELAXED, __HIP_MEMORY_SCOPE_AGENT);
        return;
    }

    const int idx = rt - 4;
    const int sh = 1 + (idx >> 2), gg = (idx >> 1) & 1, half = idx & 1;
    const int src = (rank + sh) & (NR - 1);
    const bool loc = idx < 16 && !dbg_nolocal;
    const int lrow = idx * 32 & ~63;
    constexpr int NS = 2 * NSTEP;
    const int col0 = ct * 256 + half * BN;
    const unsigned lds0 = (unsigned)(uintptr_t)(LAS char*)smem;
    volatile LAS unsigned* ctrl = (volatile LAS unsigned*)(LAS char*)(smem + LDS_CTRL);

    const i32x4 rw = make_rsrc(p.w, W_BYTES);
    unsigned* prog = (unsigned*)(wsl + WS_PROG + (size_t)src * 128);
    unsigned* tmo = (unsigned*)(wsl + WS_TMO);
    const int row_last = src * ML + (gg + 2) * BM;
    float scale = 0.f;

    if (wid < 4) {
        scale = __builtin_bit_cast(float, __builtin_amdgcn_readfirstlane(__builtin_bit_cast(int, p.sx[0] * p.sw[0])));
        {
            const i32x4 rxp = make_rsrc(p.x, SHARD_BYTES);
            const unsigned vw = (unsigned)(tid >> 4) * NL + ct * 256 + (tid & 15) * 16;
            const unsigned vx = (unsigned)(tid >> 3) * KT + (tid & 7) * 16;
            u32x4 d0, d1, d2, d3;
            for (int ks = idx; ks < NSTEP; ks += 28) {
                const unsigned so = (unsigned)ks * (BK * NL), sx = (unsigned)ks * BK;
#pragma unroll
                for (int r = 0; r < 8; r += 4) {
                    ld16_asm(d0, vw, rw, so + (r + 0) * 16 * NL); ld16_asm(d1, vw, rw, so + (r + 1) * 16 * NL);
                    ld16_asm(d2, vw, rw, so + (r + 2) * 16 * NL); ld16_asm(d3, vw, rw, so + (r + 3) * 16 * NL);
                }
#pragma unroll
                for (int r = 0; r < 16; r += 4) {
                    ld16_asm(d0, vx, rxp, sx + (r + 0) * 32 * KT); ld16_asm(d1, vx, rxp, sx + (r + 1) * 32 * KT);
                    ld16_asm(d2, vx, rxp, sx + (r + 2) * 32 * KT); ld16_asm(d3, vx, rxp, sx + (r + 3) * 32 * KT);
                }
            }
            WAIT_V(0);
            asm volatile("" :: "v"(d0), "v"(d1), "v"(d2), "v"(d3));
        }
        const int wm = wid & 1, wn = wid >> 1;
        unsigned axl, awl, axs;
        {
            const unsigned m = lane & 15, gq = lane >> 4, q = m >> 1, h = m & 1;
            axl = lds0 + (wm * 64 + m) * BK + 16 * (gq ^ (m >> 1));
            awl = lds0 + LDS_W + (16 * gq + q) * BN + 16 * ((unsigned)(wn * 4) ^ ((q >> 1) | ((gq & 1) << 2))) + 8 * h;
            axs = lds0 + LDS_XL + (wm * 32 + m) * BK + 16 * (gq ^ (m >> 1));
        }
        f32x4 acc[4][4];
#pragma unroll
        for (int i = 0; i < 4; ++i)
#pragma unroll
            for (int j = 0; j < 4; ++j) acc[i][j] = (f32x4){0.f, 0.f, 0.f, 0.f};
        Frags fa, fb;
        f32x4 pend[8];
        float* pend_o;
        BAR();
        BAR();
        unsigned cx = 0, cw = 0;
        auto step = [&](int s, Frags& fc, Frags& fn) {
            cx = next_x(cx); cw = next_w(cw);
            const unsigned ax = axl + cx, aw = awl + cw;
            WAIT_L0();
            SCHED();
#pragma unroll
            for (int i = 0; i < PRE_BAR; ++i) { mma1(acc, fc, i >> 2, i & 3); SCHED(); }
            BAR();
            TRACE_STEP(s);
            SCHED();
#pragma unroll
            for (int i = PRE_BAR; i < 16; ++i) {
                mma1(acc, fc, i >> 2, i & 3);
                if (2 * (i - PRE_BAR) < 24) { read_nth(ax, aw, fn, 2 * (i - PRE_BAR)); read_nth(ax, aw, fn, 2 * (i - PRE_BAR) + 1); }
                SCHED();
            }
        };
        if (loc) {
            f32x4 accl[4][2];
#pragma unroll
            for (int i = 0; i < 4; ++i) accl[i][0] = accl[i][1] = (f32x4){0.f, 0.f, 0.f, 0.f};
            i32x8 xl[2];
            auto read_xl = [&](unsigned a, int mb, int q) {
                put4(xl[mb], q, *(LAS i32x4*)(size_t)((q ? (a ^ 64u) : a) + mb * 2048));
            };
#pragma unroll
            for (int r = 0; r < 24; ++r) read_nth(axl, awl, fa, r);
            read_xl(axs, 0, 0); read_xl(axs, 0, 1); read_xl(axs, 1, 0); read_xl(axs, 1, 1);
            auto cstep = [&](int s, Frags& fc, Frags& fn) {
                cx = next_x(cx); cw = next_w(cw);
                const unsigned ax = axl + cx, aw = awl + cw, al = axs + (cw >> 1);
                WAIT_L0();
                SCHED();
#pragma unroll
                for (int i = 0; i < PRE_BAR; ++i) { mma1(acc, fc, i >> 2, i & 3); SCHED(); }
                BAR();
                TRACE_STEP(s);
                SCHED();
#pragma unroll
                for (int g2 = 0; g2 < 8; ++g2) {
                    accl[g2 >> 1][g2 & 1] = __builtin_amdgcn_mfma_scale_f32_16x16x128_f8f6f4(fc.w[g2 >> 1], xl[g2 & 1], accl[g2 >> 1][g2 & 1], 1, 1, 0, 0, 0, 0);
                    read_nth(ax, aw, fn, 2 * g2); read_nth(ax, aw, fn, 2 * g2 + 1);
                    SCHED();
                }
#pragma unroll
                for (int i = PRE_BAR; i < 16; ++i) {
                    mma1(acc, fc, i >> 2, i & 3);
                    const int g2 = i - PRE_BAR;
                    if (g2 < 4) { read_nth(ax, aw, fn, 16 + 2 * g2); read_nth(ax, aw, fn, 17 + 2 * g2); }
                    else if (g2 < 6) { read_xl(al, g2 - 4, 0); read_xl(al, g2 - 4, 1); }
                    SCHED();
                }
            };
#pragma unroll 1
            for (int s = 0; s < NSTEP; s += 2) {
                cstep(s, fa, fb);
                cstep(s + 1, fb, fa);
            }
            int tz = threadIdx.x;
            asm volatile("" : "+v"(tz));
            const int ln = tz & 63;
            float* o = p.out + (size_t)(rank * ML + lrow + wm * 32 + (ln & 15)) * NL + (col0 + wn * 64 + 4 * (ln >> 4));
            const unsigned pk = lds0 + LDS_XL + (tz >> 6) * 6144 + ln * 16;
#pragma unroll
            for (int i = 0; i < 8; ++i) {
                const f32x4 v = epi(accl[i & 3][i >> 2], scale, false);
                if (i < 2) st_out(o + i * 16, v);
                else *(LAS f32x4*)(size_t)(pk + (i - 2) * 1024) = v;
            }
        } else {
#pragma unroll
            for (int r = 0; r < 24; ++r) read_nth(axl, awl, fa, r);
#pragma unroll 1
            for (int s = 0; s < NSTEP; s += 2) {
                step(s, fa, fb);
                step(s + 1, fb, fa);
            }
        }
        int tp = threadIdx.x;
        asm volatile("" : "+v"(tp));
        const unsigned park = lds0 + LDS_PARK + (tp >> 6) * 3072 + (tp & 63) * 16;
        {
            const bool bad = __builtin_amdgcn_readfirstlane((int)ctrl[4]) != 0;
            pend_o = store_tile(acc, pend, park, p.out, src * ML + gg * BM, col0, scale, bad);
        }
#pragma unroll 1
        for (int s = NSTEP; s < NS; s += 2) {
            step(s, fa, fb);
            const int k = s - NSTEP;
            if (k >= 2 && k <= 12) {
                float* po = pend_o + (size_t)32 * NL;
                if (k == 2) { st_out(po, pend[0]); st_out(po + 16, pend[1]); }
                else if (k == 4) { st_out(po + 32, pend[2]); st_out(po + 48, pend[3]); }
                else if (k == 6) { st_out(po + 16 * NL, pend[4]); st_out(po + 16 * NL + 16, pend[5]); }
                else if (k == 8) { st_out(po + 16 * NL + 32, pend[6]); st_out(po + 16 * NL + 48, pend[7]); }
                else {
                    if (k == 10) {
                        const f32x4 v0 = *(LAS f32x4*)(size_t)(park), v1 = *(LAS f32x4*)(size_t)(park + 1024);
                        st_out(pend_o + 16 * NL, v0);
                        st_out(pend_o + 16 * NL + 16, v1);
                    } else {
                        const f32x4 v2 = *(LAS f32x4*)(size_t)(park + 2048);
                        st_out(pend_o + 16 * NL + 32, v2);
                    }
                    WAIT_L0();
                }
            } else if (loc && k >= 14 && k <= 18) {
                int tz = threadIdx.x;
                asm volatile("" : "+v"(tz));
                const int ln = tz & 63, i = k - 12;
                float* o = p.out + (size_t)(rank * ML + lrow + wm * 32 + (i >> 2) * 16 + (ln & 15)) * NL
                                 + (col0 + wn * 64 + (i & 3) * 16 + 4 * (ln >> 4));
                const unsigned pk = lds0 + LDS_XL + (tz >> 6) * 6144 + ln * 16 + (i - 2) * 1024;
                const f32x4 v0 = *(LAS f32x4*)(size_t)(pk), v1 = *(LAS f32x4*)(size_t)(pk + 1024);
                st_out(o, v0);
                st_out(o + 16, v1);
                WAIT_L0();
            }
            step(s + 1, fb, fa);
        }
        const bool bad = __builtin_amdgcn_readfirstlane((int)ctrl[4]) != 0;
        int tq = threadIdx.x;
        asm volatile("" : "+v"(tq));
        const unsigned lq = tq & 63;
#pragma unroll
        for (int mb = 0; mb < 4; ++mb) {
            const unsigned row = wm * 64 + mb * 16 + (lq & 15);
#pragma unroll
            for (int nb = 0; nb < 4; ++nb) {
                const unsigned chunk = wn * 16 + nb * 4 + (lq >> 4);
                *(LAS f32x4*)(size_t)(lds0 + row * 512 + ((chunk ^ (row & 7u)) << 4)) = epi(acc[nb][mb], scale, bad);
            }
        }
    } else {
        unsigned vw[8];
        if (wid < 6) {
            const unsigned p8 = lane & 7, f0 = (unsigned)(lane >> 4) & 3u;
#pragma unroll
            for (int i = 0; i < 8; ++i) {
                const unsigned pi = (wid - 4) * 8 + i;
                vw[i] = (pi * 8 + (lane >> 3)) * NL + col0 + 16 * (p8 ^ (f0 | (((pi >> 1) & 1u) << 2))) - (i & 3) * 1024;
            }
        }
        const unsigned ldw = lds0 + LDS_W + (wid - 4) * 8192;
        const i32x4 rxl = make_rsrc(p.x + (size_t)lrow * KT, 64 * KT);
        unsigned vl[4];
        {
            const unsigned p8 = lane & 7, f0 = (unsigned)(lane >> 4);
#pragma unroll
            for (int i = 0; i < 4; ++i)
                vl[i] = (((wid & 1) * 4 + i) * 8 + (lane >> 3)) * KT + 16 * (p8 ^ ((f0 + 4u * (i & 1)) & 7u)) - i * 1024;
        }
        auto issue_w = [&](int j, unsigned slot) {
            if (dbg_nodma) return;
            const unsigned so = (unsigned)(j & 31) * (BK * NL);
            dma16x4(ldw + slot, vw[0], vw[1], vw[2], vw[3], rw, so);
            dma16x4(ldw + slot + 4096, vw[4], vw[5], vw[6], vw[7], rw, so);
            if (loc && j < NSTEP)
                dma16x4(lds0 + LDS_XL + (slot >> 1) + (wid & 1) * 4096, vl[0], vl[1], vl[2], vl[3], rxl, (unsigned)j * BK);
        };
        auto wait_w = [&](int s) {
            if (s + 2 < NS) { if (loc && s + 2 < NSTEP) { WAIT_V(12); } else { WAIT_V(8); } } else { WAIT_V(0); }
        };
        auto first_w = [&]() { issue_w(0, 0); issue_w(1, WB); issue_w(2, 2 * WB); };
        if (wid == 5) {
            first_w();
            BAR();
            WAIT_V(16);
            BAR();
            unsigned cur = 0;
#pragma unroll 1
            for (int s = 0; s < NS; ++s) {
                wait_w(s);
                BAR();
                if (s + NWS < NS) issue_w(s + NWS, cur);
                cur = next_w(cur);
            }
        } else {
            const int xw = wid & 1;
            unsigned cnt_old;
            asm volatile("global_load_dword %0, %1, off sc1" : "=v"(cnt_old) : "v"(cntp) : "memory");
            const unsigned posv = p.sched.pos[(gg + 2 * (lane >> 5)) * 32 + (lane & 31)];
            if (wid == 4) first_w();
            WAIT_V(0);
            asm volatile("" : "+v"(cnt_old));
            const unsigned call = (unsigned)__builtin_amdgcn_readfirstlane(cnt_old) + 1u;
            const unsigned pbase = call * 64u;
            const unsigned pj = posv & 3u;
            const unsigned need = pbase + (posv >> 2) + 1u;
            auto ready_count = [&](unsigned v) -> int {
                const unsigned long long m = __builtin_amdgcn_ballot_w64((int)(v - need) >= 0);
                return m == ~0ull ? NS : __builtin_ctzll(~m);
            };
            int R = dbg_nowait ? NS : 0;
            auto refresh = [&]() {
                const int r = ready_count(ctrl[pj]);
                if (r > R) R = r;
            };
            if (wid == 4) {
                __builtin_amdgcn_fence(__ATOMIC_ACQUIRE, "");
                if (lane < 8) ctrl[lane] = lane < 4 ? pbase : 0u;
                const i32x4 rf = make_rsrc(prog, 16);
                auto watch = [&](int st) {
                    if (R < NS) refresh();
                    if (R > st || st >= NS) return;
                    unsigned long long t0 = __builtin_amdgcn_s_memrealtime();
                    for (unsigned it = 1;; ++it) {
                        const unsigned v = __hip_atomic_load(prog + pj, __ATOMIC_RELAXED, __HIP_MEMORY_SCOPE_SYSTEM);
                        const int r = ready_count(v);
                        if (r > R) { R = r; ctrl[pj] = v; }
                        if (R > st) break;
                        if ((it & 15u) == 0u) {
                            const unsigned tv = (unsigned)__builtin_amdgcn_readfirstlane(
                                __hip_atomic_load(tmo, __ATOMIC_RELAXED, __HIP_MEMORY_SCOPE_AGENT));
                            if (tv == call || __builtin_amdgcn_s_memrealtime() - t0 > DEADLINE_TICKS) {
                                if (lane == 0) __hip_atomic_store(tmo, call, __ATOMIC_RELAXED, __HIP_MEMORY_SCOPE_AGENT);
                                if (lane == 0) ctrl[4] = 1u;
                                if (lane < 4) ctrl[lane] = pbase + 63u;
                                R = NS;
                                break;
                            }
                        }
                        __builtin_amdgcn_s_sleep(1);
                    }
                };
                WAIT_V(0);
                WAIT_L0();
                BAR();
                unsigned cur = 0;
#pragma unroll 1
                for (int s = -1; s < NS; ++s) {
                    watch(s + 1);
                    if (s >= 0) wait_w(s);
                    WAIT_L0();
                    BAR();
                    if (R < NS) { if (lane < 4) dma4_sys(lds0 + LDS_CTRL, (unsigned)lane * 4u, rf); }
                    if (s >= 0) {
                        if (s + NWS < NS) issue_w(s + NWS, cur);
                        cur = next_w(cur);
                    }
                }
                if (lane == 0) __hip_atomic_store(cntp, call, __ATOMIC_RELAXED, __HIP_MEMORY_SCOPE_AGENT);
            } else {
                i32x4 rxr = make_rsrc(wsl + WS_STAGE + ((size_t)(call & 1u) * NR + src) * SHARD_BYTES, SHARD_BYTES);
                if (AGK_MODE & 128) rxr.z = 0;
                const unsigned vr = xw * 8192 + lane * 16;
                const unsigned ldx = lds0 + xw * 8192;
                int nix = 0;
                unsigned isl = 0;
                auto issue_x = [&]() {
                    if (!dbg_nodma) {
                        const unsigned so = (unsigned)((gg + 2 * (nix >> 5)) * 32 + (nix & 31)) * XB;
                        dma16x4_x(ldx + isl, vr, rxr, so);
                        dma16x4_x(ldx + isl + 4096, vr, rxr, so + 4096);
                    }
                    isl = next_x(isl);
                    ++nix;
                };
                BAR();
#pragma unroll 1
                for (int s = -1; s < NS; ++s) {
                    const int st = s + 1;
                    if (st < NS) {
                        if (R < NS) refresh();
                        while (R <= st) { __builtin_amdgcn_s_sleep(1); refresh(); }
                        const int lim = s + NXS < NS ? s + NXS : NS;
                        const int l = R < lim ? R : lim;
                        while (nix < l) issue_x();
                        const int ahead = nix - 1 - st;
                        if (ahead >= 3) { WAIT_V(24); } else if (ahead == 2) { WAIT_V(16); } else if (ahead == 1) { WAIT_V(8); } else { WAIT_V(0); }
                    }
                    WAIT_L0();
                    BAR();
                    if (nix < R && nix < NS && nix <= s + NXS) issue_x();
                }
            }
        }
    }
    WAIT_L0();
    BAR();
    {
        int tz = threadIdx.x;
        asm volatile("" : "+v"(tz));
        const int ln = tz & 63, wv = tz >> 6;
        float* o = p.out + (size_t)(row_last + wv * 16 + (ln >> 5)) * NL + col0 + (ln & 31) * 4;
        f32x4 v[8];
#pragma unroll
        for (int i = 0; i < 8; ++i) {
            const unsigned row = wv * 16 + 2 * i + (ln >> 5);
            v[i] = *(LAS f32x4*)(size_t)(lds0 + row * 512 + (((unsigned)(ln & 31) ^ (row & 7u)) << 4));
        }
#pragma unroll
        for (int i = 0; i < 8; ++i) {
            float* dst = o + (size_t)i * 2 * NL;
            asm volatile("global_store_dwordx4 %0, %1, off sc1\n\ts_nop 1" :: "v"(dst), "v"(v[i]) : "memory");
        }
    }
}

}

extern "C" void kernel_launch(int rank, int nranks, ncclComm_t comm,
                              hipStream_t stream,
                              void* const* d_in, const int* in_sizes, int n_in,
                              void* d_out, int out_size,
                              const int* in_dtypes, int out_dtype,
                              void* const* d_out_all,
                              void* const* d_ws_all, size_t ws_bytes) {
    using namespace agk;
    (void)comm; (void)in_sizes; (void)n_in; (void)out_size; (void)in_dtypes; (void)out_dtype; (void)d_out_all;
    if (nranks != NR || ws_bytes < WS_NEEDED || rank < 0 || rank >= NR) {
        fprintf(stderr, "unexpected configuration\n");
        return;
    }
    static bool attr_set[NR] = {};
    if (!attr_set[rank]) {
        HIP_CHECK(hipFuncSetAttribute((const void*)ag_gemm_fused, hipFuncAttributeMaxDynamicSharedMemorySize, LDS_BYTES));
        attr_set[rank] = true;
    }
    static const Sched sched = make_sched();
    Params p;
    p.x = (const uint8_t*)d_in[0];
    p.w = (const uint8_t*)d_in[1];
    p.sx = (const float*)d_in[2];
    p.sw = (const float*)d_in[3];
    p.out = (float*)d_out;
    for (int i = 0; i < NR; ++i) p.ws[i] = (uint8_t*)d_ws_all[i];
    p.rank = rank;
    p.pad = 0;
    p.sched = sched;
    hipLaunchKernelGGL(ag_gemm_fused, dim3(NWG), dim3(512), LDS_BYTES, stream, p);
    HIP_CHECK(hipGetLastError());
}
```
